```python
import numpy as np

kernel_name = "gpu_ag_gemm_m4096_k4096_n16384_e4m3e4m3_dqsilu"


ABSMAX_RTOL = 0.03

COMPUTE_DTYPE = "float32"
EPILOGUE = "dequant_silu"
IN_QUANT = None
OUT_QUANT = None
IN_FP8 = "e4m3e4m3"
IN_DTYPES = {"x": "e4m3", "w_mat": "e4m3",
             "scale_x": "float32", "scale_w": "float32"}
WORLD_SIZE = 8

def _snap_fp8(x, mant_bits, max_abs, min_normal):
    a = np.abs(x.astype(np.float64))
    _m, _e = np.frexp(a)
    q = np.where(a >= min_normal, np.ldexp(1.0, _e - 1 - mant_bits),
                 min_normal * 2.0 ** -mant_bits)
    snapped = np.minimum(np.round(a / q) * q, max_abs)
    return (np.sign(x) * snapped).astype(np.float32)

def setup_inputs():

    x = _snap_fp8(np.random.randn(4096, 4096).astype(np.float32), 3, 448.0, 2.0 ** -6)
    w_mat = _snap_fp8(np.random.randn(4096, 16384).astype(np.float32), 3, 448.0, 2.0 ** -6)
    scale_x = (0.02 * (1.0 + np.random.rand(1))).astype(np.float32)
    scale_w = (0.02 * (1.0 + np.random.rand(1))).astype(np.float32)
    return {"x": x, "w_mat": w_mat, "scale_x": scale_x, "scale_w": scale_w}

def reference(x, w_mat, scale_x, scale_w):
    acc = (x @ w_mat).astype(np.float32)
    y = (acc * (scale_x * scale_w)).astype(np.float32)
    return (y / (1.0 + np.exp(-np.clip(y, -60.0, 60.0)))).astype(np.float32)
```

```cpp
#include <hip/hip_runtime.h>
#include <rccl/rccl.h>
#include <cstdio>

#define HIP_CHECK(expr) do { hipError_t _e = (expr); if (_e != hipSuccess) { \
    fprintf(stderr, "HIP error %s at %s:%d\n", hipGetErrorString(_e), __FILE__, __LINE__); \
    return; } } while (0)

typedef int      v8i   __attribute__((ext_vector_type(8)));
typedef int      v2i   __attribute__((ext_vector_type(2)));
typedef unsigned u32x4 __attribute__((ext_vector_type(4)));
typedef float    v4f   __attribute__((ext_vector_type(4)));
#define LAS __attribute__((address_space(3)))
#define CAS __attribute__((address_space(4)))
#define DI __device__ __forceinline__

constexpr int      NR        = 8;
constexpr int      KD        = 4096, NL = 2048, MS = 512;
constexpr unsigned SHARD     = MS * KD;
constexpr unsigned BLKB      = 16 * KD;
constexpr int      NBLK      = MS / 16;
constexpr size_t   STAGE_B   = (size_t)NR * SHARD;
constexpr size_t   FLAG_OFF  = 2 * STAGE_B;
constexpr unsigned FLAG_STR  = NBLK * 128;
constexpr size_t   CNT_OFF   = FLAG_OFF + (NR + 1) * FLAG_STR;
constexpr size_t   TMO_OFF   = CNT_OFF + 256 * 128;
constexpr size_t   DIAG_OFF  = TMO_OFF + 4096;
constexpr size_t   WS_NEED   = DIAG_OFF + 256 * 8 * 64 * 8;
constexpr int      NSEND     = 32, GRID = 256, THREADS = 512;
constexpr unsigned PROG_OFF  = 131072;
constexpr unsigned LDS_BYTES = PROG_OFF + 64;
#define DEADLINE_TICKS_LIT 100000000
constexpr int      SRD3      = 0x00020000;

#define PACE 0
#define OUT_AUX 17
#define MAPMODE 0

struct Params {
  unsigned char* ws[NR];
  const unsigned char* x;
  const unsigned char* w;
  const float* sx;
  const float* sw;
  float* out;
  int rank;
  int pad;
};

DI unsigned rfl(unsigned v) { return __builtin_amdgcn_readfirstlane(v); }
DI __amdgpu_buffer_rsrc_t mk_rsrc(const void* p, unsigned bytes) {
  unsigned long long a = (unsigned long long)p;
  unsigned lo = rfl((unsigned)a), hi = rfl((unsigned)(a >> 32));
  return __builtin_amdgcn_make_buffer_rsrc((void*)(((unsigned long long)hi << 32) | lo), (short)0, (int)bytes, SRD3);
}

DI u32x4 mk_srd(const void* p, unsigned bytes) {
  const unsigned long long a = (unsigned long long)p;
  return u32x4{rfl((unsigned)a), rfl((unsigned)(a >> 32) & 0xffffu), bytes, (unsigned)SRD3};
}

#define STAMP(slot) do {} while (0)

#define VMCNT(n)  asm volatile("s_waitcnt vmcnt(" #n ")" ::: "memory")
#define LGKM(n)   asm volatile("s_waitcnt lgkmcnt(" #n ")" ::: "memory")
#define LGKM0()   LGKM(0)
#define BARRIER() do { __builtin_amdgcn_s_barrier(); asm volatile("" ::: "memory"); } while (0)
#define SCHED()   __builtin_amdgcn_sched_barrier(0)

#define CLOB10(p) "v" #p "0", "v" #p "1", "v" #p "2", "v" #p "3", "v" #p "4", "v" #p "5", "v" #p "6", "v" #p "7", "v" #p "8", "v" #p "9"
#define CLOB_V "v28", "v29", CLOB10(3), CLOB10(4), CLOB10(5), CLOB10(6), CLOB10(7), CLOB10(8), CLOB10(9), CLOB10(10), CLOB10(11), CLOB10(12),   \
    CLOB10(13), CLOB10(14), CLOB10(15), CLOB10(16), CLOB10(17), CLOB10(18), CLOB10(19), CLOB10(20), CLOB10(21), CLOB10(22), CLOB10(23),        \
    CLOB10(24), "v250", "v251", "v252", "v253", "v254", "v255"
#define STR_(x) #x
#define STR(x) STR_(x)
#define OUT_BITS " sc0 sc1"
#define ASTAMP ".macro ASTAMP off, idx\n.endm\n.macro CSTAMP off\n.endm\n"
#define FAKE_AV ""
#define STAMPN 1
#define COMPUTE_ASM ".set STAMPN, " STR(STAMPN) "\n" ASTAMP R"(
.macro WDMA s
  .irp j, 0,1,2,3,4,5,6,7
    s_add_u32 m0, %[lb], ((\s)&1)*8192+\j*1024
    s_add_u32 %[t0], %[sw], ((\s)*128+\j*16)*2048
    .if \j & 1
      buffer_load_dwordx4 v48, %[rW], %[t0] offen lds
    .else
      buffer_load_dwordx4 %[wv0], %[rW], %[t0] offen lds
    .endif
  .endr
.endm
.macro WTR s
  .irp t, 0,1,2,3
    ds_read_b64_tr_b8 v[128+32*\s+8*\t:129+32*\s+8*\t], v[49+\t] offset:(\s&1)*8192
    ds_read_b64_tr_b8 v[130+32*\s+8*\t:131+32*\s+8*\t], v[49+\t] offset:(\s&1)*8192+512
    ds_read_b64_tr_b8 v[132+32*\s+8*\t:133+32*\s+8*\t], v[49+\t] offset:(\s&1)*8192+4096
    ds_read_b64_tr_b8 v[134+32*\s+8*\t:135+32*\s+8*\t], v[49+\t] offset:(\s&1)*8192+4608
  .endr
.endm
.macro LD p, i
  .if \i < 4
    buffer_load_dwordx4 v[64+32*\p+4*\i:67+32*\p+4*\i], %[vstg], s[72:75], %[nso] offen offset:1024*\i sc0 sc1
  .else
    buffer_load_dwordx4 v[64+32*\p+4*\i:67+32*\p+4*\i], %[vstg2], s[72:75], %[nso] offen offset:1024*(\i-4) sc0 sc1
  .endif
.endm
.macro LDONE p
  s_add_u32 %[ops], %[ops], 8
  s_mov_b32 s[68+\p], %[ops]
  s_add_u32 %[iss], %[iss], 1
  s_add_u32 %[nso], %[nso], %[nst]
  s_cmp_eq_u32 %[iss], %[nl]
  s_cselect_b32 %[nso], %[nso1], %[nso]
  s_cselect_b32 %[nst], 65536, %[nst]
.endm
.macro BURST p
  .irp i, 0,1,2,3,4,5,6,7
    LD \p, \i
  .endr
  LDONE \p
.endm
.macro POLL own=0
  .if \own
    buffer_load_dword v31, %[vown], %[rF], 0 offen sc0 sc1
  .else
    s_sub_u32 %[pa], %[av], %[nl]
    s_lshl_b32 %[t1], %[pa], 7
    buffer_load_dword v31, %[vflag], %[rF], %[t1] offen sc0 sc1
  .endif
.endm
.macro TOAVAIL
  s_add_u32 %[t0], %[pa], %[base]
  v_add_u32 v30, %[t0], %[vexp]
  v_cmp_ge_u32 vcc, v31, v30
  s_ff0_i32_b32 %[t0], vcc_lo
  s_lshr_b32 %[t0], %[t0], 2
  s_add_u32 %[t0], %[t0], %[pa]
  s_min_u32 %[t0], %[t0], 32
  s_add_u32 %[t0], %[t0], %[nl]
  s_max_i32 %[av], %[av], %[t0]
.endm
.macro TOOWN
  v_cmp_le_u32 vcc, %[base], v31
  s_cmp_eq_u32 vcc_lo, %[omask]
  s_cselect_b32 %[av], %[nl], 0
.endm
.macro SPIN own=0
  s_memrealtime s[64:65]
  s_mov_b32 %[lim], 0
.Lspin\@:
  s_cmp_gt_i32 %[av], %[iss]
  s_cbranch_scc1 .Lgot\@
  POLL \own
  s_waitcnt vmcnt(0)
  .if \own
    TOOWN
  .else
    TOAVAIL
  .endif
  s_add_u32 %[lim], %[lim], 1
  s_and_b32 %[t0], %[lim], 31
  s_cbranch_scc1 .Lspin\@
  v_mov_b32 v30, 0
  global_load_dword v29, v30, %[tmo] sc1
  s_memrealtime s[66:67]
  s_waitcnt vmcnt(0) lgkmcnt(0)
  v_readfirstlane_b32 %[t0], v29
  s_cmp_eq_u32 %[t0], %[call]
  s_cbranch_scc1 .Ldead\@
  s_sub_u32 %[t0], s66, s64
  s_cmp_lt_u32 %[t0], )" STR(DEADLINE_TICKS_LIT) R"(
  s_cbranch_scc1 .Lspin\@
.Ldead\@:
  v_mov_b32 v29, %[call]
  global_store_dword v30, v29, %[tmo] sc1
  v_mov_b32 %[scale], 0x7fc00000
  s_add_u32 %[av], %[nl], 32
.Lgot\@:
.endm
.macro MM p, s, t
  .if \s == 0
    v_mfma_f32_16x16x128_f8f6f4 v[32+4*\t:35+4*\t], v[128+8*\t:135+8*\t], v[64+32*\p:71+32*\p], 0
  .elseif \s == 1
    v_mfma_f32_16x16x128_f8f6f4 v[48+4*\t:51+4*\t], v[160+8*\t:167+8*\t], v[72+32*\p:79+32*\p], v[32+4*\t:35+4*\t]
  .else
    v_mfma_f32_16x16x128_f8f6f4 v[48+4*\t:51+4*\t], v[128+32*\s+8*\t:135+32*\s+8*\t], v[64+32*\p+8*\s:71+32*\p+8*\s], v[48+4*\t:51+4*\t]
  .endif
.endm
.macro MG p, s
  .irp t, 0,1,2,3
    MM \p, \s, \t
  .endr
.endm
.macro MPROG p, base
  s_waitcnt vmcnt(\base+6)
  MG \p, 0
  s_waitcnt vmcnt(\base+4)
  MG \p, 1
  s_waitcnt vmcnt(\base+2)
  MG \p, 2
  s_waitcnt vmcnt(\base)
  POLL
  MG \p, 3
.endm
.macro TWR p, t
  .if \t == 0
    ds_write_b128 %[wr0], v[48:51] offset:4096*(1-\p)
  .elseif \t == 1
    ds_write_b128 %[wr1], v[52:55] offset:4096*(1-\p)
  .elseif \t == 2
    ds_write_b128 %[wr2], v[56:59] offset:4096*(1-\p)
  .else
    ds_write_b128 %[wr3], v[60:63] offset:4096*(1-\p)
  .endif
.endm
.macro TPUB p
  v_mov_b32 v30, %[n]
  ds_write_b32 %[vpw], v30
.endm
.macro WFLIP
  v_xor_b32 %[wr0], 8192, %[wr0]
  v_xor_b32 %[wr1], 8192, %[wr1]
  v_xor_b32 %[wr2], 8192, %[wr2]
  v_xor_b32 %[wr3], 8192, %[wr3]
.endm
.macro TADDR e
  s_and_b32 %[t1], \e, 3
  s_lshl_b32 %[t1], %[t1], 12
  v_add_u32 v28, %[t1], %[rd]
  s_mul_i32 %[oso], \e, 7*16*8192
  s_add_u32 %[oso], %[oso], %[oso0]
  s_lshl_b32 %[t1], \e, 17
  s_add_u32 %[t1], %[t1], %[oso1]
  s_cmp_lt_i32 \e, %[nl]
  s_cselect_b32 %[oso], %[oso], %[t1]
  s_cmp_lt_i32 \e, 0
  s_cselect_b32 %[oso], 0x40000000, %[oso]
.endm
.macro TREAD
  ds_read_b128 v[32:35], v28
  ds_read_b128 v[36:39], v28 offset:16384
  ds_read_b128 v[40:43], v28 offset:32768
  ds_read_b128 v[44:47], v28 offset:49152
.endm
.macro TSUM1
  v_add_f32 v32, v32, v36
  v_add_f32 v33, v33, v37
  v_add_f32 v34, v34, v38
  v_add_f32 v35, v35, v39
  v_add_f32 v40, v40, v44
  v_add_f32 v41, v41, v45
  v_add_f32 v42, v42, v46
  v_add_f32 v43, v43, v47
.endm
.macro TSUM2
  v_add_f32 v32, v32, v40
  v_add_f32 v34, v34, v42
  v_add_f32 v33, v33, v41
  v_add_f32 v35, v35, v43
  s_nop 0
  v_permlane32_swap_b32 v32, v34
  s_nop 0
  v_permlane32_swap_b32 v33, v35
  v_add_f32 v32, v32, v34
  v_add_f32 v33, v33, v35
.endm
.macro TEXP
  v_mul_f32 v32, %[scale], v32
  v_mul_f32 v33, %[scale], v33
  v_med3_f32 v34, v32, -s70, s70
  v_med3_f32 v35, v33, -s70, s70
  v_mul_f32 v34, 0xbfb8aa3b, v34
  v_mul_f32 v35, 0xbfb8aa3b, v35
  v_exp_f32 v34, v34
  v_exp_f32 v35, v35
  s_nop 0
.endm
.macro TDIV
  v_add_f32 v34, 1.0, v34
  v_add_f32 v35, 1.0, v35
  v_rcp_f32 v34, v34
  v_rcp_f32 v35, v35
  s_nop 0
  v_mul_f32 v36, v32, v34
  v_mul_f32 v37, v33, v35
  v_mul_f32 v38, v32, v34
  v_mul_f32 v39, v33, v35
  s_nop 0
.endm
.macro TSTORE
  buffer_store_dwordx4 v[36:39], %[vout], %[rO], %[oso] offen)" OUT_BITS R"(
  s_add_u32 %[ops], %[ops], 1
.endm
.macro HOT p
.Ltop\p:
  s_add_u32 %[t0], %[n], 2
  s_cmp_lg_u32 %[iss], %[t0]
  s_cbranch_scc1 .Lcold\p
  s_cmp_ge_i32 %[t0], %[av]
  s_cbranch_scc1 .Lref\p
.Lgo\p:
  s_sub_u32 %[t0], %[ops], s[68+\p]
  s_cmp_lt_u32 %[t0], 9
  s_cbranch_scc1 .Lpre\p
.Lhot\p:
  .if STAMPN
  ASTAMP 2, %[n]
  .endif
  s_sub_u32 %[tp], %[n], 2
  s_waitcnt vmcnt(15)
  MM \p, 0, 0
  TWR \p, 0
  TADDR %[tp]
  MM \p, 0, 1
  TWR \p, 1
  MM \p, 0, 2
  TWR \p, 2
  MM \p, 0, 3
  TWR \p, 3
  TPUB \p
  LD \p, 0
  LD \p, 1
  s_waitcnt vmcnt(15)
  MM \p, 1, 0
  ds_read_b32 v29, %[vpr]
  .if \p == 0
    WFLIP
  .endif
  MM \p, 1, 1
  MM \p, 1, 2
  MM \p, 1, 3
  LD \p, 2
  LD \p, 3
  s_waitcnt vmcnt(15) lgkmcnt(0)
  MM \p, 2, 0
  v_cmp_lt_i32 vcc, %[tp], v29
  s_cmp_eq_u32 vcc_lo, -1
  s_cbranch_scc0 .Lwait\p
.Lwaited\p:
  MM \p, 2, 1
  TREAD
  MM \p, 2, 2
  MM \p, 2, 3
  LD \p, 4
  LD \p, 5
  s_waitcnt vmcnt(15) lgkmcnt(0)
  MM \p, 3, 0
  s_add_u32 %[t0], %[iss], 2
  s_cmp_ge_i32 %[t0], %[av]
  s_cbranch_scc1 .Lpoll\p
.Lpolled\p:
  TSUM1
  MM \p, 3, 1
  TSUM2
  MM \p, 3, 2
  TEXP
  MM \p, 3, 3
  LD \p, 6
  LD \p, 7
  TDIV
  LDONE \p
  v_permlane32_swap_b32 v36, v38
  v_permlane32_swap_b32 v37, v39
  TSTORE
  s_add_u32 %[tp], %[tp], 1
  s_max_i32 %[tq], %[tq], %[tp]
  s_add_u32 %[n], %[n], 1
.endm
.macro COLD p
.Lpoll\p:
  POLL
  s_branch .Lpolled\p
.Lwait\p:
  ds_read_b32 v29, %[vpr]
  s_waitcnt lgkmcnt(0)
  v_cmp_lt_i32 vcc, %[tp], v29
  s_cmp_eq_u32 vcc_lo, -1
  s_cbranch_scc0 .Lwait\p
  s_branch .Lwaited\p
.Lpre\p:
  s_cmp_lt_u32 %[t0], 8
  s_cbranch_scc1 .Lpre0\p
  s_waitcnt vmcnt(8)
  s_branch .Lhot\p
.Lpre0\p:
  s_waitcnt vmcnt(0)
  s_branch .Lhot\p
.Lref\p:
  TOAVAIL
  s_cmp_lt_i32 %[iss], %[av]
  s_cbranch_scc1 .Lgo\p
.Lcold\p:
  TOAVAIL
  s_cmp_ge_i32 %[iss], %[av]
  s_cbranch_scc1 .Lc2\p
  s_cmp_lg_u32 %[iss], %[n]
  s_cbranch_scc1 .Lc1\p
  BURST \p
  s_cmp_ge_i32 %[iss], %[av]
  s_cbranch_scc1 .Lc2\p
.Lc1\p:
  s_add_u32 %[t0], %[n], 1
  s_cmp_lg_u32 %[iss], %[t0]
  s_cbranch_scc1 .Lc2\p
  BURST (1-\p)
  s_branch .Ltop\p
.Lc2\p:
  TWR \p, 0
  TWR \p, 1
  TWR \p, 2
  TWR \p, 3
  TPUB \p
  .if \p == 0
    WFLIP
  .endif
.Lc3\p:
  s_cmp_ge_i32 %[tq], %[n]
  s_cbranch_scc1 .Lc5\p
  TADDR %[tq]
.Lc4\p:
  ds_read_b32 v29, %[vpr]
  s_waitcnt lgkmcnt(0)
  v_cmp_lt_i32 vcc, %[tq], v29
  s_cmp_eq_u32 vcc_lo, -1
  s_cbranch_scc0 .Lc4\p
  TREAD
  s_waitcnt lgkmcnt(0)
  TSUM1
  TSUM2
  TEXP
  TDIV
  v_permlane32_swap_b32 v36, v38
  v_permlane32_swap_b32 v37, v39
  TSTORE
  s_add_u32 %[tq], %[tq], 1
  s_branch .Lc3\p
.Lc5\p:
  s_cmp_gt_u32 %[n], %[last]
  s_cbranch_scc1 .Ldone
  s_cmp_lg_u32 %[iss], %[n]
  s_cbranch_scc1 .Lc6\p
  SPIN
  BURST \p
.Lc6\p:
  .if STAMPN
  ASTAMP 2, %[n]
  .endif
  s_sub_u32 %[t0], %[ops], s[68+\p]
  s_add_u32 %[n], %[n], 1
  s_cmp_ge_u32 %[t0], 8
  s_cbranch_scc1 .Lc8\p
  s_cmp_ge_u32 %[t0], 2
  s_cbranch_scc1 .Lc7\p
  s_cmp_eq_u32 %[t0], 1
  s_cbranch_scc1 .Lc9\p
  MPROG \p, 0
  s_branch .Lnext\p
.Lc9\p:
  MPROG \p, 1
  s_branch .Lnext\p
.Lc7\p:
  MPROG \p, 2
  s_branch .Lnext\p
.Lc8\p:
  MPROG \p, 8
  s_branch .Lnext\p
.endm

  s_load_dword %[call], %[cnt], 0
  s_load_dword %[t1], %[psx], 0
  s_load_dword %[t2], %[psw], 0
  buffer_load_dwordx4 v[64:67], %[vcs], %[rX], 0 offen
  buffer_load_dwordx4 v[68:71], %[vcs], %[rX], 0 offen offset:64
  v_xor_b32 v48, 32, %[wv0]
  v_mov_b32 v30, 0
  ds_write_b32 %[vpw], v30
  WDMA 0
  v_mov_b32 v49, %[tr0]
  v_xor_b32 v50, 16, %[tr0]
  v_xor_b32 v51, 32, %[tr0]
  v_xor_b32 v52, 48, %[tr0]
  v_add_u32 %[wr0], 8192, %[wr0]
  v_xor_b32 %[wr1], 64, %[wr0]
  v_xor_b32 %[wr2], 128, %[wr0]
  v_xor_b32 %[wr3], 192, %[wr0]
  v_add_u32 %[vstg2], 4096, %[vstg]
  v_mov_b32 v31, 0
  s_mov_b32 s70, 0x42700000
  s_mov_b32 %[n], 0
  s_mov_b32 %[iss], 0
  s_mov_b32 %[ops], 0
  s_mov_b32 %[nso], %[nso0]
  s_mov_b32 %[nst], 7*65536
  s_mov_b32 %[tq], 0
  s_mov_b32 %[pa], 0
  s_waitcnt lgkmcnt(0)
  s_barrier
  s_add_u32 %[call], %[call], 1
  s_lshl_b32 %[base], %[call], 6
  v_mov_b32 %[scale], %[t1]
  v_mul_f32 %[scale], %[t2], %[scale]
  s_and_b32 %[t0], %[call], 1
  s_lshl_b32 %[t0], %[t0], 24
  s_mov_b64 s[72:73], %[aS]
  s_add_u32 s72, s72, %[t0]
  s_addc_u32 s73, s73, 0
  s_mov_b32 s74, 0x1000000
  s_mov_b32 s75, 0x20000
  v_mov_b32 v72, %[base]
  s_waitcnt vmcnt(8)
  buffer_store_dwordx4 v[64:67], %[vcd], s[72:75], 0 offen sc0 sc1
  buffer_store_dwordx4 v[68:71], %[vcd], s[72:75], 0 offen offset:1024 sc0 sc1
  WDMA 1
  s_waitcnt vmcnt(10)
  WTR 0
  ASTAMP 41, 0
  s_waitcnt vmcnt(8) lgkmcnt(0)
  buffer_store_dword v72, %[vcf], %[rF], 0 offen sc0 sc1
  ASTAMP 42, 0
  WDMA 2
  s_waitcnt vmcnt(9)
  WTR 1
  s_waitcnt lgkmcnt(0)
  WDMA 3
  POLL 1
  s_waitcnt vmcnt(9)
  WTR 2
  s_waitcnt vmcnt(0)
  WTR 3
  ASTAMP 43, 0
  TOOWN
)" FAKE_AV R"(
  SPIN 1
  ASTAMP 44, 0
  v_mov_b32 v31, 0
  BURST 0
  BURST 1
  s_waitcnt lgkmcnt(0)
  ASTAMP 1, 0
  CSTAMP 48
.Lnext1:
  HOT 0
.Lnext0:
  HOT 1
  s_branch .Lnext1
  COLD 0
  COLD 1
.Ldone:
  CSTAMP 49
)"

__global__ __launch_bounds__(THREADS, 2) void ag_gemm_silu(Params p) {
  extern __shared__ __attribute__((aligned(16))) unsigned char smem[];
  const int b = blockIdx.x, lane = threadIdx.x & 63;
  const int wv = (int)rfl(threadIdx.x >> 6);
  const unsigned rank = (unsigned)p.rank & (NR - 1);
  unsigned char* me = p.ws[rank];
  unsigned* cnt = (unsigned*)(me + CNT_OFF + (size_t)b * 128);
  unsigned* tmo = (unsigned*)(me + TMO_OFF);
  const unsigned long long t_start = __builtin_amdgcn_s_memrealtime();
  STAMP(0);

  if (b < NSEND) {
    const unsigned call = *(const CAS unsigned*)cnt + 1;
    const __amdgpu_buffer_rsrc_t rs = mk_rsrc(p.x, SHARD);
    if (wv == 7) return;
    const int i = (b & 7) * (NSEND / 8) + (b >> 3);
    const int u0 = (i * 7 + wv) * 2;
    const unsigned li = (unsigned)u0 >> 6, un = (unsigned)u0 & 63;
    const unsigned peer = (rank + 1 + li) & (NR - 1);
    const unsigned svo0 = (lane & 15) * KD + (lane >> 4) * 16 + un * 64, svo1 = svo0 + 64;
    constexpr int D = 4;
    u32x4 buf[D][2];
    auto lda = [&](u32x4 (&B)[2], int blk) {
      B[0] = __builtin_amdgcn_raw_buffer_load_b128(rs, svo0, blk * BLKB, 0);
      B[1] = __builtin_amdgcn_raw_buffer_load_b128(rs, svo1, blk * BLKB, 0);
    };
#pragma unroll
    for (int j = 0; j < D; j++) lda(buf[j], j);
    unsigned char* pw = p.ws[peer];
    const __amdgpu_buffer_rsrc_t rd = mk_rsrc(pw + (call & 1) * STAGE_B + (size_t)rank * SHARD, SHARD);
    const __amdgpu_buffer_rsrc_t rf = mk_rsrc(pw + FLAG_OFF + rank * FLAG_STR, FLAG_STR);
    const unsigned dvo = lane * 16 + un * 1024;
    const int first = max(i * 14, (int)li * 64), last = min((i + 1) * 14, (int)(li + 1) * 64);
    const bool pub = (u0 == first) && lane < (last - first) / 2;
    const unsigned fo = (un / 2 + lane) * 4;
    __builtin_amdgcn_s_waitcnt(0);
    STAMP(1);
#pragma unroll 1
    for (int c = 0; c < NBLK / D; c++) {
#pragma unroll
      for (int j = 0; j < D; j++) {
        const int blk = c * D + j;
        if (PACE) { while ((long long)(__builtin_amdgcn_s_memrealtime() - t_start) < (long long)(blk * PACE) / 10) __builtin_amdgcn_s_sleep(1); }
        __builtin_amdgcn_raw_buffer_store_b128(buf[j][0], rd, dvo, blk * BLKB, 17);
        __builtin_amdgcn_raw_buffer_store_b128(buf[j][1], rd, dvo + 1024, blk * BLKB, 17);
        asm volatile("" ::: "memory");
        lda(buf[j], blk + D);
        VMCNT(6);
        __builtin_amdgcn_s_barrier();
        if (blk > 0 && pub) __builtin_amdgcn_raw_buffer_store_b32(call * 64 + blk, rf, fo, (blk - 1) * 128, 17);
      }
    }
    VMCNT(0);
    __builtin_amdgcn_s_barrier();
    if (pub) __builtin_amdgcn_raw_buffer_store_b32(call * 64 + NBLK, rf, fo, (NBLK - 1) * 128, 17);
    if (threadIdx.x == 0) __hip_atomic_store(cnt, call, __ATOMIC_RELAXED, __HIP_MEMORY_SCOPE_AGENT);
    STAMP(2);
    return;
  }

  const int xcd = b & 7, slot = (b >> 3) - NSEND / 8;
  int cg, rg;
  cg = xcd * 4 + (slot & 3); rg = slot >> 2;
  const unsigned lds0 = (unsigned)(unsigned long long)(LAS unsigned char*)smem;
  const unsigned src = (rank + 1 + rg) & (NR - 1);
  const int q = lane >> 4, m = lane & 15;
  const unsigned wvo0 = (lane >> 2) * NL + cg * 64 + (((lane & 3) ^ (q & 1)) * 16);
  const unsigned tr0 = lds0 + wv * 16384 + (16 * q + (m >> 1)) * 64 + ((((m >> 3) & 1) | ((q & 1) << 1)) * 16) + 8 * (lane & 1);
  const unsigned vstg = lane * 16 + wv * 8192;
  const int nl = (rg < 4) ? 5 : 4;
  const unsigned vflag = lane < 16 ? src * FLAG_STR + (lane >> 2) * 128 + (wv * 4 + (lane & 3)) * 4 : 0x80000000u;
  const unsigned vexp = (lane >> 2) + 1;
  const unsigned vown = lane < 4 * nl ? rank * FLAG_STR + (rg + 7 * (lane >> 2)) * 128 + (wv * 4 + (lane & 3)) * 4 : 0x80000000u;
  const unsigned item = (unsigned)(b - NSEND) * 8 + wv, off_rng = item < 1024 ? 0u : 0x80000000u;
  const unsigned vcs = ((item >> 5) * BLKB + m * KD + (item & 31) * 128 + q * 16) | off_rng;
  const unsigned vcd = (rank * SHARD + item * 2048 + lane * 16) | off_rng;
  const unsigned vcf = lane == 0 ? (rank * FLAG_STR + item * 4) | off_rng : 0x80000000u;
  unsigned wr0 = lds0 + wv * 16384 + m * 256 + ((q ^ m) * 16);
  const int m_rd = 2 * wv + (q & 1);
  const unsigned rd = lds0 + (lane >> 5) * 65536 + m_rd * 256 + ((m ^ m_rd) * 16);
  const unsigned vout = lane < 32 ? m_rd * (NL * 4) + cg * 256 + m * 16 : 0x80000000u;
  const unsigned long long aS = (unsigned long long)me & 0xffffffffffffull;
  const u32x4 rW = mk_srd(p.w, KD * NL), rX = mk_srd(p.x, SHARD);
  const u32x4 rF = mk_srd(me + FLAG_OFF, (NR + 1) * FLAG_STR), rO = mk_srd(p.out, (unsigned)NR * MS * NL * 4u);
  unsigned call, base, n, iss, av, lim, ops, nso, nst, oso, tp, tq, pa, t0, t1, t2, t3;
  unsigned vstg2, wr1, wr2, wr3, scale;
  asm volatile(COMPUTE_ASM
      : [call] "=&s"(call), [base] "=&s"(base), [n] "=&s"(n), [iss] "=&s"(iss), [av] "=&s"(av), [lim] "=&s"(lim), [ops] "=&s"(ops),
        [nso] "=&s"(nso), [nst] "=&s"(nst), [oso] "=&s"(oso), [pa] "=&s"(pa), [tp] "=&s"(tp), [tq] "=&s"(tq), [t0] "=&s"(t0), [t1] "=&s"(t1), [t2] "=&s"(t2), [t3] "=&s"(t3),
        [wr0] "+v"(wr0), [vstg2] "=&v"(vstg2), [wr1] "=&v"(wr1), [wr2] "=&v"(wr2), [wr3] "=&v"(wr3), [scale] "=&v"(scale)
      : [rW] "s"(rW), [rX] "s"(rX), [rF] "s"(rF), [rO] "s"(rO), [lb] "s"(rfl(lds0 + wv * 16384)), [sw] "s"(rfl(wv * 512 * NL)),
        [nl] "s"(nl), [omask] "s"((1u << (4 * nl)) - 1), [last] "s"(nl + NBLK - 1),
        [nso0] "s"(rank * SHARD + rg * BLKB), [nso1] "s"(src * SHARD), [oso0] "s"((rank * MS + rg * 16) * (NL * 4)), [oso1] "s"((src * MS - nl * 16) * (NL * 4)),
        [aS] "s"(aS), [cnt] "s"(cnt), [tmo] "s"(tmo), [psx] "s"(p.sx), [psw] "s"(p.sw),
        [wv0] "v"(wvo0), [tr0] "v"(tr0), [vstg] "v"(vstg), [vcs] "v"(vcs), [vcd] "v"(vcd), [vcf] "v"(vcf), [vflag] "v"(vflag), [vown] "v"(vown), [vexp] "v"(vexp), [rd] "v"(rd), [vpw] "v"(lds0 + PROG_OFF + wv * 4), [vpr] "v"(lds0 + PROG_OFF + (lane & 7) * 4), [vout] "v"(vout)
      : "memory", "scc", "vcc", "s64", "s65", "s66", "s67", "s68", "s69", "s70", "s71", "s72", "s73", "s74", "s75", CLOB_V);
  if (threadIdx.x == 0) __hip_atomic_store(cnt, call, __ATOMIC_RELAXED, __HIP_MEMORY_SCOPE_AGENT);
}

extern "C" void kernel_launch(int rank, int nranks, ncclComm_t comm, hipStream_t stream,
                              void* const* d_in, const int* in_sizes, int n_in,
                              void* d_out, int out_size,
                              const int* in_dtypes, int out_dtype,
                              void* const* d_out_all,
                              void* const* d_ws_all, size_t ws_bytes) {
  if (nranks != NR || rank < 0 || rank >= NR || n_in != 4 || ws_bytes < WS_NEED ||
      in_sizes[0] != (int)SHARD || in_sizes[1] != KD * NL || out_size != NR * MS * NL) {
    fprintf(stderr, "ag_gemm_silu: unexpected problem shape\n");
    return;
  }
  HIP_CHECK(hipFuncSetAttribute((const void*)ag_gemm_silu, hipFuncAttributeMaxDynamicSharedMemorySize, LDS_BYTES));
  Params p;
  for (int r = 0; r < NR; r++) p.ws[r] = (unsigned char*)d_ws_all[r];
  p.x = (const unsigned char*)d_in[0];
  p.w = (const unsigned char*)d_in[1];
  p.sx = (const float*)d_in[2];
  p.sw = (const float*)d_in[3];
  p.out = (float*)d_out;
  p.rank = rank;
  p.pad = 0;
  ag_gemm_silu<<<GRID, THREADS, LDS_BYTES, stream>>>(p);
  HIP_CHECK(hipGetLastError());
}
```

```cpp
#include <hip/hip_runtime.h>
#include <rccl/rccl.h>
#include <cstdio>

#define HIP_CHECK(expr) do { hipError_t _e = (expr); if (_e != hipSuccess) { \
    fprintf(stderr, "HIP error %s at %s:%d\n", hipGetErrorString(_e), __FILE__, __LINE__); \
    return; } } while (0)

typedef int      v8i   __attribute__((ext_vector_type(8)));
typedef int      v2i   __attribute__((ext_vector_type(2)));
typedef unsigned u32x4 __attribute__((ext_vector_type(4)));
typedef float    v4f   __attribute__((ext_vector_type(4)));
#define LAS __attribute__((address_space(3)))
#define CAS __attribute__((address_space(4)))
#define DI __device__ __forceinline__

constexpr int      NR        = 8;
constexpr int      KD        = 4096, NL = 2048, MS = 512;
constexpr unsigned SHARD     = MS * KD;
constexpr unsigned BLKB      = 16 * KD;
constexpr int      NBLK      = MS / 16;
constexpr size_t   STAGE_B   = (size_t)NR * SHARD;
constexpr size_t   FLAG_OFF  = 2 * STAGE_B;
constexpr unsigned FLAG_STR  = NBLK * 128;
constexpr size_t   CNT_OFF   = FLAG_OFF + (NR + 1) * FLAG_STR;
constexpr size_t   TMO_OFF   = CNT_OFF + 256 * 128;
constexpr size_t   DIAG_OFF  = TMO_OFF + 4096;
constexpr size_t   WS_NEED   = DIAG_OFF + 256 * 8 * 64 * 8;
constexpr int      NSEND     = 32, GRID = 256, THREADS = 512;
constexpr unsigned PROG_OFF  = 131072;
constexpr unsigned LDS_BYTES = PROG_OFF + 64;
#define DEADLINE_TICKS_LIT 100000000
constexpr int      SRD3      = 0x00020000;

#define PACE 0
#define OUT_AUX 17
#define MAPMODE 0

struct Params {
  unsigned char* me;
  const unsigned char* x;
  const unsigned char* w;
  const float* sx;
  const float* sw;
  float* out;
  int rank;
  int pad[3];
  unsigned char* peer[NR];
};
typedef unsigned u32x16 __attribute__((ext_vector_type(16)));
DI unsigned long long mk64(unsigned lo, unsigned hi) { return ((unsigned long long)hi << 32) | lo; }

DI unsigned rfl(unsigned v) { return __builtin_amdgcn_readfirstlane(v); }
DI __amdgpu_buffer_rsrc_t mk_rsrc(const void* p, unsigned bytes) {
  unsigned long long a = (unsigned long long)p;
  unsigned lo = rfl((unsigned)a), hi = rfl((unsigned)(a >> 32));
  return __builtin_amdgcn_make_buffer_rsrc((void*)(((unsigned long long)hi << 32) | lo), (short)0, (int)bytes, SRD3);
}

DI u32x4 mk_srd(const void* p, unsigned bytes) {
  const unsigned long long a = (unsigned long long)p;
  return u32x4{rfl((unsigned)a), rfl((unsigned)(a >> 32) & 0xffffu), bytes, (unsigned)SRD3};
}

#define STAMP(slot) do {} while (0)

#define VMCNT(n)  asm volatile("s_waitcnt vmcnt(" #n ")" ::: "memory")
#define LGKM(n)   asm volatile("s_waitcnt lgkmcnt(" #n ")" ::: "memory")
#define LGKM0()   LGKM(0)
#define BARRIER() do { __builtin_amdgcn_s_barrier(); asm volatile("" ::: "memory"); } while (0)
#define SCHED()   __builtin_amdgcn_sched_barrier(0)

#define CLOB10(p) "v" #p "0", "v" #p "1", "v" #p "2", "v" #p "3", "v" #p "4", "v" #p "5", "v" #p "6", "v" #p "7", "v" #p "8", "v" #p "9"
#define CLOB_V "v20", "v21", "v22", "v23", "v24", "v25", "v26", "v27", "v28", "v29", CLOB10(3), CLOB10(4), CLOB10(5), CLOB10(6), "v70", "v71", "v72", "v73", "v81", "v82", "v83", "v84", "v85", "v86", "v87", "v88", "v89", CLOB10(9), CLOB10(10), CLOB10(11), CLOB10(12),   \
    CLOB10(13), CLOB10(14), CLOB10(15), CLOB10(16), CLOB10(17), CLOB10(18), CLOB10(19), CLOB10(20), CLOB10(21), CLOB10(22), CLOB10(23),        \
    CLOB10(24), "v250", "v251", "v252", "v253", "v254", "v255"
#define STR_(x) #x
#define STR(x) STR_(x)
#define OUT_BITS " sc0 sc1"
#define ASTAMP ".macro ASTAMP off, idx\n.endm\n.macro CSTAMP off\n.endm\n"
#define FAKE_AV ""
#define NPF 1
#define NPROBE 2
#define PTH 5
#define STAMPN 1
#define COMPUTE_ASM ".set STAMPN, " STR(STAMPN) "\n.set PTH, " STR(PTH) "\n.set NPF, " STR(NPF) "\n" ASTAMP R"(
.macro WDMA s
  .irp j, 0,1,2,3,4,5,6,7
    s_add_u32 m0, %[lb], ((\s)&1)*8192+\j*1024
    s_add_u32 %[t0], %[sw], ((\s)*128+\j*16)*2048
    .if \j & 1
      buffer_load_dwordx4 v48, %[rW], %[t0] offen lds
    .else
      buffer_load_dwordx4 %[wv0], %[rW], %[t0] offen lds
    .endif
  .endr
.endm
.macro WPF s
  .if NPF
    s_mov_b32 %[t0], \s*128*2048
    buffer_load_dword v73, %[vpf], %[rW], %[t0] offen
  .endif
.endm
.macro WTR s
  .irp t, 0,1,2,3
    ds_read_b64_tr_b8 v[128+32*\s+8*\t:129+32*\s+8*\t], v[49+\t] offset:(\s&1)*8192
    ds_read_b64_tr_b8 v[130+32*\s+8*\t:131+32*\s+8*\t], v[49+\t] offset:(\s&1)*8192+512
    ds_read_b64_tr_b8 v[132+32*\s+8*\t:133+32*\s+8*\t], v[49+\t] offset:(\s&1)*8192+4096
    ds_read_b64_tr_b8 v[134+32*\s+8*\t:135+32*\s+8*\t], v[49+\t] offset:(\s&1)*8192+4608
  .endr
.endm
.macro LD p, i
  .if \i < 4
    buffer_load_dwordx4 v[64+32*\p+4*\i:67+32*\p+4*\i], %[vstg], s[72:75], %[nso] offen offset:1024*\i sc0 sc1
  .else
    buffer_load_dwordx4 v[64+32*\p+4*\i:67+32*\p+4*\i], %[vstg2], s[72:75], %[nso] offen offset:1024*(\i-4) sc0 sc1
  .endif
.endm
.macro LDONE p
  s_add_u32 %[ops], %[ops], 8
  s_mov_b32 s[68+\p], %[ops]
  s_add_u32 %[iss], %[iss], 1
  s_add_u32 %[nso], %[nso], %[nst]
  s_cmp_eq_u32 %[iss], %[nl]
  s_cselect_b32 %[nso], %[nso1], %[nso]
  s_cselect_b32 %[nst], 65536, %[nst]
.endm
.macro BURST p
  .irp i, 0,1,2,3,4,5,6,7
    LD \p, \i
  .endr
  LDONE \p
.endm
.macro POLL own=0
  .if \own
    buffer_load_dword v31, %[vown], %[rF], 0 offen sc0 sc1
  .else
    s_sub_u32 %[pa], %[av], %[nl]
    s_lshl_b32 %[t1], %[pa], 7
    buffer_load_dword v31, %[vflag], %[rF], %[t1] offen sc0 sc1
  .endif
.endm
.macro TOAVAIL
  s_add_u32 %[t0], %[pa], %[base]
  v_add_u32 v30, %[t0], %[vexp]
  v_cmp_ge_u32 vcc, v31, v30
  s_ff0_i32_b32 %[t0], vcc_lo
  s_lshr_b32 %[t0], %[t0], 2
  s_add_u32 %[t0], %[t0], %[pa]
  s_min_u32 %[t0], %[t0], 32
  s_add_u32 %[t0], %[t0], %[nl]
  s_max_i32 %[av], %[av], %[t0]
.endm
.macro TOOWN
  v_cmp_le_u32 vcc, %[base], v31
  s_cmp_eq_u32 vcc_lo, %[omask]
  s_cselect_b32 %[av], %[nl], 0
.endm
.macro SPIN own=0
  s_memrealtime s[64:65]
  s_mov_b32 %[lim], 0
.Lspin\@:
  s_cmp_gt_i32 %[av], %[iss]
  s_cbranch_scc1 .Lgot\@
  POLL \own
  s_waitcnt vmcnt(0)
  .if \own
    TOOWN
  .else
    TOAVAIL
  .endif
  s_add_u32 %[lim], %[lim], 1
  s_and_b32 %[t0], %[lim], 31
  s_cbranch_scc1 .Lspin\@
  v_mov_b32 v30, 0
  global_load_dword v29, v30, %[tmo] sc1
  s_memrealtime s[66:67]
  s_waitcnt vmcnt(0) lgkmcnt(0)
  v_readfirstlane_b32 %[t0], v29
  s_cmp_eq_u32 %[t0], %[call]
  s_cbranch_scc1 .Ldead\@
  s_sub_u32 %[t0], s66, s64
  s_cmp_lt_u32 %[t0], )" STR(DEADLINE_TICKS_LIT) R"(
  s_cbranch_scc1 .Lspin\@
.Ldead\@:
  v_mov_b32 v29, %[call]
  global_store_dword v30, v29, %[tmo] sc1
  v_mov_b32 %[scale], 0x7fc00000
  s_add_u32 %[av], %[nl], 32
.Lgot\@:
.endm
.macro MM p, s, t
  .if \s == 0
    v_mfma_f32_16x16x128_f8f6f4 v[48-16*\p+4*\t:51-16*\p+4*\t], v[128+8*\t:135+8*\t], v[64+32*\p:71+32*\p], 0
  .else
    v_mfma_f32_16x16x128_f8f6f4 v[48-16*\p+4*\t:51-16*\p+4*\t], v[128+32*\s+8*\t:135+32*\s+8*\t], v[64+32*\p+8*\s:71+32*\p+8*\s], v[48-16*\p+4*\t:51-16*\p+4*\t]
  .endif
.endm
.macro MG p, s
  .irp t, 0,1,2,3
    MM \p, \s, \t
  .endr
.endm
.macro MPROG p, base
  s_waitcnt vmcnt(\base+6)
  MG \p, 0
  s_waitcnt vmcnt(\base+4)
  MG \p, 1
  s_waitcnt vmcnt(\base+2)
  MG \p, 2
  s_waitcnt vmcnt(\base)
  POLL
  MG \p, 3
.endm
.macro TWR p, t
  .if \t == 0
    ds_write_b128 %[wr0], v[48-16*\p:51-16*\p] offset:4096*\p
  .elseif \t == 1
    ds_write_b128 %[wr1], v[52-16*\p:55-16*\p] offset:4096*\p
  .elseif \t == 2
    ds_write_b128 %[wr2], v[56-16*\p:59-16*\p] offset:4096*\p
  .else
    ds_write_b128 %[wr3], v[60-16*\p:63-16*\p] offset:4096*\p
  .endif
.endm
.macro WFLIP
  v_xor_b32 %[wr0], 8192, %[wr0]
  v_xor_b32 %[wr1], 8192, %[wr1]
  v_xor_b32 %[wr2], 8192, %[wr2]
  v_xor_b32 %[wr3], 8192, %[wr3]
.endm
.macro TADDR e
  s_and_b32 %[t1], \e, 3
  s_lshl_b32 %[t1], %[t1], 12
  v_add_u32 v28, %[t1], %[rd]
  s_lshl_b32 %[oso], \e, 17
  s_add_u32 %[oso], %[oso], %[oso1]
.endm
.macro TOWNROW e
  s_mul_i32 %[oso], \e, 7*16*8192
  s_add_u32 %[oso], %[oso], %[oso0]
  s_cmp_lt_i32 \e, 0
  s_cselect_b32 %[oso], 0x40000000, %[oso]
.endm
.macro RD r, a, v
  ds_read_b128 v[\r:\r+3], \a offset:16384*\v
.endm
.macro OP4 op, d, a, b
  .irp i, 0,1,2,3
    \op v[\d+\i], v[\a+\i], v[\b+\i]
  .endr
.endm
.macro K4 op, d, k, a
  .irp i, 0,1,2,3
    \op v[\d+\i], \k, v[\a+\i]
  .endr
.endm
.macro U4 op, d, a
  .irp i, 0,1,2,3
    \op v[\d+\i], v[\a+\i]
  .endr
.endm
.macro MED4
  .irp i, 0,1,2,3
    v_med3_f32 v[24+\i], v[20+\i], -s70, s70
  .endr
.endm
.macro TSTORE
  buffer_store_dwordx4 v[24:27], %[vout], %[rO], %[oso] offen)" OUT_BITS R"(
.endm
.macro TFIN
  K4 v_mul_f32, 20, %[scale], 20
  MED4
  K4 v_mul_f32, 24, 0xbfb8aa3b, 24
  U4 v_exp_f32, 24, 24
  K4 v_add_f32, 24, 1.0, 24
  U4 v_rcp_f32, 24, 24
  OP4 v_mul_f32, 24, 20, 24
  TSTORE
.endm
.macro T tl, x:vararg
  .if \tl
    \x
  .endif
.endm
.macro LD7 p
  LD \p, 7
  s_add_u32 %[nso], %[nso], %[nst]
  s_add_u32 %[t0], %[n], 2
  s_cmp_eq_u32 %[t0], %[nl]
  s_cselect_b32 %[nso], %[nso1], %[nso]
  s_cselect_b32 %[nst], 65536, %[nst]
.endm
.macro HOT p, q, g, l
.Lhot\p\g\l:
  .if STAMPN
  ASTAMP 2, %[n]
  .endif
  T \l==1, s_waitcnt vmcnt(13)
  MM \p, 0, 0
  T \q==\g, K4 v_mul_f32, 20, %[scale], 20
  T \p==\g, s_nop 0
  TWR \q, 0
  MM \p, 0, 1
  T \l==1, LD7 \q
  T \q==\g, MED4
  MM \p, 0, 2
  TWR \q, 1
  T \q==\g, K4 v_mul_f32, 24, 0xbfb8aa3b, 24
  MM \p, 0, 3
  T \l, LD \p, 0
  T \q==\g, U4 v_exp_f32, 24, 24
  T \l==1, s_waitcnt vmcnt(13)
  MM \p, 1, 0
  TWR \q, 2
  T \q==\g, K4 v_add_f32, 24, 1.0, 24
  MM \p, 1, 1
  T \l, LD \p, 1
  T \q==\g, U4 v_rcp_f32, 24, 24
  v_mov_b32 v30, %[n]
  MM \p, 1, 2
  TWR \q, 3
  ds_write_b32 %[vpw], v30
  ds_read_b32 v29, %[vpr]
  T \q==\g, OP4 v_mul_f32, 24, 20, 24
  MM \p, 1, 3
  T \l, LD \p, 2
  s_sub_u32 %[tp], %[n], 2
  T \p==\g, TADDR %[tp]
  T \p==\g, s_cmp_lt_i32 %[tp], %[nl]
  T \p==\g, s_cbranch_scc1 .Lown\p\g\l
.Lowned\p\g\l:
  T \l==1, s_waitcnt vmcnt(13)
  MM \p, 2, 0
  T \q==\g, TSTORE
  s_waitcnt lgkmcnt(0)
  v_cmp_lt_i32 vcc, %[tp], v29
  s_cmp_eq_u32 vcc_lo, -1
  s_cbranch_scc0 .Lwait\p\g\l
.Lwaited\p\g\l:
  T \p==\g, RD 20, v28, 0
  T \p==\g, RD 32+16*\p, v28, 1
  T \p==\g, RD 36+16*\p, v28, 2
  T \p==\g, RD 40+16*\p, v28, 3
  MM \p, 2, 1
  T \l, LD \p, 3
  T \p==\g, v_add_u32 v29, 0x10000, v28
  T \p==\g, RD 44+16*\p, v29, 0
  T \p==\g, RD 24, v29, 1
  MM \p, 2, 2
  .if \q
    WFLIP
  .endif
  .if \l
    s_add_u32 %[t0], %[n], PTH
    s_cmp_gt_i32 %[t0], %[av]
    s_cbranch_scc1 .Lpoll\p\g\l
.Lpolled\p\g\l:
  .else
    POLL
  .endif
  MM \p, 2, 3
  T \l, LD \p, 4
  T \p==\g, s_waitcnt lgkmcnt(2)
  T \p==\g, OP4 v_add_f32, 20, 20, 32+16*\p
  T \p==\g, RD 32+16*\p, v29, 2
  T \l==1, s_waitcnt vmcnt(13)
  MM \p, 3, 0
  T \p==\g, OP4 v_add_f32, 36+16*\p, 36+16*\p, 40+16*\p
  T \p==\g, RD 40+16*\p, v29, 3
  MM \p, 3, 1
  T \l, LD \p, 5
  T \p==\g, OP4 v_add_f32, 20, 20, 36+16*\p
  MM \p, 3, 2
  T \p==\g, s_waitcnt lgkmcnt(2)
  T \p==\g, OP4 v_add_f32, 24, 44+16*\p, 24
  T \p==\g, OP4 v_add_f32, 20, 20, 24
  s_add_u32 %[n], %[n], 1
  s_add_u32 %[t0], %[n], 2
  MM \p, 3, 3
  T \l, LD \p, 6
  T \p==\g, s_waitcnt lgkmcnt(0)
  T \p==\g, OP4 v_add_f32, 24, 32+16*\p, 40+16*\p
  T \p==\g, OP4 v_add_f32, 20, 20, 24
  .if \l
    s_cmp_ge_i32 %[t0], %[av]
    s_cbranch_scc1 .Lexit\p\g
    T \l==2, s_branch .Lhot\q\g\()1
  .else
    T \q==\g, s_add_u32 %[ops], %[ops], 1
    s_sub_u32 %[t0], %[n], 2-((\p==\g)&1)
    s_max_i32 %[tq], %[tq], %[t0]
    s_branch .Ltop\q\g
  .endif
.endm
.macro STUBS p, g, l
  .if \l
.Lpoll\p\g\l:
    POLL
    s_branch .Lpolled\p\g\l
  .endif
.Lown\p\g\l:
  TOWNROW %[tp]
  s_branch .Lowned\p\g\l
.Lwait\p\g\l:
  ds_read_b32 v29, %[vpr]
  s_waitcnt lgkmcnt(0)
  v_cmp_lt_i32 vcc, %[tp], v29
  s_cmp_eq_u32 vcc_lo, -1
  s_cbranch_scc0 .Lwait\p\g\l
  s_branch .Lwaited\p\g\l
.endm
.macro COLD p, q, g
  STUBS \p, \g, 0
  STUBS \p, \g, 1
  STUBS \p, \g, 2
.Lexit\p\g:
  TOAVAIL
  s_add_u32 %[iss], %[n], 2
  s_cmp_lt_i32 %[iss], %[av]
  s_cbranch_scc1 .Lhot\q\g\()1
  LD7 \p
  s_mov_b32 s[68+\p], %[ops]
  s_sub_u32 s[69-\p], %[ops], 8
  s_sub_u32 %[t0], %[n], 2-((\p==\g)&1)
  s_max_i32 %[tq], %[tq], %[t0]
  s_waitcnt vmcnt(8)
  s_branch .Lhot\q\g\()0
.Ltop\p\g:
  TOAVAIL
  s_cmp_eq_u32 %[iss], %[n]
  s_cbranch_scc1 .Lcold\p\g
  s_add_u32 %[t0], %[n], 1
  s_cmp_lg_u32 %[iss], %[t0]
  s_cbranch_scc1 .Lt2\p\g
  s_cmp_ge_i32 %[iss], %[av]
  s_cbranch_scc1 .Lt1\p\g
  BURST \q
.Lt2\p\g:
  s_waitcnt vmcnt(8)
  s_cmp_lt_i32 %[iss], %[av]
  s_cbranch_scc1 .Lhot\p\g\()2
  s_branch .Lhot\p\g\()0
.Lt1\p\g:
  s_cmp_eq_u32 %[ops], s[68+\p]
  s_cbranch_scc1 .Lt0\p\g
  s_waitcnt vmcnt(1)
  s_branch .Lhot\p\g\()0
.Lt0\p\g:
  s_waitcnt vmcnt(0)
  s_branch .Lhot\p\g\()0
.Lcold\p\g:
  s_cmp_ge_i32 %[iss], %[av]
  s_cbranch_scc1 .Lc1\p\g
  BURST \p
.Lc1\p\g:
  v_mov_b32 v30, %[n]
  TWR \q, 0
  TWR \q, 1
  TWR \q, 2
  TWR \q, 3
  ds_write_b32 %[vpw], v30
  .if \q==\g
    TFIN
    s_add_u32 %[ops], %[ops], 1
  .endif
  s_cmp_gt_u32 %[n], %[last]
  s_cbranch_scc1 .Lc3\p\g
  s_cmp_lg_u32 %[iss], %[n]
  s_cbranch_scc1 .Lc2\p\g
  SPIN
  BURST \p
.Lc2\p\g:
  s_cmp_ge_i32 %[iss], %[av]
  s_cbranch_scc1 .Lc3\p\g
  BURST \q
.Lc3\p\g:
  s_cmp_ge_i32 %[tq], %[n]
  s_cbranch_scc1 .Lc5\p\g
  TADDR %[tq]
  s_cmp_ge_i32 %[tq], %[nl]
  s_cbranch_scc1 .Lc4\p\g
  TOWNROW %[tq]
.Lc4\p\g:
  ds_read_b32 v29, %[vpr]
  s_waitcnt lgkmcnt(0)
  v_cmp_lt_i32 vcc, %[tq], v29
  s_cmp_eq_u32 vcc_lo, -1
  s_cbranch_scc0 .Lc4\p\g
  RD 20, v28, 0
  RD 24, v28, 1
  RD 32+16*\q, v28, 2
  RD 36+16*\q, v28, 3
  v_add_u32 v29, 0x10000, v28
  RD 40+16*\q, v29, 0
  RD 44+16*\q, v29, 1
  s_waitcnt lgkmcnt(4)
  OP4 v_add_f32, 20, 20, 24
  RD 24, v29, 2
  s_waitcnt lgkmcnt(3)
  OP4 v_add_f32, 32+16*\q, 32+16*\q, 36+16*\q
  RD 36+16*\q, v29, 3
  OP4 v_add_f32, 20, 20, 32+16*\q
  s_waitcnt lgkmcnt(2)
  OP4 v_add_f32, 40+16*\q, 40+16*\q, 44+16*\q
  OP4 v_add_f32, 20, 20, 40+16*\q
  s_waitcnt lgkmcnt(0)
  OP4 v_add_f32, 24, 24, 36+16*\q
  OP4 v_add_f32, 20, 20, 24
  TFIN
  s_add_u32 %[ops], %[ops], 1
  s_add_u32 %[tq], %[tq], 2
  s_branch .Lc3\p\g
.Lc5\p\g:
  s_cmp_gt_u32 %[n], %[last]
  s_cbranch_scc1 .Ldone
  .if \q
    WFLIP
  .endif
  s_mov_b32 %[oso], 0x40000000
  .if STAMPN
  ASTAMP 2, %[n]
  .endif
  s_sub_u32 %[t0], %[ops], s[68+\p]
  s_add_u32 %[n], %[n], 1
  s_cmp_ge_u32 %[t0], 8
  s_cbranch_scc1 .Lc8\p\g
  s_cmp_ge_u32 %[t0], 2
  s_cbranch_scc1 .Lc7\p\g
  s_cmp_ge_u32 %[t0], 1
  s_cbranch_scc1 .Lc9\p\g
  MPROG \p, 0
  s_branch .Ltop\q\g
.Lc9\p\g:
  MPROG \p, 1
  s_branch .Ltop\q\g
.Lc7\p\g:
  MPROG \p, 2
  s_branch .Ltop\q\g
.Lc8\p\g:
  MPROG \p, 8
  s_branch .Ltop\q\g
.endm
.macro MAIN g
  HOT 0, 1, \g, 1
  HOT 1, 0, \g, 1
  s_branch .Lhot0\g\()1
  HOT 0, 1, \g, 2
  HOT 1, 0, \g, 2
  HOT 0, 1, \g, 0
  HOT 1, 0, \g, 0
  COLD 0, 1, \g
  COLD 1, 0, \g
.endm

  v_mov_b32 v30, 0
  ds_write_b32 %[vpw], v30
  s_waitcnt lgkmcnt(0)
  s_load_dword %[call], %[cnt], 0
  s_load_dword %[t1], %[psx], 0
  s_load_dword %[t2], %[psw], 0
  s_barrier
  v_xor_b32 v48, 32, %[wv0]
  s_cmp_eq_u32 %[cp], 0
  s_cbranch_scc1 .Lnocopy
  buffer_load_dwordx4 v[64:67], %[vcs], %[rX], 0 offen
  buffer_load_dwordx4 v[68:71], %[vcs], %[rX], 0 offen offset:64
.Lnocopy:
  WPF 0
  WPF 1
  s_cmp_eq_u32 %[cp], 0
  s_cbranch_scc0 .Lcopy1
  WDMA 0
  WPF 2
  WDMA 1
  WPF 3
.Lcopy1:
  v_mov_b32 v49, %[tr0]
  v_xor_b32 v50, 16, %[tr0]
  v_xor_b32 v51, 32, %[tr0]
  v_xor_b32 v52, 48, %[tr0]
  v_add_u32 %[wr0], 8192, %[wr0]
  s_mov_b32 %[oso], 0x40000000
  v_xor_b32 %[wr1], 64, %[wr0]
  v_xor_b32 %[wr2], 128, %[wr0]
  v_xor_b32 %[wr3], 192, %[wr0]
  v_add_u32 %[vstg2], 4096, %[vstg]
  s_mov_b32 s70, 0x42700000
  s_mov_b32 %[n], 0
  s_mov_b32 %[iss], 0
  s_mov_b32 %[ops], 0
  s_mov_b32 %[nso], %[nso0]
  s_mov_b32 %[nst], 7*65536
  s_mov_b32 %[tq], %[grp]
  s_mov_b32 %[pa], 0
  s_waitcnt lgkmcnt(0)
  s_add_u32 %[call], %[call], 1
  s_lshl_b32 %[base], %[call], 6
  v_mov_b32 %[scale], %[t1]
  v_mul_f32 %[scale], %[t2], %[scale]
  s_and_b32 %[t0], %[call], 1
  s_lshl_b32 %[t0], %[t0], 24
  s_mov_b64 s[72:73], %[aS]
  s_add_u32 s72, s72, %[t0]
  s_addc_u32 s73, s73, 0
  s_mov_b32 s74, 0x1000000
  s_mov_b32 s75, 0x20000
  s_cmp_eq_u32 %[cp], 0
  s_cbranch_scc1 .Lcopied
  v_mov_b32 v72, %[base]
  WPF 2
  WPF 3
  s_waitcnt vmcnt(4*NPF)
  buffer_store_dwordx4 v[64:67], %[vcd], s[72:75], 0 offen sc0 sc1
  buffer_store_dwordx4 v[68:71], %[vcd], s[72:75], 0 offen offset:1024 sc0 sc1
  s_waitcnt vmcnt(0)
  buffer_store_dword v72, %[vcf], %[rF], 0 offen sc0 sc1
  WDMA 0
  WDMA 1
.Lcopied:
  ASTAMP 42, 0
  s_waitcnt vmcnt(8)
  WTR 0
  ASTAMP 41, 0
  s_waitcnt lgkmcnt(0)
  WDMA 2
  s_waitcnt vmcnt(8)
  WTR 1
  s_waitcnt lgkmcnt(0)
  WDMA 3
  POLL 1
  s_waitcnt vmcnt(9)
  WTR 2
  s_waitcnt vmcnt(0)
  WTR 3
  ASTAMP 43, 0
  TOOWN
)" FAKE_AV R"(
  SPIN 1
  ASTAMP 44, 0
  v_mov_b32 v31, 0
  BURST 0
  BURST 1
  s_waitcnt lgkmcnt(0)
  ASTAMP 1, 0
  CSTAMP 48
  s_cmp_eq_u32 %[tq], 0
  s_cbranch_scc1 .Ltop00
  s_branch .Ltop01
  MAIN 0
  MAIN 1
.Ldone:
  CSTAMP 49
)"

__global__ __launch_bounds__(THREADS, 2) void ag_gemm_silu(Params) {
  extern __shared__ __attribute__((aligned(16))) unsigned char smem[];
  const int b = blockIdx.x, lane = threadIdx.x & 63;
  const int wv = (int)rfl(threadIdx.x >> 6);
  const int si = (b & 7) * (NSEND / 8) + (b >> 3), u0 = (si * 7 + wv) * 2;
  const unsigned li = ((unsigned)u0 >> 6) & (NR - 1), un = (unsigned)u0 & 63;
  u32x16 ka;
  unsigned long long pw_;
  asm volatile("s_load_dwordx16 %0, %2, 0\n\ts_load_dwordx2 %1, %2, %3\n\ts_waitcnt lgkmcnt(0)"
               : "=&s"(ka), "=&s"(pw_) : "s"(__builtin_amdgcn_kernarg_segment_ptr()), "s"(64 + li * 8));
  Params p;
  p.me = (unsigned char*)mk64(ka[0], ka[1]); p.x = (const unsigned char*)mk64(ka[2], ka[3]); p.w = (const unsigned char*)mk64(ka[4], ka[5]);
  p.sx = (const float*)mk64(ka[6], ka[7]); p.sw = (const float*)mk64(ka[8], ka[9]); p.out = (float*)mk64(ka[10], ka[11]);
  const unsigned rank = ka[12] & (NR - 1);
  unsigned char* me = p.me;
  unsigned* cnt = (unsigned*)(me + CNT_OFF + (size_t)b * 128);
  unsigned* tmo = (unsigned*)(me + TMO_OFF);

  if (b < NSEND) {
    const unsigned call = *(const CAS unsigned*)cnt + 1;
    const __amdgpu_buffer_rsrc_t rs = mk_rsrc(p.x, SHARD);
    if (wv == 7) return;
    const int i = si;
    const unsigned svo0 = (lane & 15) * KD + (lane >> 4) * 16 + un * 64, svo1 = svo0 + 64;
    constexpr int D = 4;
    u32x4 buf[D][2];
    auto lda = [&](u32x4 (&B)[2], int blk) {
      B[0] = __builtin_amdgcn_raw_buffer_load_b128(rs, svo0, blk * BLKB, 0);
      B[1] = __builtin_amdgcn_raw_buffer_load_b128(rs, svo1, blk * BLKB, 0);
    };
#pragma unroll
    for (int j = 0; j < D; j++) lda(buf[j], j);
    unsigned char* pw = (unsigned char*)pw_;
    const __amdgpu_buffer_rsrc_t rd = mk_rsrc(pw + (call & 1) * STAGE_B + (size_t)rank * SHARD, SHARD);
    const __amdgpu_buffer_rsrc_t rf = mk_rsrc(pw + FLAG_OFF + rank * FLAG_STR, FLAG_STR);
    const unsigned dvo = lane * 16 + un * 1024;
    const int first = max(i * 14, (int)li * 64), last = min((i + 1) * 14, (int)(li + 1) * 64);
    const bool pub = (u0 == first) && lane < (last - first) / 2;
    const unsigned fo = (un / 2 + lane) * 4;
    STAMP(1);
#pragma unroll 1
    for (int c = 0; c < NBLK / D; c++) {
#pragma unroll
      for (int j = 0; j < D; j++) {
        const int blk = c * D + j;
        __builtin_amdgcn_raw_buffer_store_b128(buf[j][0], rd, dvo, blk * BLKB, 17);
        __builtin_amdgcn_raw_buffer_store_b128(buf[j][1], rd, dvo + 1024, blk * BLKB, 17);
        asm volatile("" ::: "memory");
        lda(buf[j], blk + D);
        VMCNT(6);
        __builtin_amdgcn_s_barrier();
        if (blk > 0 && pub) __builtin_amdgcn_raw_buffer_store_b32(call * 64 + blk, rf, fo, (blk - 1) * 128, 17);
      }
    }
    VMCNT(0);
    __builtin_amdgcn_s_barrier();
    if (pub) __builtin_amdgcn_raw_buffer_store_b32(call * 64 + NBLK, rf, fo, (NBLK - 1) * 128, 17);
    if (threadIdx.x == 0) __hip_atomic_store(cnt, call, __ATOMIC_RELAXED, __HIP_MEMORY_SCOPE_AGENT);
    STAMP(2);
    return;
  }

  const int xcd = b & 7, slot = (b >> 3) - NSEND / 8;
  int cg, rg;
  cg = xcd * 4 + (slot & 3); rg = slot >> 2;
  const unsigned lds0 = (unsigned)(unsigned long long)(LAS unsigned char*)smem;
  const unsigned src = (rank + 1 + rg) & (NR - 1);
  const int q = lane >> 4, m = lane & 15;
  unsigned wvo0 = (lane >> 2) * NL + cg * 64 + (((lane & 3) ^ (q & 1)) * 16);
  unsigned tr0 = lds0 + wv * 16384 + (16 * q + (m >> 1)) * 64 + ((((m >> 3) & 1) | ((q & 1) << 1)) * 16) + 8 * (lane & 1);
  const unsigned vstg = lane * 16 + wv * 8192;
  const int nl = (rg < 4) ? 5 : 4;
  const unsigned vflag = lane < 4 * NPROBE ? src * FLAG_STR + (lane >> 2) * 128 + (wv * 4 + (lane & 3)) * 4 : 0x80000000u;
  const unsigned vexp = (lane >> 2) + 1;
  unsigned vown = lane < 4 * nl ? rank * FLAG_STR + (rg + 7 * (lane >> 2)) * 128 + (wv * 4 + (lane & 3)) * 4 : 0x80000000u;
  const unsigned item = wv * (GRID - NSEND) + (unsigned)(b - NSEND);
  unsigned vcs = (item >> 5) * BLKB + m * KD + (item & 31) * 128 + q * 16;
  unsigned vcd = rank * SHARD + item * 2048 + lane * 16;
  unsigned vcf = lane == 0 ? rank * FLAG_STR + item * 4 : 0x80000000u;
  constexpr int LPR = MAPMODE == 0 ? 2 : MAPMODE == 3 ? 4 : 8, PL = (1024 * LPR + 223) / 224;
  const unsigned pfl = (slot * 8 + wv) * PL + lane, pfr = pfl / LPR;
  unsigned vpf = (lane < PL && pfr < 1024) ? ((pfr >> 7) * 512 + (pfr & 127)) * NL + ((cg >> 1) / LPR * LPR + pfl % LPR) * 128 : 0x80000000u;
  unsigned wr0 = lds0 + wv * 16384 + m * 256 + ((q ^ m) * 16);
  const int m_rd = 4 * (wv & 3) + q;
  const unsigned rd = lds0 + m_rd * 256 + ((m ^ m_rd) * 16);
  const unsigned vout = m_rd * (NL * 4) + cg * 256 + m * 16;
  const unsigned long long aS = (unsigned long long)me & 0xffffffffffffull;
  const u32x4 rW = mk_srd(p.w, KD * NL), rX = mk_srd(p.x, SHARD);
  const u32x4 rF = mk_srd(me + FLAG_OFF, (NR + 1) * FLAG_STR), rO = mk_srd(p.out, (unsigned)NR * MS * NL * 4u);
  unsigned call, base, n, iss, av, lim, ops, nso, nst, oso, tp, tq, pa, t0, t1, t2, t3;
  unsigned vstg2, wr1, wr2, wr3, scale;
  asm volatile(COMPUTE_ASM
      : [call] "=&s"(call), [base] "=&s"(base), [n] "=&s"(n), [iss] "=&s"(iss), [av] "=&s"(av), [lim] "=&s"(lim), [ops] "=&s"(ops),
        [nso] "=&s"(nso), [nst] "=&s"(nst), [oso] "=&s"(oso), [pa] "=&s"(pa), [tp] "=&s"(tp), [tq] "=&s"(tq), [t0] "=&s"(t0), [t1] "=&s"(t1), [t2] "=&s"(t2), [t3] "=&s"(t3),
        [wr0] "+v"(wr0), [vown] "+{v80}"(vown), [vcf] "+{v78}"(vcf), [vcd] "+{v77}"(vcd), [vcs] "+{v76}"(vcs), [tr0] "+{v75}"(tr0), [wv0] "+{v74}"(wvo0), [vpf] "+{v79}"(vpf), [vstg2] "=&v"(vstg2), [wr1] "=&v"(wr1), [wr2] "=&v"(wr2), [wr3] "=&v"(wr3), [scale] "=&v"(scale)
      : [rW] "s"(rW), [rX] "s"(rX), [rF] "s"(rF), [rO] "s"(rO), [lb] "s"(rfl(lds0 + wv * 16384)), [sw] "s"(rfl(wv * 512 * NL)),
        [nl] "s"(nl), [grp] "s"(rfl(wv >> 2)), [omask] "s"((1u << (4 * nl)) - 1), [last] "s"(nl + NBLK - 1),
        [nso0] "s"(rank * SHARD + rg * BLKB), [nso1] "s"(src * SHARD), [oso0] "s"((rank * MS + rg * 16) * (NL * 4)), [oso1] "s"((src * MS - nl * 16) * (NL * 4)),
        [aS] "s"(aS), [cp] "s"(rfl(1024 - (int)item > 0 ? 1024 - item : 0)), [cnt] "s"(cnt), [tmo] "s"(tmo), [psx] "s"(p.sx), [psw] "s"(p.sw),
        [vstg] "v"(vstg), [vflag] "v"(vflag), [vexp] "v"(vexp), [rd] "v"(rd), [vpw] "v"(lds0 + PROG_OFF + wv * 4), [vpr] "v"(lds0 + PROG_OFF + (lane & 7) * 4), [vout] "v"(vout)
      : "memory", "scc", "vcc", "s64", "s65", "s66", "s67", "s68", "s69", "s70", "s71", "s72", "s73", "s74", "s75", CLOB_V);
  if (threadIdx.x == 0) __hip_atomic_store(cnt, call, __ATOMIC_RELAXED, __HIP_MEMORY_SCOPE_AGENT);
}

extern "C" void kernel_launch(int rank, int nranks, ncclComm_t comm, hipStream_t stream,
                              void* const* d_in, const int* in_sizes, int n_in,
                              void* d_out, int out_size,
                              const int* in_dtypes, int out_dtype,
                              void* const* d_out_all,
                              void* const* d_ws_all, size_t ws_bytes) {
  if (nranks != NR || rank < 0 || rank >= NR || n_in != 4 || ws_bytes < WS_NEED ||
      in_sizes[0] != (int)SHARD || in_sizes[1] != KD * NL || out_size != NR * MS * NL) {
    fprintf(stderr, "ag_gemm_silu: unexpected problem shape\n");
    return;
  }
  HIP_CHECK(hipFuncSetAttribute((const void*)ag_gemm_silu, hipFuncAttributeMaxDynamicSharedMemorySize, LDS_BYTES));
  Params p;
  p.me = (unsigned char*)d_ws_all[rank];
  for (int r = 0; r < NR; r++) p.peer[r] = (unsigned char*)d_ws_all[(rank + 1 + r) % NR];
  p.x = (const unsigned char*)d_in[0];
  p.w = (const unsigned char*)d_in[1];
  p.sx = (const float*)d_in[2];
  p.sw = (const float*)d_in[3];
  p.out = (float*)d_out;
  p.rank = rank;
  p.pad[0] = p.pad[1] = p.pad[2] = 0;
  ag_gemm_silu<<<GRID, THREADS, LDS_BYTES, stream>>>(p);
  HIP_CHECK(hipGetLastError());
}
```

```cpp
#include <hip/hip_runtime.h>
#include <rccl/rccl.h>
#include <cstdio>

#define HIP_CHECK(expr) do { hipError_t _e = (expr); if (_e != hipSuccess) { \
    fprintf(stderr, "HIP error %s at %s:%d\n", hipGetErrorString(_e), __FILE__, __LINE__); \
    return; } } while (0)

typedef int      v8i   __attribute__((ext_vector_type(8)));
typedef int      v2i   __attribute__((ext_vector_type(2)));
typedef unsigned u32x4 __attribute__((ext_vector_type(4)));
typedef float    v4f   __attribute__((ext_vector_type(4)));
#define LAS __attribute__((address_space(3)))
#define CAS __attribute__((address_space(4)))
#define DI __device__ __forceinline__

constexpr int      NR        = 8;
constexpr int      KD        = 4096, NL = 2048, MS = 512;
constexpr unsigned SHARD     = MS * KD;
constexpr unsigned BLKB      = 16 * KD;
constexpr int      NBLK      = MS / 16;
constexpr size_t   STAGE_B   = (size_t)NR * SHARD;
constexpr size_t   FLAG_OFF  = 2 * STAGE_B;
constexpr unsigned FLAG_STR  = NBLK * 1024;
constexpr size_t   CNT_OFF   = FLAG_OFF + (NR + 1) * FLAG_STR;
constexpr size_t   TMO_OFF   = CNT_OFF + 256 * 128;
constexpr size_t   DIAG_OFF  = TMO_OFF + 4096;
constexpr size_t   WS_NEED   = DIAG_OFF + 256 * 8 * 64 * 8;
constexpr int      NSEND     = 32, GRID = 256, THREADS = 512;
constexpr unsigned PROG_OFF  = 131072;
constexpr unsigned LDS_BYTES = PROG_OFF + 64;
#define DEADLINE_TICKS_LIT 100000000
constexpr int      SRD3      = 0x00020000;

#define PACE 0
#define PACE0 0
#define SD 2
#define WIN 2
#define OUT_AUX 0
#define MAPMODE 2
#define PFSWEEP

struct Params {
  unsigned char* me;
  const unsigned char* x;
  const unsigned char* w;
  const float* sx;
  const float* sw;
  float* out;
  int rank;
  int pad[3];
  unsigned char* peer[NR];
};
typedef unsigned u32x16 __attribute__((ext_vector_type(16)));
DI unsigned long long mk64(unsigned lo, unsigned hi) { return ((unsigned long long)hi << 32) | lo; }

DI unsigned rfl(unsigned v) { return __builtin_amdgcn_readfirstlane(v); }
DI __amdgpu_buffer_rsrc_t mk_rsrc(const void* p, unsigned bytes) {
  unsigned long long a = (unsigned long long)p;
  unsigned lo = rfl((unsigned)a), hi = rfl((unsigned)(a >> 32));
  return __builtin_amdgcn_make_buffer_rsrc((void*)(((unsigned long long)hi << 32) | lo), (short)0, (int)bytes, SRD3);
}

DI u32x4 mk_srd(const void* p, unsigned bytes) {
  const unsigned long long a = (unsigned long long)p;
  return u32x4{rfl((unsigned)a), rfl((unsigned)(a >> 32) & 0xffffu), bytes, (unsigned)SRD3};
}

#define STAMP(slot) do {} while (0)

#define VMCNT(n)  asm volatile("s_waitcnt vmcnt(" #n ")" ::: "memory")
#define LGKM(n)   asm volatile("s_waitcnt lgkmcnt(" #n ")" ::: "memory")
#define LGKM0()   LGKM(0)
#define BARRIER() do { __builtin_amdgcn_s_barrier(); asm volatile("" ::: "memory"); } while (0)
#define SCHED()   __builtin_amdgcn_sched_barrier(0)

#define CLOB10(p) "v" #p "0", "v" #p "1", "v" #p "2", "v" #p "3", "v" #p "4", "v" #p "5", "v" #p "6", "v" #p "7", "v" #p "8", "v" #p "9"
#define CLOB_V "v20", "v21", "v22", "v23", "v24", "v25", "v26", "v27", "v28", "v29", CLOB10(3), CLOB10(4), CLOB10(5), CLOB10(6), "v70", "v71", "v72", "v73", "v81", "v82", "v83", "v84", "v85", "v86", "v87", "v88", "v89", CLOB10(9), CLOB10(10), CLOB10(11), CLOB10(12),   \
    CLOB10(13), CLOB10(14), CLOB10(15), CLOB10(16), CLOB10(17), CLOB10(18), CLOB10(19), CLOB10(20), CLOB10(21), CLOB10(22), CLOB10(23),        \
    CLOB10(24), "v250", "v251", "v252", "v253", "v254", "v255"
#define STR_(x) #x
#define STR(x) STR_(x)
#define OUT_BITS ""
#define ASTAMP ".macro ASTAMP off, idx\n.endm\n.macro CSTAMP off\n.endm\n"
#define FAKE_AV ""
#define NPF 1
#define NEARTH 2
#define SPERM 1
#define TOPWAIT 1
#define SSLEEP 0
#define NPROBE 4
#define PTH 5
#define STAMPN 1
#define COMPUTE_ASM ".set STAMPN, " STR(STAMPN) "\n.set NEARTH, " STR(NEARTH) "\n.set TOPWAIT, " STR(TOPWAIT) "\n.set SSLEEP, " STR(SSLEEP) "\n.set PTH, " STR(PTH) "\n.set NPF, " STR(NPF) "\n" ASTAMP R"(
.macro WDMA s
  .irp j, 0,1,2,3,4,5,6,7
    s_add_u32 m0, %[lb], ((\s)&1)*8192+\j*1024
    s_add_u32 %[t0], %[sw], ((\s)*128+\j*16)*2048
    .if \j & 1
      buffer_load_dwordx4 v48, %[rW], %[t0] offen lds
    .else
      buffer_load_dwordx4 %[wv0], %[rW], %[t0] offen lds
    .endif
  .endr
.endm
.macro WPF s
  .if NPF
    s_mov_b32 %[t0], \s*128*2048
    buffer_load_dword v73, %[vpf], %[rW], %[t0] offen
  .endif
.endm
.macro WTR s
  .irp t, 0,1,2,3
    ds_read_b64_tr_b8 v[128+32*\s+8*\t:129+32*\s+8*\t], v[49+\t] offset:(\s&1)*8192
    ds_read_b64_tr_b8 v[130+32*\s+8*\t:131+32*\s+8*\t], v[49+\t] offset:(\s&1)*8192+512
    ds_read_b64_tr_b8 v[132+32*\s+8*\t:133+32*\s+8*\t], v[49+\t] offset:(\s&1)*8192+4096
    ds_read_b64_tr_b8 v[134+32*\s+8*\t:135+32*\s+8*\t], v[49+\t] offset:(\s&1)*8192+4608
  .endr
.endm
.macro LD p, i
  .if \i < 4
    buffer_load_dwordx4 v[64+32*\p+4*\i:67+32*\p+4*\i], %[vstg], s[72:75], %[nso] offen offset:1024*\i sc0 sc1
  .else
    buffer_load_dwordx4 v[64+32*\p+4*\i:67+32*\p+4*\i], %[vstg2], s[72:75], %[nso] offen offset:1024*(\i-4) sc0 sc1
  .endif
.endm
.macro LDONE p
  s_add_u32 %[ops], %[ops], 8
  s_mov_b32 s[68+\p], %[ops]
  s_add_u32 %[iss], %[iss], 1
  s_add_u32 %[nso], %[nso], %[nst]
  s_cmp_eq_u32 %[iss], %[nl]
  s_cselect_b32 %[nso], %[nso1], %[nso]
  s_cselect_b32 %[nst], 65536, %[nst]
.endm
.macro BURST p
  .irp i, 0,1,2,3,4,5,6,7
    LD \p, \i
  .endr
  LDONE \p
.endm
.macro POLL own=0
  .if \own
    buffer_load_dword v31, %[vown], %[rF], 0 offen sc0 sc1
  .else
    s_sub_u32 %[pa], %[av], %[nl]
    s_lshl_b32 %[t1], %[pa], 10
    buffer_load_dword v31, %[vflag], %[rF], %[t1] offen sc0 sc1
  .endif
.endm
.macro TOAVAIL
  s_add_u32 %[t0], %[pa], %[base]
  v_add_u32 v30, %[t0], %[vexp]
  v_cmp_ge_u32 vcc, v31, v30
  s_ff0_i32_b32 %[t0], vcc_lo
  s_lshr_b32 %[t0], %[t0], 2
  s_add_u32 %[t0], %[t0], %[pa]
  s_min_u32 %[t0], %[t0], 32
  s_add_u32 %[t0], %[t0], %[nl]
  s_max_i32 %[av], %[av], %[t0]
.endm
.macro TOOWN
  v_cmp_le_u32 vcc, %[base], v31
  s_cmp_eq_u32 vcc_lo, %[omask]
  s_cselect_b32 %[av], %[nl], 0
.endm
.macro SPIN own=0
  s_memrealtime s[64:65]
  s_mov_b32 %[lim], 0
.Lspin\@:
  s_cmp_gt_i32 %[av], %[iss]
  s_cbranch_scc1 .Lgot\@
  POLL \own
  s_waitcnt vmcnt(0)
  .if \own
    TOOWN
  .else
    TOAVAIL
  .endif
  s_add_u32 %[lim], %[lim], 1
  s_and_b32 %[t0], %[lim], 31
  s_cbranch_scc1 .Lspin\@
  v_mov_b32 v30, 0
  global_load_dword v29, v30, %[tmo] sc1
  s_memrealtime s[66:67]
  s_waitcnt vmcnt(0) lgkmcnt(0)
  v_readfirstlane_b32 %[t0], v29
  s_cmp_eq_u32 %[t0], %[call]
  s_cbranch_scc1 .Ldead\@
  s_sub_u32 %[t0], s66, s64
  s_cmp_lt_u32 %[t0], )" STR(DEADLINE_TICKS_LIT) R"(
  s_cbranch_scc1 .Lspin\@
.Ldead\@:
  v_mov_b32 v29, %[call]
  global_store_dword v30, v29, %[tmo] sc1
  v_mov_b32 %[scale], 0x7fc00000
  s_add_u32 %[av], %[nl], 32
.Lgot\@:
.endm
.macro NSTAMP
  .if STAMPN == 1
    ASTAMP 2, %[n]
  .elseif STAMPN == 2
    s_and_b32 %[t0], %[n], 7
    s_cbranch_scc1 .Lns\@
    ASTAMP 2, %[n]
.Lns\@:
  .endif
.endm
.macro MM p, s, t
  .if \s == 0
    v_mfma_f32_16x16x128_f8f6f4 v[48-16*\p+4*\t:51-16*\p+4*\t], v[128+8*\t:135+8*\t], v[64+32*\p:71+32*\p], 0
  .else
    v_mfma_f32_16x16x128_f8f6f4 v[48-16*\p+4*\t:51-16*\p+4*\t], v[128+32*\s+8*\t:135+32*\s+8*\t], v[64+32*\p+8*\s:71+32*\p+8*\s], v[48-16*\p+4*\t:51-16*\p+4*\t]
  .endif
.endm
.macro MG p, s
  .irp t, 0,1,2,3
    MM \p, \s, \t
  .endr
.endm
.macro MPROG p, base
  s_waitcnt vmcnt(\base+6)
  MG \p, 0
  s_waitcnt vmcnt(\base+4)
  MG \p, 1
  s_waitcnt vmcnt(\base+2)
  MG \p, 2
  s_waitcnt vmcnt(\base)
  POLL
  MG \p, 3
.endm
.macro TWR p, t
  .if \t == 0
    ds_write_b128 %[wr0], v[48-16*\p:51-16*\p] offset:4096*\p
  .elseif \t == 1
    ds_write_b128 %[wr1], v[52-16*\p:55-16*\p] offset:4096*\p
  .elseif \t == 2
    ds_write_b128 %[wr2], v[56-16*\p:59-16*\p] offset:4096*\p
  .else
    ds_write_b128 %[wr3], v[60-16*\p:63-16*\p] offset:4096*\p
  .endif
.endm
.macro WFLIP
  v_xor_b32 %[wr0], 8192, %[wr0]
  v_xor_b32 %[wr1], 8192, %[wr1]
  v_xor_b32 %[wr2], 8192, %[wr2]
  v_xor_b32 %[wr3], 8192, %[wr3]
.endm
.macro TADDR e
  s_and_b32 %[t1], \e, 3
  s_lshl_b32 %[t1], %[t1], 12
  v_add_u32 v28, %[t1], %[rd]
  s_lshl_b32 %[oso], \e, 17
  s_add_u32 %[oso], %[oso], %[oso1]
.endm
.macro TOWNROW e
  s_mul_i32 %[oso], \e, 7*16*8192
  s_add_u32 %[oso], %[oso], %[oso0]
  s_cmp_lt_i32 \e, 0
  s_cselect_b32 %[oso], 0x40000000, %[oso]
.endm
.macro RD r, a, v
  ds_read_b128 v[\r:\r+3], \a offset:16384*\v
.endm
.macro OP4 op, d, a, b
  .irp i, 0,1,2,3
    \op v[\d+\i], v[\a+\i], v[\b+\i]
  .endr
.endm
.macro K4 op, d, k, a
  .irp i, 0,1,2,3
    \op v[\d+\i], \k, v[\a+\i]
  .endr
.endm
.macro U4 op, d, a
  .irp i, 0,1,2,3
    \op v[\d+\i], v[\a+\i]
  .endr
.endm
.macro MED4
  .irp i, 0,1,2,3
    v_med3_f32 v[24+\i], v[20+\i], -s70, s70
  .endr
.endm
.macro TSTORE
  buffer_store_dwordx4 v[24:27], %[vout], %[rO], %[oso] offen)" OUT_BITS R"(
.endm
.macro TFIN
  K4 v_mul_f32, 20, %[scale], 20
  MED4
  K4 v_mul_f32, 24, 0xbfb8aa3b, 24
  U4 v_exp_f32, 24, 24
  K4 v_add_f32, 24, 1.0, 24
  U4 v_rcp_f32, 24, 24
  OP4 v_mul_f32, 24, 20, 24
  TSTORE
.endm
.macro T tl, x:vararg
  .if \tl
    \x
  .endif
.endm
.macro LD7 p
  LD \p, 7
  s_add_u32 %[nso], %[nso], %[nst]
  s_add_u32 %[t0], %[n], 2
  s_cmp_eq_u32 %[t0], %[nl]
  s_cselect_b32 %[nso], %[nso1], %[nso]
  s_cselect_b32 %[nst], 65536, %[nst]
.endm
.macro HOT p, q, g, l
.Lhot\p\g\l:
  NSTAMP
  T \l==1, s_waitcnt vmcnt(13)
  T \l==4, s_waitcnt vmcnt(6)
  T \l==3, s_waitcnt vmcnt(6)
  MM \p, 0, 0
  T \l==3, POLL
  T \l==3, LD \q, 0
  T \q==\g, K4 v_mul_f32, 20, %[scale], 20
  T \p==\g, s_nop 0
  TWR \q, 0
  MM \p, 0, 1
  T \l==3, LD \q, 1
  T \l==1, LD7 \q
  T \q==\g, MED4
  MM \p, 0, 2
  T \l==3, LD \q, 2
  TWR \q, 1
  T \q==\g, K4 v_mul_f32, 24, 0xbfb8aa3b, 24
  MM \p, 0, 3
  T \l==3, LD \q, 3
  T (\l==1)||(\l==2), LD \p, 0
  T \q==\g, U4 v_exp_f32, 24, 24
  T \l==1, s_waitcnt vmcnt(13)
  T \l==4, s_waitcnt vmcnt(4)
  T \l==3, s_waitcnt vmcnt(9)
  MM \p, 1, 0
  T \l==3, LD \q, 4
  TWR \q, 2
  T \q==\g, K4 v_add_f32, 24, 1.0, 24
  MM \p, 1, 1
  T \l==3, LD \q, 5
  T (\l==1)||(\l==2), LD \p, 1
  T \q==\g, U4 v_rcp_f32, 24, 24
  v_mov_b32 v30, %[n]
  MM \p, 1, 2
  T \l==3, LD \q, 6
  TWR \q, 3
  ds_write_b32 %[vpw], v30
  ds_read_b32 v29, %[vpr]
  T \q==\g, OP4 v_mul_f32, 24, 20, 24
  MM \p, 1, 3
  T \l==3, LD \q, 7
  T \l==3, LDONE \q
  T (\l==1)||(\l==2), LD \p, 2
  s_sub_u32 %[tp], %[n], 2
  T \p==\g, TADDR %[tp]
  T \p==\g, s_cmp_lt_i32 %[tp], %[nl]
  T \p==\g, s_cbranch_scc1 .Lown\p\g\l
.Lowned\p\g\l:
  T \l==1, s_waitcnt vmcnt(13)
  T \l==4, s_waitcnt vmcnt(2)
  T \l==3, s_waitcnt vmcnt(11)
  MM \p, 2, 0
  T \q==\g, TSTORE
  s_waitcnt lgkmcnt(0)
  v_cmp_lt_i32 vcc, %[tp], v29
  s_cmp_eq_u32 vcc_lo, -1
  s_cbranch_scc0 .Lwait\p\g\l
.Lwaited\p\g\l:
  T \p==\g, RD 20, v28, 0
  T \p==\g, RD 32+16*\p, v28, 1
  T \p==\g, RD 36+16*\p, v28, 2
  T \p==\g, RD 40+16*\p, v28, 3
  MM \p, 2, 1
  T (\l==1)||(\l==2), LD \p, 3
  T \p==\g, v_add_u32 v29, 0x10000, v28
  T \p==\g, RD 44+16*\p, v29, 0
  T \p==\g, RD 24, v29, 1
  MM \p, 2, 2
  .if \q
    WFLIP
  .endif
  .if (\l==1)||(\l==2)
    s_add_u32 %[t0], %[n], PTH
    s_cmp_gt_i32 %[t0], %[av]
    s_cbranch_scc1 .Lpoll\p\g\l
.Lpolled\p\g\l:
  .elseif \l != 3
    POLL
  .endif
  MM \p, 2, 3
  T (\l==1)||(\l==2), LD \p, 4
  T \p==\g, s_waitcnt lgkmcnt(2)
  T \p==\g, OP4 v_add_f32, 20, 20, 32+16*\p
  T \p==\g, RD 32+16*\p, v29, 2
  T \l==1, s_waitcnt vmcnt(13)
  T \l==4, s_waitcnt vmcnt(1+((\q==\g)&1))
  T \l==3, s_waitcnt vmcnt(9+((\q==\g)&1))
  MM \p, 3, 0
  T \p==\g, OP4 v_add_f32, 36+16*\p, 36+16*\p, 40+16*\p
  T \p==\g, RD 40+16*\p, v29, 3
  MM \p, 3, 1
  T (\l==1)||(\l==2), LD \p, 5
  T \p==\g, OP4 v_add_f32, 20, 20, 36+16*\p
  MM \p, 3, 2
  T \p==\g, s_waitcnt lgkmcnt(2)
  T \p==\g, OP4 v_add_f32, 24, 44+16*\p, 24
  T \p==\g, OP4 v_add_f32, 20, 20, 24
  s_add_u32 %[n], %[n], 1
  s_add_u32 %[t0], %[n], 2
  MM \p, 3, 3
  T (\l==1)||(\l==2), LD \p, 6
  T \p==\g, s_waitcnt lgkmcnt(0)
  T \p==\g, OP4 v_add_f32, 24, 32+16*\p, 40+16*\p
  T \p==\g, OP4 v_add_f32, 20, 20, 24
  .if (\l==1)||(\l==2)
    s_cmp_ge_i32 %[t0], %[av]
    s_cbranch_scc1 .Lexit\p\g
    T \l==2, s_branch .Lhot\q\g\()1
  .else
    T \q==\g, s_add_u32 %[ops], %[ops], 1
    s_sub_u32 %[t0], %[n], 2-((\p==\g)&1)
    s_max_i32 %[tq], %[tq], %[t0]
    T \l==3, s_branch .Ltw\q\g
    T \l!=3, s_branch .Ltop\q\g
  .endif
.endm
.macro STUBS p, g, l
  .if (\l==1)||(\l==2)
.Lpoll\p\g\l:
    POLL
    s_branch .Lpolled\p\g\l
  .endif
.Lown\p\g\l:
  TOWNROW %[tp]
  s_branch .Lowned\p\g\l
.Lwait\p\g\l:
  ds_read_b32 v29, %[vpr]
  s_waitcnt lgkmcnt(0)
  v_cmp_lt_i32 vcc, %[tp], v29
  s_cmp_eq_u32 vcc_lo, -1
  s_cbranch_scc0 .Lwait\p\g\l
  s_branch .Lwaited\p\g\l
.endm
.macro COLD p, q, g
  STUBS \p, \g, 0
  STUBS \p, \g, 1
  STUBS \p, \g, 2
  STUBS \p, \g, 3
  STUBS \p, \g, 4
.Lexit\p\g:
  TOAVAIL
  s_add_u32 %[iss], %[n], 2
  s_cmp_lt_i32 %[iss], %[av]
  s_cbranch_scc1 .Lhot\q\g\()1
  LD7 \p
  s_mov_b32 s[68+\p], %[ops]
  s_sub_u32 s[69-\p], %[ops], 8
  s_sub_u32 %[t0], %[n], 2-((\p==\g)&1)
  s_max_i32 %[tq], %[tq], %[t0]
  s_waitcnt vmcnt(8)
  s_branch .Lhot\q\g\()0
.Ltop\p\g:
  .if TOPWAIT
    s_add_u32 %[t0], %[n], 2
    s_cmp_eq_u32 %[iss], %[t0]
    s_cbranch_scc1 .Ltw\p\g
    s_waitcnt vmcnt(0)
.Ltw\p\g:
  .endif
  TOAVAIL
  s_cmp_eq_u32 %[iss], %[n]
  s_cbranch_scc1 .Lcold\p\g
  s_add_u32 %[t0], %[n], 1
  s_cmp_lg_u32 %[iss], %[t0]
  s_cbranch_scc1 .Lt2\p\g
  s_cmp_ge_i32 %[iss], %[av]
  s_cbranch_scc1 .Lhot\p\g\()4
  s_add_u32 %[t0], %[n], NEARTH
  s_cmp_le_i32 %[av], %[t0]
  s_cbranch_scc1 .Lhot\p\g\()3
  BURST \q
.Lt2\p\g:
  s_waitcnt vmcnt(8)
  s_cmp_lt_i32 %[iss], %[av]
  s_cbranch_scc1 .Lhot\p\g\()2
  s_branch .Lhot\p\g\()0
.Lcold\p\g:
  s_cmp_ge_i32 %[iss], %[av]
  s_cbranch_scc1 .Lc1\p\g
  BURST \p
.Lc1\p\g:
  v_mov_b32 v30, %[n]
  TWR \q, 0
  TWR \q, 1
  TWR \q, 2
  TWR \q, 3
  ds_write_b32 %[vpw], v30
  .if \q==\g
    TFIN
    s_add_u32 %[ops], %[ops], 1
  .endif
  s_cmp_gt_u32 %[n], %[last]
  s_cbranch_scc1 .Lc3\p\g
  s_cmp_lg_u32 %[iss], %[n]
  s_cbranch_scc1 .Lc2\p\g
  SPIN
  BURST \p
.Lc2\p\g:
  s_cmp_ge_i32 %[iss], %[av]
  s_cbranch_scc1 .Lc3\p\g
  BURST \q
.Lc3\p\g:
  s_cmp_ge_i32 %[tq], %[n]
  s_cbranch_scc1 .Lc5\p\g
  TADDR %[tq]
  s_cmp_ge_i32 %[tq], %[nl]
  s_cbranch_scc1 .Lc4\p\g
  TOWNROW %[tq]
.Lc4\p\g:
  ds_read_b32 v29, %[vpr]
  s_waitcnt lgkmcnt(0)
  v_cmp_lt_i32 vcc, %[tq], v29
  s_cmp_eq_u32 vcc_lo, -1
  s_cbranch_scc0 .Lc4\p\g
  RD 20, v28, 0
  RD 24, v28, 1
  RD 32+16*\q, v28, 2
  RD 36+16*\q, v28, 3
  v_add_u32 v29, 0x10000, v28
  RD 40+16*\q, v29, 0
  RD 44+16*\q, v29, 1
  s_waitcnt lgkmcnt(4)
  OP4 v_add_f32, 20, 20, 24
  RD 24, v29, 2
  s_waitcnt lgkmcnt(3)
  OP4 v_add_f32, 32+16*\q, 32+16*\q, 36+16*\q
  RD 36+16*\q, v29, 3
  OP4 v_add_f32, 20, 20, 32+16*\q
  s_waitcnt lgkmcnt(2)
  OP4 v_add_f32, 40+16*\q, 40+16*\q, 44+16*\q
  OP4 v_add_f32, 20, 20, 40+16*\q
  s_waitcnt lgkmcnt(0)
  OP4 v_add_f32, 24, 24, 36+16*\q
  OP4 v_add_f32, 20, 20, 24
  TFIN
  s_add_u32 %[ops], %[ops], 1
  s_add_u32 %[tq], %[tq], 2
  s_branch .Lc3\p\g
.Lc5\p\g:
  s_cmp_gt_u32 %[n], %[last]
  s_cbranch_scc1 .Ldone
  .if \q
    WFLIP
  .endif
  s_mov_b32 %[oso], 0x40000000
  NSTAMP
  s_sub_u32 %[t0], %[ops], s[68+\p]
  s_add_u32 %[n], %[n], 1
  s_cmp_ge_u32 %[t0], 8
  s_cbranch_scc1 .Lc8\p\g
  s_cmp_ge_u32 %[t0], 2
  s_cbranch_scc1 .Lc7\p\g
  s_cmp_ge_u32 %[t0], 1
  s_cbranch_scc1 .Lc9\p\g
  MPROG \p, 0
  s_branch .Ltop\q\g
.Lc9\p\g:
  MPROG \p, 1
  s_branch .Ltop\q\g
.Lc7\p\g:
  MPROG \p, 2
  s_branch .Ltop\q\g
.Lc8\p\g:
  MPROG \p, 8
  s_branch .Ltop\q\g
.endm
.macro MAIN g
  HOT 0, 1, \g, 1
  HOT 1, 0, \g, 1
  s_branch .Lhot0\g\()1
  HOT 0, 1, \g, 2
  HOT 1, 0, \g, 2
  HOT 0, 1, \g, 0
  HOT 1, 0, \g, 0
  HOT 0, 1, \g, 3
  HOT 1, 0, \g, 3
  HOT 0, 1, \g, 4
  HOT 1, 0, \g, 4
  COLD 0, 1, \g
  COLD 1, 0, \g
.endm

  v_mov_b32 v30, 0
  ds_write_b32 %[vpw], v30
  s_waitcnt lgkmcnt(0)
  s_load_dword %[call], %[cnt], 0
  s_load_dword %[t1], %[psx], 0
  s_load_dword %[t2], %[psw], 0
  s_barrier
  v_xor_b32 v48, 32, %[wv0]
  s_cmp_eq_u32 %[cp], 0
  s_cbranch_scc1 .Lnocopy
  buffer_load_dwordx4 v[64:67], %[vcs], %[rX], 0 offen
  buffer_load_dwordx4 v[68:71], %[vcs], %[rX], 0 offen offset:64
.Lnocopy:
  WPF 0
  WPF 1
  s_cmp_eq_u32 %[cp], 0
  s_cbranch_scc0 .Lcopy1
  WDMA 0
  WPF 2
  WDMA 1
  WPF 3
.Lcopy1:
  v_mov_b32 v49, %[tr0]
  v_xor_b32 v50, 16, %[tr0]
  v_xor_b32 v51, 32, %[tr0]
  v_xor_b32 v52, 48, %[tr0]
  v_add_u32 %[wr0], 8192, %[wr0]
  s_mov_b32 %[oso], 0x40000000
  v_xor_b32 %[wr1], 64, %[wr0]
  v_xor_b32 %[wr2], 128, %[wr0]
  v_xor_b32 %[wr3], 192, %[wr0]
  v_add_u32 %[vstg2], 4096, %[vstg]
  s_mov_b32 s70, 0x42700000
  s_mov_b32 %[n], 0
  s_mov_b32 %[iss], 0
  s_mov_b32 %[ops], 0
  s_mov_b32 %[nso], %[nso0]
  s_mov_b32 %[nst], 7*65536
  s_mov_b32 %[tq], %[grp]
  s_mov_b32 %[pa], 0
  s_waitcnt lgkmcnt(0)
  s_add_u32 %[call], %[call], 1
  s_lshl_b32 %[base], %[call], 6
  v_mov_b32 %[scale], %[t1]
  v_mul_f32 %[scale], %[t2], %[scale]
  s_and_b32 %[t0], %[call], 1
  s_lshl_b32 %[t0], %[t0], 24
  s_mov_b64 s[72:73], %[aS]
  s_add_u32 s72, s72, %[t0]
  s_addc_u32 s73, s73, 0
  s_mov_b32 s74, 0x1000000
  s_mov_b32 s75, 0x20000
  s_cmp_eq_u32 %[cp], 0
  s_cbranch_scc1 .Lcopied
  v_mov_b32 v72, %[base]
  WPF 2
  WPF 3
  s_waitcnt vmcnt(4*NPF)
  buffer_store_dwordx4 v[64:67], %[vcd], s[72:75], 0 offen sc0 sc1
  buffer_store_dwordx4 v[68:71], %[vcd], s[72:75], 0 offen offset:1024 sc0 sc1
  s_waitcnt vmcnt(0)
  buffer_store_dword v72, %[vcf], %[rF], 0 offen sc0 sc1
  WDMA 0
  WDMA 1
.Lcopied:
  ASTAMP 42, 0
  s_waitcnt vmcnt(8)
  WTR 0
  ASTAMP 41, 0
  s_waitcnt lgkmcnt(0)
  WDMA 2
  s_waitcnt vmcnt(8)
  WTR 1
  s_waitcnt lgkmcnt(0)
  WDMA 3
  POLL 1
  s_waitcnt vmcnt(9)
  WTR 2
  s_waitcnt vmcnt(0)
  WTR 3
  ASTAMP 43, 0
  TOOWN
)" FAKE_AV R"(
  SPIN 1
  ASTAMP 44, 0
  v_mov_b32 v31, 0
  BURST 0
  BURST 1
  s_waitcnt lgkmcnt(0)
  ASTAMP 1, 0
  CSTAMP 48
  s_cmp_eq_u32 %[tq], 0
  s_cbranch_scc1 .Ltop00
  s_branch .Ltop01
  MAIN 0
  MAIN 1
.Ldone:
  CSTAMP 49
)"

__global__ __launch_bounds__(THREADS, 2) void ag_gemm_silu(Params) {
  extern __shared__ __attribute__((aligned(16))) unsigned char smem[];
  const int b = blockIdx.x, lane = threadIdx.x & 63;
  const int wv = (int)rfl(threadIdx.x >> 6);
  const int si = SPERM ? (b & 7) * (NSEND / 8) + (b >> 3) : b, u0 = (si * 7 + wv) * 2;
  const unsigned li = ((unsigned)u0 >> 6) & (NR - 1), un = (unsigned)u0 & 63;
  u32x16 ka;
  unsigned long long pw_;
  asm volatile("s_load_dwordx16 %0, %2, 0\n\ts_load_dwordx2 %1, %2, %3\n\ts_waitcnt lgkmcnt(0)"
               : "=&s"(ka), "=&s"(pw_) : "s"(__builtin_amdgcn_kernarg_segment_ptr()), "s"(64 + li * 8));
  Params p;
  p.me = (unsigned char*)mk64(ka[0], ka[1]); p.x = (const unsigned char*)mk64(ka[2], ka[3]); p.w = (const unsigned char*)mk64(ka[4], ka[5]);
  p.sx = (const float*)mk64(ka[6], ka[7]); p.sw = (const float*)mk64(ka[8], ka[9]); p.out = (float*)mk64(ka[10], ka[11]);
  const unsigned rank = ka[12] & (NR - 1);
  unsigned char* me = p.me;
  unsigned* cnt = (unsigned*)(me + CNT_OFF + (size_t)b * 128);
  unsigned* tmo = (unsigned*)(me + TMO_OFF);

  if (b < NSEND) {
    if (wv == 7) return;
    const unsigned svo = (lane & 15) * KD + (lane >> 4) * 16 + un * 64, dvo = lane * 16 + un * 1024;
    const int first = max(si * 14, (int)li * 64), last = min((si + 1) * 14, (int)(li + 1) * 64), ks = un / 2 + lane;
    const unsigned fo = (u0 == first && lane < (last - first) / 2) ? (ks >> 2) * 128 + (ks & 3) * 4 : 0x80000000u;
    const u32x4 rX = mk_srd(p.x, SHARD), rF = mk_srd((void*)(pw_ + FLAG_OFF + rank * FLAG_STR), FLAG_STR);
    unsigned call, so, fs, t0;
    asm volatile(R"(
.macro SLD r, d
  s_add_u32 %[t0], %[so], \d*65536
  buffer_load_dwordx4 v[32+8*\r:35+8*\r], %[svo], %[rX], %[t0] offen
  buffer_load_dwordx4 v[36+8*\r:39+8*\r], %[svo], %[rX], %[t0] offen offset:64
.endm
.macro SST r
  buffer_store_dwordx4 v[32+8*\r:35+8*\r], %[dvo], s[64:67], %[so] offen sc0 sc1
  buffer_store_dwordx4 v[36+8*\r:39+8*\r], %[dvo], s[64:67], %[so] offen offset:1024 sc0 sc1
.endm
.macro SPUB w
  s_waitcnt vmcnt(\w)
  s_barrier
  buffer_store_dword v64, %[fo], %[rF], %[fs] offen sc0 sc1
  s_add_u32 %[fs], %[fs], 1024
  s_add_u32 %[so], %[so], 65536
  v_add_u32 v64, 1, v64
.endm
  s_load_dword %[call], %[cnt], 0
  s_mov_b32 %[so], 0
  s_mov_b32 %[fs], 0
  SLD 0, 0
  SLD 1, 1
  s_mov_b64 s[64:65], %[aD]
  s_mov_b32 s66, 0x200000
  s_mov_b32 s67, 0x20000
  s_waitcnt lgkmcnt(0)
  s_add_u32 %[call], %[call], 1
  s_and_b32 %[t0], %[call], 1
  s_lshl_b32 %[t0], %[t0], 24
  s_add_u32 s64, s64, %[t0]
  s_addc_u32 s65, s65, 0
  s_lshl_b32 %[t0], %[call], 6
  v_mov_b32 v64, %[t0]
  v_add_u32 v64, 1, v64
  s_waitcnt vmcnt(2)
  SST 0
  SLD 2, 2
  SLD 3, 3
  s_add_u32 %[so], %[so], 65536
  s_waitcnt vmcnt(6)
  SST 1
  SLD 0, 3
  SPUB 8
  s_waitcnt vmcnt(7)
.Lsend:
  SST 2
  SLD 1, 3
  SPUB 7
  SST 3
  SLD 2, 3
  SPUB 7
  s_cmp_eq_u32 %[fs], 31*1024
  s_cbranch_scc1 .Lsent
  SST 0
  SLD 3, 3
  SPUB 7
  SST 1
  SLD 0, 3
  SPUB 7
  s_branch .Lsend
.Lsent:
  SPUB 0
)"      : [call] "=&s"(call), [so] "=&s"(so), [fs] "=&s"(fs), [t0] "=&s"(t0)
        : [rX] "s"(rX), [rF] "s"(rF), [cnt] "s"(cnt), [aD] "s"((pw_ + (size_t)rank * SHARD) & 0xffffffffffffull),
          [svo] "v"(svo), [dvo] "v"(dvo), [fo] "v"(fo)
        : "memory", "scc", "s64", "s65", "s66", "s67", "v32", "v33", "v34", "v35", "v36", "v37", "v38", "v39", CLOB10(4), CLOB10(5),
          "v60", "v61", "v62", "v63", "v64");
    if (threadIdx.x == 0) __hip_atomic_store(cnt, call, __ATOMIC_RELAXED, __HIP_MEMORY_SCOPE_AGENT);
    STAMP(2);
    return;
  }

  const int xcd = b & 7, slot = (b >> 3) - NSEND / 8;
  int cg, rg;
  { const int id = (xcd >> 1) * 28 + slot; cg = (xcd & 1) * 16 + (id & 15); rg = id >> 4; }
  const unsigned lds0 = (unsigned)(unsigned long long)(LAS unsigned char*)smem;
  const unsigned src = (rank + 1 + rg) & (NR - 1);
  const int q = lane >> 4, m = lane & 15;
  unsigned wvo0 = (lane >> 2) * NL + cg * 64 + (((lane & 3) ^ (q & 1)) * 16);
  unsigned tr0 = lds0 + wv * 16384 + (16 * q + (m >> 1)) * 64 + ((((m >> 3) & 1) | ((q & 1) << 1)) * 16) + 8 * (lane & 1);
  const unsigned vstg = lane * 16 + wv * 8192;
  const int nl = (rg < 4) ? 5 : 4;
  const unsigned vflag = lane < 4 * NPROBE ? src * FLAG_STR + (lane >> 2) * 1024 + wv * 128 + (lane & 3) * 4 : 0x80000000u;
  const unsigned vexp = (lane >> 2) + 1;
  unsigned vown = lane < 4 * nl ? rank * FLAG_STR + (rg + 7 * (lane >> 2)) * 1024 + wv * 128 + (lane & 3) * 4 : 0x80000000u;
  const unsigned item = wv * (GRID - NSEND) + (unsigned)(b - NSEND);
  unsigned vcs = (item >> 5) * BLKB + m * KD + (item & 31) * 128 + q * 16;
  unsigned vcd = rank * SHARD + item * 2048 + lane * 16;
  unsigned vcf = lane == 0 ? rank * FLAG_STR + (item >> 2) * 128 + (item & 3) * 4 : 0x80000000u;
  constexpr int LPR = MAPMODE == 0 ? 2 : MAPMODE == 3 ? 4 : 8, PL = (1024 * LPR + 223) / 224;
  const unsigned pfl = (slot * 8 + wv) * PL + lane, pfr = pfl / LPR;
  unsigned vpf = (lane < PL && pfr < 1024) ? ((pfr >> 7) * 512 + (pfr & 127)) * NL + ((cg >> 1) / LPR * LPR + pfl % LPR) * 128 : 0x80000000u;
  unsigned wr0 = lds0 + wv * 16384 + m * 256 + ((q ^ m) * 16);
  const int m_rd = 4 * (wv & 3) + q;
  const unsigned rd = lds0 + m_rd * 256 + ((m ^ m_rd) * 16);
  const unsigned vout = m_rd * (NL * 4) + cg * 256 + m * 16;
  const unsigned long long aS = (unsigned long long)me & 0xffffffffffffull;
  const u32x4 rW = mk_srd(p.w, KD * NL), rX = mk_srd(p.x, SHARD);
  const u32x4 rF = mk_srd(me + FLAG_OFF, (NR + 1) * FLAG_STR), rO = mk_srd(p.out, (unsigned)NR * MS * NL * 4u);
  unsigned call, base, n, iss, av, lim, ops, nso, nst, oso, tp, tq, pa, t0, t1, t2, t3;
  unsigned vstg2, wr1, wr2, wr3, scale;
  asm volatile(COMPUTE_ASM
      : [call] "=&s"(call), [base] "=&s"(base), [n] "=&s"(n), [iss] "=&s"(iss), [av] "=&s"(av), [lim] "=&s"(lim), [ops] "=&s"(ops),
        [nso] "=&s"(nso), [nst] "=&s"(nst), [oso] "=&s"(oso), [pa] "=&s"(pa), [tp] "=&s"(tp), [tq] "=&s"(tq), [t0] "=&s"(t0), [t1] "=&s"(t1), [t2] "=&s"(t2), [t3] "=&s"(t3),
        [wr0] "+v"(wr0), [vown] "+{v80}"(vown), [vcf] "+{v78}"(vcf), [vcd] "+{v77}"(vcd), [vcs] "+{v76}"(vcs), [tr0] "+{v75}"(tr0), [wv0] "+{v74}"(wvo0), [vpf] "+{v79}"(vpf), [vstg2] "=&v"(vstg2), [wr1] "=&v"(wr1), [wr2] "=&v"(wr2), [wr3] "=&v"(wr3), [scale] "=&v"(scale)
      : [rW] "s"(rW), [rX] "s"(rX), [rF] "s"(rF), [rO] "s"(rO), [lb] "s"(rfl(lds0 + wv * 16384)), [sw] "s"(rfl(wv * 512 * NL)),
        [nl] "s"(nl), [grp] "s"(rfl(wv >> 2)), [omask] "s"((1u << (4 * nl)) - 1), [last] "s"(nl + NBLK - 1),
        [nso0] "s"(rank * SHARD + rg * BLKB), [nso1] "s"(src * SHARD), [oso0] "s"((rank * MS + rg * 16) * (NL * 4)), [oso1] "s"((src * MS - nl * 16) * (NL * 4)),
        [aS] "s"(aS), [cp] "s"(rfl(1024 - (int)item > 0 ? 1024 - item : 0)), [cnt] "s"(cnt), [tmo] "s"(tmo), [psx] "s"(p.sx), [psw] "s"(p.sw),
        [vstg] "v"(vstg), [vflag] "v"(vflag), [vexp] "v"(vexp), [rd] "v"(rd), [vpw] "v"(lds0 + PROG_OFF + wv * 4), [vpr] "v"(lds0 + PROG_OFF + (lane & 7) * 4), [vout] "v"(vout)
      : "memory", "scc", "vcc", "s64", "s65", "s66", "s67", "s68", "s69", "s70", "s71", "s72", "s73", "s74", "s75", CLOB_V);
  if (threadIdx.x == 0) __hip_atomic_store(cnt, call, __ATOMIC_RELAXED, __HIP_MEMORY_SCOPE_AGENT);
}

extern "C" void kernel_launch(int rank, int nranks, ncclComm_t comm, hipStream_t stream,
                              void* const* d_in, const int* in_sizes, int n_in,
                              void* d_out, int out_size,
                              const int* in_dtypes, int out_dtype,
                              void* const* d_out_all,
                              void* const* d_ws_all, size_t ws_bytes) {
  if (nranks != NR || rank < 0 || rank >= NR || n_in != 4 || ws_bytes < WS_NEED ||
      in_sizes[0] != (int)SHARD || in_sizes[1] != KD * NL || out_size != NR * MS * NL) {
    fprintf(stderr, "ag_gemm_silu: unexpected problem shape\n");
    return;
  }
  HIP_CHECK(hipFuncSetAttribute((const void*)ag_gemm_silu, hipFuncAttributeMaxDynamicSharedMemorySize, LDS_BYTES));
  Params p;
  p.me = (unsigned char*)d_ws_all[rank];
  for (int r = 0; r < NR; r++) p.peer[r] = (unsigned char*)d_ws_all[(rank + 1 + r) % NR];
  p.x = (const unsigned char*)d_in[0];
  p.w = (const unsigned char*)d_in[1];
  p.sx = (const float*)d_in[2];
  p.sw = (const float*)d_in[3];
  p.out = (float*)d_out;
  p.rank = rank;
  p.pad[0] = p.pad[1] = p.pad[2] = 0;
  ag_gemm_silu<<<GRID, THREADS, LDS_BYTES, stream>>>(p);
  HIP_CHECK(hipGetLastError());
}
```

```cpp
#include <hip/hip_runtime.h>
#include <rccl/rccl.h>
#include <cstdio>

#define HIP_CHECK(expr) do { hipError_t _e = (expr); if (_e != hipSuccess) { \
    fprintf(stderr, "HIP error %s at %s:%d\n", hipGetErrorString(_e), __FILE__, __LINE__); \
    return; } } while (0)

typedef int      v8i   __attribute__((ext_vector_type(8)));
typedef int      v2i   __attribute__((ext_vector_type(2)));
typedef unsigned u32x4 __attribute__((ext_vector_type(4)));
typedef float    v4f   __attribute__((ext_vector_type(4)));
#define LAS __attribute__((address_space(3)))
#define CAS __attribute__((address_space(4)))
#define DI __device__ __forceinline__

constexpr int      NR        = 8;
constexpr int      KD        = 4096, NL = 2048, MS = 512;
constexpr unsigned SHARD     = MS * KD;
constexpr unsigned BLKB      = 16 * KD;
constexpr int      NBLK      = MS / 16;
constexpr size_t   STAGE_B   = (size_t)NR * SHARD;
constexpr size_t   FLAG_OFF  = 2 * STAGE_B;
constexpr unsigned FLAG_STR  = NBLK * 1024;
constexpr size_t   CNT_OFF   = FLAG_OFF + (NR + 1) * FLAG_STR;
constexpr size_t   TMO_OFF   = CNT_OFF + 256 * 128;
constexpr size_t   DIAG_OFF  = TMO_OFF + 4096;
constexpr size_t   WS_NEED   = DIAG_OFF + 256 * 8 * 64 * 8;
constexpr int      NSEND     = 32, GRID = 256, THREADS = 512;
constexpr unsigned PROG_OFF  = 131072;
constexpr unsigned LDS_BYTES = PROG_OFF + 64;
#define DEADLINE_TICKS_LIT 100000000
constexpr int      SRD3      = 0x00020000;

#define PACE 0
#define PACE0 0
#define SD 2
#define WIN 2
#define OUT_AUX 17
#define MAPMODE 2
#define PFSWEEP

struct Params {
  unsigned char* me;
  const unsigned char* x;
  const unsigned char* w;
  const float* sx;
  const float* sw;
  float* out;
  int rank;
  int pad[3];
  unsigned char* peer[NR];
};
typedef unsigned u32x16 __attribute__((ext_vector_type(16)));
DI unsigned long long mk64(unsigned lo, unsigned hi) { return ((unsigned long long)hi << 32) | lo; }

DI unsigned rfl(unsigned v) { return __builtin_amdgcn_readfirstlane(v); }
DI __amdgpu_buffer_rsrc_t mk_rsrc(const void* p, unsigned bytes) {
  unsigned long long a = (unsigned long long)p;
  unsigned lo = rfl((unsigned)a), hi = rfl((unsigned)(a >> 32));
  return __builtin_amdgcn_make_buffer_rsrc((void*)(((unsigned long long)hi << 32) | lo), (short)0, (int)bytes, SRD3);
}

DI u32x4 mk_srd(const void* p, unsigned bytes) {
  const unsigned long long a = (unsigned long long)p;
  return u32x4{rfl((unsigned)a), rfl((unsigned)(a >> 32) & 0xffffu), bytes, (unsigned)SRD3};
}

#define STAMP(slot) do {} while (0)

#define VMCNT(n)  asm volatile("s_waitcnt vmcnt(" #n ")" ::: "memory")
#define LGKM(n)   asm volatile("s_waitcnt lgkmcnt(" #n ")" ::: "memory")
#define LGKM0()   LGKM(0)
#define BARRIER() do { __builtin_amdgcn_s_barrier(); asm volatile("" ::: "memory"); } while (0)
#define SCHED()   __builtin_amdgcn_sched_barrier(0)

#define CLOB10(p) "v" #p "0", "v" #p "1", "v" #p "2", "v" #p "3", "v" #p "4", "v" #p "5", "v" #p "6", "v" #p "7", "v" #p "8", "v" #p "9"
#define CLOB_V "v20", "v21", "v22", "v23", "v24", "v25", "v26", "v27", "v28", "v29", CLOB10(3), CLOB10(4), CLOB10(5), CLOB10(6), "v70", "v71", "v72", "v73", "v81", "v82", "v83", "v84", "v85", "v86", "v87", "v88", "v89", CLOB10(9), CLOB10(10), CLOB10(11), CLOB10(12),   \
    CLOB10(13), CLOB10(14), CLOB10(15), CLOB10(16), CLOB10(17), CLOB10(18), CLOB10(19), CLOB10(20), CLOB10(21), CLOB10(22), CLOB10(23),        \
    CLOB10(24), "v250", "v251", "v252", "v253", "v254", "v255"
#define STR_(x) #x
#define STR(x) STR_(x)
#define OUT_BITS " sc0 sc1"
#define ASTAMP ".macro ASTAMP off, idx\n.endm\n.macro CSTAMP off\n.endm\n"
#define FAKE_AV ""
#define NPF 1
#define NEARTH 2
#define SPERM 1
#define TOPWAIT 1
#define SSLEEP 0
#define CODEPF 1
#define NPROBE 4
#define PTH 5
#define STAMPN 1
#define COMPUTE_ASM ".set STAMPN, " STR(STAMPN) "\n.set NEARTH, " STR(NEARTH) "\n.set TOPWAIT, " STR(TOPWAIT) "\n.set SSLEEP, " STR(SSLEEP) "\n.set CODEPF, " STR(CODEPF) "\n.set PTH, " STR(PTH) "\n.set NPF, " STR(NPF) "\n" ASTAMP R"(
.macro WDMA s
  .irp j, 0,1,2,3,4,5,6,7
    s_add_u32 m0, %[lb], ((\s)&1)*8192+\j*1024
    s_add_u32 %[t0], %[sw], ((\s)*128+\j*16)*2048
    .if \j & 1
      buffer_load_dwordx4 v48, %[rW], %[t0] offen lds
    .else
      buffer_load_dwordx4 %[wv0], %[rW], %[t0] offen lds
    .endif
  .endr
.endm
.macro WPF s
  .if NPF
    s_mov_b32 %[t0], \s*128*2048
    buffer_load_dword v73, %[vpf], %[rW], %[t0] offen
  .endif
.endm
.macro WTR s
  .irp t, 0,1,2,3
    ds_read_b64_tr_b8 v[128+32*\s+8*\t:129+32*\s+8*\t], v[49+\t] offset:(\s&1)*8192
    ds_read_b64_tr_b8 v[130+32*\s+8*\t:131+32*\s+8*\t], v[49+\t] offset:(\s&1)*8192+512
    ds_read_b64_tr_b8 v[132+32*\s+8*\t:133+32*\s+8*\t], v[49+\t] offset:(\s&1)*8192+4096
    ds_read_b64_tr_b8 v[134+32*\s+8*\t:135+32*\s+8*\t], v[49+\t] offset:(\s&1)*8192+4608
  .endr
.endm
.macro LD p, i
  .if \i < 4
    buffer_load_dwordx4 v[64+32*\p+4*\i:67+32*\p+4*\i], %[vstg], s[72:75], %[nso] offen offset:1024*\i sc0 sc1
  .else
    buffer_load_dwordx4 v[64+32*\p+4*\i:67+32*\p+4*\i], %[vstg2], s[72:75], %[nso] offen offset:1024*(\i-4) sc0 sc1
  .endif
.endm
.macro LDONE p
  s_add_u32 %[ops], %[ops], 8
  s_mov_b32 s[68+\p], %[ops]
  s_add_u32 %[iss], %[iss], 1
  s_add_u32 %[nso], %[nso], %[nst]
  s_cmp_eq_u32 %[iss], %[nl]
  s_cselect_b32 %[nso], %[nso1], %[nso]
  s_cselect_b32 %[nst], 65536, %[nst]
.endm
.macro BURST p
  .irp i, 0,1,2,3,4,5,6,7
    LD \p, \i
  .endr
  LDONE \p
.endm
.macro POLL own=0
  .if \own
    buffer_load_dword v31, %[vown], %[rF], 0 offen sc0 sc1
  .else
    s_sub_u32 %[pa], %[av], %[nl]
    s_lshl_b32 %[t1], %[pa], 10
    buffer_load_dword v31, %[vflag], %[rF], %[t1] offen sc0 sc1
  .endif
.endm
.macro TOAVAIL
  s_add_u32 %[t0], %[pa], %[base]
  v_add_u32 v30, %[t0], %[vexp]
  v_cmp_ge_u32 vcc, v31, v30
  s_ff0_i32_b32 %[t0], vcc_lo
  s_lshr_b32 %[t0], %[t0], 2
  s_add_u32 %[t0], %[t0], %[pa]
  s_min_u32 %[t0], %[t0], 32
  s_add_u32 %[t0], %[t0], %[nl]
  s_max_i32 %[av], %[av], %[t0]
.endm
.macro TOOWN
  v_cmp_le_u32 vcc, %[base], v31
  s_cmp_eq_u32 vcc_lo, %[omask]
  s_cselect_b32 %[av], %[nl], 0
.endm
.macro SPIN own=0
  s_memrealtime s[64:65]
  s_mov_b32 %[lim], 0
.Lspin\@:
  s_cmp_gt_i32 %[av], %[iss]
  s_cbranch_scc1 .Lgot\@
  POLL \own
  s_waitcnt vmcnt(0)
  .if \own
    TOOWN
  .else
    TOAVAIL
  .endif
  s_add_u32 %[lim], %[lim], 1
  s_and_b32 %[t0], %[lim], 31
  s_cbranch_scc1 .Lspin\@
  v_mov_b32 v30, 0
  global_load_dword v29, v30, %[tmo] sc1
  s_memrealtime s[66:67]
  s_waitcnt vmcnt(0) lgkmcnt(0)
  v_readfirstlane_b32 %[t0], v29
  s_cmp_eq_u32 %[t0], %[call]
  s_cbranch_scc1 .Ldead\@
  s_sub_u32 %[t0], s66, s64
  s_cmp_lt_u32 %[t0], )" STR(DEADLINE_TICKS_LIT) R"(
  s_cbranch_scc1 .Lspin\@
.Ldead\@:
  v_mov_b32 v29, %[call]
  global_store_dword v30, v29, %[tmo] sc1
  v_mov_b32 %[scale], 0x7fc00000
  s_add_u32 %[av], %[nl], 32
.Lgot\@:
.endm
.macro NSTAMP
  .if STAMPN == 1
    ASTAMP 2, %[n]
  .elseif STAMPN == 2
    s_and_b32 %[t0], %[n], 7
    s_cbranch_scc1 .Lns\@
    ASTAMP 2, %[n]
.Lns\@:
  .endif
.endm
.macro MM p, s, t
  .if \s == 0
    v_mfma_f32_16x16x128_f8f6f4 v[48-16*\p+4*\t:51-16*\p+4*\t], v[128+8*\t:135+8*\t], v[64+32*\p:71+32*\p], 0
  .else
    v_mfma_f32_16x16x128_f8f6f4 v[48-16*\p+4*\t:51-16*\p+4*\t], v[128+32*\s+8*\t:135+32*\s+8*\t], v[64+32*\p+8*\s:71+32*\p+8*\s], v[48-16*\p+4*\t:51-16*\p+4*\t]
  .endif
.endm
.macro MG p, s
  .irp t, 0,1,2,3
    MM \p, \s, \t
  .endr
.endm
.macro MPROG p, base
  s_waitcnt vmcnt(\base+6)
  MG \p, 0
  s_waitcnt vmcnt(\base+4)
  MG \p, 1
  s_waitcnt vmcnt(\base+2)
  MG \p, 2
  s_waitcnt vmcnt(\base)
  POLL
  MG \p, 3
.endm
.macro TWR p, t
  .if \t == 0
    ds_write_b128 %[wr0], v[48-16*\p:51-16*\p] offset:4096*\p
  .elseif \t == 1
    ds_write_b128 %[wr1], v[52-16*\p:55-16*\p] offset:4096*\p
  .elseif \t == 2
    ds_write_b128 %[wr2], v[56-16*\p:59-16*\p] offset:4096*\p
  .else
    ds_write_b128 %[wr3], v[60-16*\p:63-16*\p] offset:4096*\p
  .endif
.endm
.macro WFLIP
  v_xor_b32 %[wr0], 8192, %[wr0]
  v_xor_b32 %[wr1], 8192, %[wr1]
  v_xor_b32 %[wr2], 8192, %[wr2]
  v_xor_b32 %[wr3], 8192, %[wr3]
.endm
.macro TADDR e
  s_and_b32 %[t1], \e, 3
  s_lshl_b32 %[t1], %[t1], 12
  v_add_u32 v28, %[t1], %[rd]
  s_lshl_b32 %[oso], \e, 17
  s_add_u32 %[oso], %[oso], %[oso1]
.endm
.macro TOWNROW e
  s_mul_i32 %[oso], \e, 7*16*8192
  s_add_u32 %[oso], %[oso], %[oso0]
  s_cmp_lt_i32 \e, 0
  s_cselect_b32 %[oso], 0x40000000, %[oso]
.endm
.macro RD r, a, v
  ds_read_b128 v[\r:\r+3], \a offset:16384*\v
.endm
.macro OP4 op, d, a, b
  .irp i, 0,1,2,3
    \op v[\d+\i], v[\a+\i], v[\b+\i]
  .endr
.endm
.macro K4 op, d, k, a
  .irp i, 0,1,2,3
    \op v[\d+\i], \k, v[\a+\i]
  .endr
.endm
.macro U4 op, d, a
  .irp i, 0,1,2,3
    \op v[\d+\i], v[\a+\i]
  .endr
.endm
.macro MED4
  .irp i, 0,1,2,3
    v_med3_f32 v[24+\i], v[20+\i], -s70, s70
  .endr
.endm
.macro TSTORE
  buffer_store_dwordx4 v[24:27], %[vout], %[rO], %[oso] offen)" OUT_BITS R"(
.endm
.macro TFIN
  K4 v_mul_f32, 20, %[scale], 20
  MED4
  K4 v_mul_f32, 24, 0xbfb8aa3b, 24
  U4 v_exp_f32, 24, 24
  K4 v_add_f32, 24, 1.0, 24
  U4 v_rcp_f32, 24, 24
  OP4 v_mul_f32, 24, 20, 24
  TSTORE
.endm
.macro T tl, x:vararg
  .if \tl
    \x
  .endif
.endm
.macro LD7 p
  LD \p, 7
  s_add_u32 %[nso], %[nso], %[nst]
  s_add_u32 %[t0], %[n], 2
  s_cmp_eq_u32 %[t0], %[nl]
  s_cselect_b32 %[nso], %[nso1], %[nso]
  s_cselect_b32 %[nst], 65536, %[nst]
.endm
.macro HOT p, q, g, l
.Lhot\p\g\l:
  NSTAMP
  T \l==1, s_waitcnt vmcnt(13)
  T \l==4, s_waitcnt vmcnt(6)
  T \l==3, s_waitcnt vmcnt(6)
  MM \p, 0, 0
  T \l==3, POLL
  T \l==3, LD \q, 0
  T \q==\g, K4 v_mul_f32, 20, %[scale], 20
  T \p==\g, s_nop 0
  TWR \q, 0
  MM \p, 0, 1
  T \l==3, LD \q, 1
  T \l==1, LD7 \q
  T \q==\g, MED4
  MM \p, 0, 2
  T \l==3, LD \q, 2
  TWR \q, 1
  T \q==\g, K4 v_mul_f32, 24, 0xbfb8aa3b, 24
  MM \p, 0, 3
  T \l==3, LD \q, 3
  T (\l==1)||(\l==2), LD \p, 0
  T \q==\g, U4 v_exp_f32, 24, 24
  T \l==1, s_waitcnt vmcnt(13)
  T \l==4, s_waitcnt vmcnt(4)
  T \l==3, s_waitcnt vmcnt(9)
  MM \p, 1, 0
  T \l==3, LD \q, 4
  TWR \q, 2
  T \q==\g, K4 v_add_f32, 24, 1.0, 24
  MM \p, 1, 1
  T \l==3, LD \q, 5
  T (\l==1)||(\l==2), LD \p, 1
  T \q==\g, U4 v_rcp_f32, 24, 24
  v_mov_b32 v30, %[n]
  MM \p, 1, 2
  T \l==3, LD \q, 6
  TWR \q, 3
  ds_write_b32 %[vpw], v30
  ds_read_b32 v29, %[vpr]
  T \q==\g, OP4 v_mul_f32, 24, 20, 24
  MM \p, 1, 3
  T \l==3, LD \q, 7
  T \l==3, LDONE \q
  T (\l==1)||(\l==2), LD \p, 2
  s_sub_u32 %[tp], %[n], 2
  T \p==\g, TADDR %[tp]
  T \p==\g, s_cmp_lt_i32 %[tp], %[nl]
  T \p==\g, s_cbranch_scc1 .Lown\p\g\l
.Lowned\p\g\l:
  T \l==1, s_waitcnt vmcnt(13)
  T \l==4, s_waitcnt vmcnt(2)
  T \l==3, s_waitcnt vmcnt(11)
  MM \p, 2, 0
  T \q==\g, TSTORE
  s_waitcnt lgkmcnt(0)
  v_cmp_lt_i32 vcc, %[tp], v29
  s_cmp_eq_u32 vcc_lo, -1
  s_cbranch_scc0 .Lwait\p\g\l
.Lwaited\p\g\l:
  T \p==\g, RD 20, v28, 0
  T \p==\g, RD 32+16*\p, v28, 1
  T \p==\g, RD 36+16*\p, v28, 2
  T \p==\g, RD 40+16*\p, v28, 3
  MM \p, 2, 1
  T (\l==1)||(\l==2), LD \p, 3
  T \p==\g, v_add_u32 v29, 0x10000, v28
  T \p==\g, RD 44+16*\p, v29, 0
  T \p==\g, RD 24, v29, 1
  MM \p, 2, 2
  .if \q
    WFLIP
  .endif
  .if (\l==1)||(\l==2)
    s_add_u32 %[t0], %[n], PTH
    s_cmp_gt_i32 %[t0], %[av]
    s_cbranch_scc1 .Lpoll\p\g\l
.Lpolled\p\g\l:
  .elseif \l != 3
    POLL
  .endif
  MM \p, 2, 3
  T (\l==1)||(\l==2), LD \p, 4
  T \p==\g, s_waitcnt lgkmcnt(2)
  T \p==\g, OP4 v_add_f32, 20, 20, 32+16*\p
  T \p==\g, RD 32+16*\p, v29, 2
  T \l==1, s_waitcnt vmcnt(13)
  T \l==4, s_waitcnt vmcnt(1+((\q==\g)&1))
  T \l==3, s_waitcnt vmcnt(9+((\q==\g)&1))
  MM \p, 3, 0
  T \p==\g, OP4 v_add_f32, 36+16*\p, 36+16*\p, 40+16*\p
  T \p==\g, RD 40+16*\p, v29, 3
  MM \p, 3, 1
  T (\l==1)||(\l==2), LD \p, 5
  T \p==\g, OP4 v_add_f32, 20, 20, 36+16*\p
  MM \p, 3, 2
  T \p==\g, s_waitcnt lgkmcnt(2)
  T \p==\g, OP4 v_add_f32, 24, 44+16*\p, 24
  T \p==\g, OP4 v_add_f32, 20, 20, 24
  s_add_u32 %[n], %[n], 1
  s_add_u32 %[t0], %[n], 2
  MM \p, 3, 3
  T (\l==1)||(\l==2), LD \p, 6
  T \p==\g, s_waitcnt lgkmcnt(0)
  T \p==\g, OP4 v_add_f32, 24, 32+16*\p, 40+16*\p
  T \p==\g, OP4 v_add_f32, 20, 20, 24
  .if (\l==1)||(\l==2)
    s_cmp_ge_i32 %[t0], %[av]
    s_cbranch_scc1 .Lexit\p\g
    T \l==2, s_branch .Lhot\q\g\()1
  .else
    T \q==\g, s_add_u32 %[ops], %[ops], 1
    s_sub_u32 %[t0], %[n], 2-((\p==\g)&1)
    s_max_i32 %[tq], %[tq], %[t0]
    T \l==3, s_branch .Ltw\q\g
    T \l!=3, s_branch .Ltop\q\g
  .endif
.endm
.macro STUBS p, g, l
  .if (\l==1)||(\l==2)
.Lpoll\p\g\l:
    POLL
    s_branch .Lpolled\p\g\l
  .endif
.Lown\p\g\l:
  TOWNROW %[tp]
  s_branch .Lowned\p\g\l
.Lwait\p\g\l:
  ds_read_b32 v29, %[vpr]
  s_waitcnt lgkmcnt(0)
  v_cmp_lt_i32 vcc, %[tp], v29
  s_cmp_eq_u32 vcc_lo, -1
  s_cbranch_scc0 .Lwait\p\g\l
  s_branch .Lwaited\p\g\l
.endm
.macro COLD p, q, g
  STUBS \p, \g, 0
  STUBS \p, \g, 1
  STUBS \p, \g, 2
  STUBS \p, \g, 3
  STUBS \p, \g, 4
.Lexit\p\g:
  TOAVAIL
  s_add_u32 %[iss], %[n], 2
  s_cmp_lt_i32 %[iss], %[av]
  s_cbranch_scc1 .Lhot\q\g\()1
  LD7 \p
  s_mov_b32 s[68+\p], %[ops]
  s_sub_u32 s[69-\p], %[ops], 8
  s_sub_u32 %[t0], %[n], 2-((\p==\g)&1)
  s_max_i32 %[tq], %[tq], %[t0]
  s_waitcnt vmcnt(8)
  s_branch .Lhot\q\g\()0
.Ltop\p\g:
  .if TOPWAIT
    s_add_u32 %[t0], %[n], 2
    s_cmp_eq_u32 %[iss], %[t0]
    s_cbranch_scc1 .Ltw\p\g
    s_waitcnt vmcnt(0)
.Ltw\p\g:
  .endif
  TOAVAIL
  s_cmp_eq_u32 %[iss], %[n]
  s_cbranch_scc1 .Lcold\p\g
  s_add_u32 %[t0], %[n], 1
  s_cmp_lg_u32 %[iss], %[t0]
  s_cbranch_scc1 .Lt2\p\g
  s_cmp_ge_i32 %[iss], %[av]
  s_cbranch_scc1 .Lhot\p\g\()4
  s_add_u32 %[t0], %[n], NEARTH
  s_cmp_le_i32 %[av], %[t0]
  s_cbranch_scc1 .Lhot\p\g\()3
  BURST \q
.Lt2\p\g:
  s_waitcnt vmcnt(8)
  s_cmp_lt_i32 %[iss], %[av]
  s_cbranch_scc1 .Lhot\p\g\()2
  s_branch .Lhot\p\g\()0
.Lcold\p\g:
  s_cmp_ge_i32 %[iss], %[av]
  s_cbranch_scc1 .Lc1\p\g
  BURST \p
.Lc1\p\g:
  v_mov_b32 v30, %[n]
  TWR \q, 0
  TWR \q, 1
  TWR \q, 2
  TWR \q, 3
  ds_write_b32 %[vpw], v30
  .if \q==\g
    TFIN
    s_add_u32 %[ops], %[ops], 1
  .endif
  s_cmp_gt_u32 %[n], %[last]
  s_cbranch_scc1 .Lc3\p\g
  s_cmp_lg_u32 %[iss], %[n]
  s_cbranch_scc1 .Lc2\p\g
  SPIN
  BURST \p
.Lc2\p\g:
  s_cmp_ge_i32 %[iss], %[av]
  s_cbranch_scc1 .Lc3\p\g
  BURST \q
.Lc3\p\g:
  s_cmp_ge_i32 %[tq], %[n]
  s_cbranch_scc1 .Lc5\p\g
  TADDR %[tq]
  s_cmp_ge_i32 %[tq], %[nl]
  s_cbranch_scc1 .Lc4\p\g
  TOWNROW %[tq]
.Lc4\p\g:
  ds_read_b32 v29, %[vpr]
  s_waitcnt lgkmcnt(0)
  v_cmp_lt_i32 vcc, %[tq], v29
  s_cmp_eq_u32 vcc_lo, -1
  s_cbranch_scc0 .Lc4\p\g
  RD 20, v28, 0
  RD 24, v28, 1
  RD 32+16*\q, v28, 2
  RD 36+16*\q, v28, 3
  v_add_u32 v29, 0x10000, v28
  RD 40+16*\q, v29, 0
  RD 44+16*\q, v29, 1
  s_waitcnt lgkmcnt(4)
  OP4 v_add_f32, 20, 20, 24
  RD 24, v29, 2
  s_waitcnt lgkmcnt(3)
  OP4 v_add_f32, 32+16*\q, 32+16*\q, 36+16*\q
  RD 36+16*\q, v29, 3
  OP4 v_add_f32, 20, 20, 32+16*\q
  s_waitcnt lgkmcnt(2)
  OP4 v_add_f32, 40+16*\q, 40+16*\q, 44+16*\q
  OP4 v_add_f32, 20, 20, 40+16*\q
  s_waitcnt lgkmcnt(0)
  OP4 v_add_f32, 24, 24, 36+16*\q
  OP4 v_add_f32, 20, 20, 24
  TFIN
  s_add_u32 %[ops], %[ops], 1
  s_add_u32 %[tq], %[tq], 2
  s_branch .Lc3\p\g
.Lc5\p\g:
  s_cmp_gt_u32 %[n], %[last]
  s_cbranch_scc1 .Ldone
  .if \q
    WFLIP
  .endif
  s_mov_b32 %[oso], 0x40000000
  NSTAMP
  s_sub_u32 %[t0], %[ops], s[68+\p]
  s_add_u32 %[n], %[n], 1
  s_cmp_ge_u32 %[t0], 8
  s_cbranch_scc1 .Lc8\p\g
  s_cmp_ge_u32 %[t0], 2
  s_cbranch_scc1 .Lc7\p\g
  s_cmp_ge_u32 %[t0], 1
  s_cbranch_scc1 .Lc9\p\g
  MPROG \p, 0
  s_branch .Ltop\q\g
.Lc9\p\g:
  MPROG \p, 1
  s_branch .Ltop\q\g
.Lc7\p\g:
  MPROG \p, 2
  s_branch .Ltop\q\g
.Lc8\p\g:
  MPROG \p, 8
  s_branch .Ltop\q\g
.endm
.macro MAIN g
  HOT 0, 1, \g, 1
  HOT 1, 0, \g, 1
  s_branch .Lhot0\g\()1
  HOT 0, 1, \g, 2
  HOT 1, 0, \g, 2
  HOT 0, 1, \g, 0
  HOT 1, 0, \g, 0
  HOT 0, 1, \g, 3
  HOT 1, 0, \g, 3
  HOT 0, 1, \g, 4
  HOT 1, 0, \g, 4
  COLD 0, 1, \g
  COLD 1, 0, \g
.endm

  v_mov_b32 v30, 0
  ds_write_b32 %[vpw], v30
  s_waitcnt lgkmcnt(0)
  s_load_dword %[call], %[cnt], 0
  s_load_dword %[t1], %[psx], 0
  s_load_dword %[t2], %[psw], 0
  s_barrier
  v_xor_b32 v48, 32, %[wv0]
  s_cmp_eq_u32 %[cp], 0
  s_cbranch_scc1 .Lnocopy
  buffer_load_dwordx4 v[64:67], %[vcs], %[rX], 0 offen
  buffer_load_dwordx4 v[68:71], %[vcs], %[rX], 0 offen offset:64
.Lnocopy:
  WPF 0
  WPF 1
  s_cmp_eq_u32 %[cp], 0
  s_cbranch_scc0 .Lcopy1
  WDMA 0
  WPF 2
  WDMA 1
  WPF 3
.Lcopy1:
  v_mov_b32 v49, %[tr0]
  v_xor_b32 v50, 16, %[tr0]
  v_xor_b32 v51, 32, %[tr0]
  v_xor_b32 v52, 48, %[tr0]
  v_add_u32 %[wr0], 8192, %[wr0]
  s_mov_b32 %[oso], 0x40000000
  v_xor_b32 %[wr1], 64, %[wr0]
  v_xor_b32 %[wr2], 128, %[wr0]
  v_xor_b32 %[wr3], 192, %[wr0]
  v_add_u32 %[vstg2], 4096, %[vstg]
  s_mov_b32 s70, 0x42700000
  s_mov_b32 %[n], 0
  s_mov_b32 %[iss], 0
  s_mov_b32 %[ops], 0
  s_mov_b32 %[nso], %[nso0]
  s_mov_b32 %[nst], 7*65536
  s_mov_b32 %[tq], %[grp]
  s_mov_b32 %[pa], 0
  s_waitcnt lgkmcnt(0)
  s_add_u32 %[call], %[call], 1
  s_lshl_b32 %[base], %[call], 6
  v_mov_b32 %[scale], %[t1]
  v_mul_f32 %[scale], %[t2], %[scale]
  s_and_b32 %[t0], %[call], 1
  s_lshl_b32 %[t0], %[t0], 24
  s_mov_b64 s[72:73], %[aS]
  s_add_u32 s72, s72, %[t0]
  s_addc_u32 s73, s73, 0
  s_mov_b32 s74, 0x1000000
  s_mov_b32 s75, 0x20000
  s_cmp_eq_u32 %[cp], 0
  s_cbranch_scc1 .Lcopied
  v_mov_b32 v72, %[base]
  WPF 2
  WPF 3
  s_waitcnt vmcnt(4*NPF)
  buffer_store_dwordx4 v[64:67], %[vcd], s[72:75], 0 offen sc0 sc1
  buffer_store_dwordx4 v[68:71], %[vcd], s[72:75], 0 offen offset:1024 sc0 sc1
  s_waitcnt vmcnt(0)
  buffer_store_dword v72, %[vcf], %[rF], 0 offen sc0 sc1
  WDMA 0
  WDMA 1
.Lcopied:
  ASTAMP 42, 0
  s_waitcnt vmcnt(8)
  WTR 0
  ASTAMP 41, 0
  s_waitcnt lgkmcnt(0)
  WDMA 2
  s_waitcnt vmcnt(8)
  WTR 1
  s_waitcnt lgkmcnt(0)
  WDMA 3
  POLL 1
  s_waitcnt vmcnt(9)
  WTR 2
  s_waitcnt vmcnt(0)
  WTR 3
  ASTAMP 43, 0
  TOOWN
)" FAKE_AV R"(
  SPIN 1
  ASTAMP 44, 0
  v_mov_b32 v31, 0
  BURST 0
  BURST 1
  s_waitcnt lgkmcnt(0)
  ASTAMP 1, 0
  CSTAMP 48
  s_cmp_eq_u32 %[tq], 0
  s_cbranch_scc1 .Ltop00
  s_branch .Ltop01
  MAIN 0
  MAIN 1
.Ldone:
  CSTAMP 49
)"

__global__ __launch_bounds__(THREADS, 2) void ag_gemm_silu(Params) {
  extern __shared__ __attribute__((aligned(16))) unsigned char smem[];
  const int b = blockIdx.x, lane = threadIdx.x & 63;
  const int wv = (int)rfl(threadIdx.x >> 6);
  const int si = SPERM ? (b & 7) * (NSEND / 8) + (b >> 3) : b, u0 = (si * 7 + wv) * 2;
  const unsigned li = ((unsigned)u0 >> 6) & (NR - 1), un = (unsigned)u0 & 63;
  u32x16 ka;
  unsigned long long pw_;
  asm volatile("s_load_dwordx16 %0, %2, 0\n\ts_load_dwordx2 %1, %2, %3\n\ts_waitcnt lgkmcnt(0)"
               : "=&s"(ka), "=&s"(pw_) : "s"(__builtin_amdgcn_kernarg_segment_ptr()), "s"(64 + li * 8));
  Params p;
  p.me = (unsigned char*)mk64(ka[0], ka[1]); p.x = (const unsigned char*)mk64(ka[2], ka[3]); p.w = (const unsigned char*)mk64(ka[4], ka[5]);
  p.sx = (const float*)mk64(ka[6], ka[7]); p.sw = (const float*)mk64(ka[8], ka[9]); p.out = (float*)mk64(ka[10], ka[11]);
  const unsigned rank = ka[12] & (NR - 1);
  unsigned char* me = p.me;
  unsigned* cnt = (unsigned*)(me + CNT_OFF + (size_t)b * 128);
  unsigned* tmo = (unsigned*)(me + TMO_OFF);

  if (__builtin_expect(b < NSEND, 1)) {
    if (wv == 7) return;
    const unsigned svo = (lane & 15) * KD + (lane >> 4) * 16 + un * 64, dvo = lane * 16 + un * 1024;
    const int first = max(si * 14, (int)li * 64), last = min((si + 1) * 14, (int)(li + 1) * 64), ks = un / 2 + lane;
    const unsigned fo = (u0 == first && lane < (last - first) / 2) ? (ks >> 2) * 128 + (ks & 3) * 4 : 0x80000000u;
    const u32x4 rX = mk_srd(p.x, SHARD), rF = mk_srd((void*)(pw_ + FLAG_OFF + rank * FLAG_STR), FLAG_STR);
    unsigned call, so, fs, t0;
    asm volatile(R"(
.macro SLD r, d
  s_add_u32 %[t0], %[so], \d*65536
  buffer_load_dwordx4 v[32+8*\r:35+8*\r], %[svo], %[rX], %[t0] offen
  buffer_load_dwordx4 v[36+8*\r:39+8*\r], %[svo], %[rX], %[t0] offen offset:64
.endm
.macro SST r
  buffer_store_dwordx4 v[32+8*\r:35+8*\r], %[dvo], s[64:67], %[so] offen sc0 sc1
  buffer_store_dwordx4 v[36+8*\r:39+8*\r], %[dvo], s[64:67], %[so] offen offset:1024 sc0 sc1
.endm
.macro SPUB w
  s_waitcnt vmcnt(\w)
  s_barrier
  buffer_store_dword v64, %[fo], %[rF], %[fs] offen sc0 sc1
  s_add_u32 %[fs], %[fs], 1024
  s_add_u32 %[so], %[so], 65536
  v_add_u32 v64, 1, v64
.endm
  s_load_dword %[call], %[cnt], 0
  s_mov_b32 %[so], 0
  s_mov_b32 %[fs], 0
  SLD 0, 0
  SLD 1, 1
  s_mov_b64 s[64:65], %[aD]
  s_mov_b32 s66, 0x200000
  s_mov_b32 s67, 0x20000
  s_waitcnt lgkmcnt(0)
  s_add_u32 %[call], %[call], 1
  s_and_b32 %[t0], %[call], 1
  s_lshl_b32 %[t0], %[t0], 24
  s_add_u32 s64, s64, %[t0]
  s_addc_u32 s65, s65, 0
  s_lshl_b32 %[t0], %[call], 6
  v_mov_b32 v64, %[t0]
  v_add_u32 v64, 1, v64
  s_waitcnt vmcnt(2)
  SST 0
  SLD 2, 2
  SLD 3, 3
  s_add_u32 %[so], %[so], 65536
  s_waitcnt vmcnt(6)
  SST 1
  SLD 0, 3
  SPUB 8
  s_waitcnt vmcnt(7)
.Lsend:
  SST 2
  SLD 1, 3
  SPUB 7
  SST 3
  SLD 2, 3
  SPUB 7
  s_cmp_eq_u32 %[fs], 31*1024
  s_cbranch_scc1 .Lsent
  SST 0
  SLD 3, 3
  SPUB 7
  SST 1
  SLD 0, 3
  SPUB 7
  s_branch .Lsend
.Lsent:
  SPUB 0
)"      : [call] "=&s"(call), [so] "=&s"(so), [fs] "=&s"(fs), [t0] "=&s"(t0)
        : [rX] "s"(rX), [rF] "s"(rF), [cnt] "s"(cnt), [aD] "s"((pw_ + (size_t)rank * SHARD) & 0xffffffffffffull),
          [svo] "v"(svo), [dvo] "v"(dvo), [fo] "v"(fo)
        : "memory", "scc", "s64", "s65", "s66", "s67", "v32", "v33", "v34", "v35", "v36", "v37", "v38", "v39", CLOB10(4), CLOB10(5),
          "v60", "v61", "v62", "v63", "v64");
    if (threadIdx.x == 0) __hip_atomic_store(cnt, call, __ATOMIC_RELAXED, __HIP_MEMORY_SCOPE_AGENT);
    STAMP(2);
    return;
  }

  const int xcd = b & 7, slot = (b >> 3) - NSEND / 8;
  int cg, rg;
  { const int id = (xcd >> 1) * 28 + slot; cg = (xcd & 1) * 16 + (id & 15); rg = id >> 4; }
  const unsigned lds0 = (unsigned)(unsigned long long)(LAS unsigned char*)smem;
  const unsigned src = (rank + 1 + rg) & (NR - 1);
  const int q = lane >> 4, m = lane & 15;
  unsigned wvo0 = (lane >> 2) * NL + cg * 64 + (((lane & 3) ^ (q & 1)) * 16);
  unsigned tr0 = lds0 + wv * 16384 + (16 * q + (m >> 1)) * 64 + ((((m >> 3) & 1) | ((q & 1) << 1)) * 16) + 8 * (lane & 1);
  const unsigned vstg = lane * 16 + wv * 8192;
  const int nl = (rg < 4) ? 5 : 4;
  const unsigned vflag = lane < 4 * NPROBE ? src * FLAG_STR + (lane >> 2) * 1024 + wv * 128 + (lane & 3) * 4 : 0x80000000u;
  const unsigned vexp = (lane >> 2) + 1;
  unsigned vown = lane < 4 * nl ? rank * FLAG_STR + (rg + 7 * (lane >> 2)) * 1024 + wv * 128 + (lane & 3) * 4 : 0x80000000u;
  const unsigned item = wv * (GRID - NSEND) + (unsigned)(b - NSEND);
  unsigned vcs = (item >> 5) * BLKB + m * KD + (item & 31) * 128 + q * 16;
  unsigned vcd = rank * SHARD + item * 2048 + lane * 16;
  unsigned vcf = lane == 0 ? rank * FLAG_STR + (item >> 2) * 128 + (item & 3) * 4 : 0x80000000u;
  constexpr int LPR = MAPMODE == 0 ? 2 : MAPMODE == 3 ? 4 : 8, PL = (1024 * LPR + 223) / 224;
  const unsigned pfl = (slot * 8 + wv) * PL + lane, pfr = pfl / LPR;
  unsigned vpf = (lane < PL && pfr < 1024) ? ((pfr >> 7) * 512 + (pfr & 127)) * NL + ((cg >> 1) / LPR * LPR + pfl % LPR) * 128 : 0x80000000u;
  unsigned wr0 = lds0 + wv * 16384 + m * 256 + ((q ^ m) * 16);
  const int m_rd = 4 * (wv & 3) + q;
  const unsigned rd = lds0 + m_rd * 256 + ((m ^ m_rd) * 16);
  const unsigned vout = m_rd * (NL * 4) + cg * 256 + m * 16;
  const unsigned long long aS = (unsigned long long)me & 0xffffffffffffull;
  const u32x4 rW = mk_srd(p.w, KD * NL), rX = mk_srd(p.x, SHARD);
  const u32x4 rF = mk_srd(me + FLAG_OFF, (NR + 1) * FLAG_STR), rO = mk_srd(p.out, (unsigned)NR * MS * NL * 4u);
  unsigned call, base, n, iss, av, lim, ops, nso, nst, oso, tp, tq, pa, t0, t1, t2, t3;
  unsigned vstg2, wr1, wr2, wr3, scale;
  asm volatile(COMPUTE_ASM
      : [call] "=&s"(call), [base] "=&s"(base), [n] "=&s"(n), [iss] "=&s"(iss), [av] "=&s"(av), [lim] "=&s"(lim), [ops] "=&s"(ops),
        [nso] "=&s"(nso), [nst] "=&s"(nst), [oso] "=&s"(oso), [pa] "=&s"(pa), [tp] "=&s"(tp), [tq] "=&s"(tq), [t0] "=&s"(t0), [t1] "=&s"(t1), [t2] "=&s"(t2), [t3] "=&s"(t3),
        [wr0] "+v"(wr0), [vown] "+{v80}"(vown), [vcf] "+{v78}"(vcf), [vcd] "+{v77}"(vcd), [vcs] "+{v76}"(vcs), [tr0] "+{v75}"(tr0), [wv0] "+{v74}"(wvo0), [vpf] "+{v79}"(vpf), [vstg2] "=&v"(vstg2), [wr1] "=&v"(wr1), [wr2] "=&v"(wr2), [wr3] "=&v"(wr3), [scale] "=&v"(scale)
      : [rW] "s"(rW), [rX] "s"(rX), [rF] "s"(rF), [rO] "s"(rO), [lb] "s"(rfl(lds0 + wv * 16384)), [sw] "s"(rfl(wv * 512 * NL)),
        [nl] "s"(nl), [grp] "s"(rfl(wv >> 2)), [omask] "s"((1u << (4 * nl)) - 1), [last] "s"(nl + NBLK - 1),
        [nso0] "s"(rank * SHARD + rg * BLKB), [nso1] "s"(src * SHARD), [oso0] "s"((rank * MS + rg * 16) * (NL * 4)), [oso1] "s"((src * MS - nl * 16) * (NL * 4)),
        [aS] "s"(aS), [cw] "s"(rfl(slot * 8 + (7 - wv))), [cp] "s"(rfl(1024 - (int)item > 0 ? 1024 - item : 0)), [cnt] "s"(cnt), [tmo] "s"(tmo), [psx] "s"(p.sx), [psw] "s"(p.sw),
        [vstg] "v"(vstg), [vflag] "v"(vflag), [vexp] "v"(vexp), [rd] "v"(rd), [vpw] "v"(lds0 + PROG_OFF + wv * 4), [vpr] "v"(lds0 + PROG_OFF + (lane & 7) * 4), [vout] "v"(vout)
      : "memory", "scc", "vcc", "s64", "s65", "s66", "s67", "s68", "s69", "s70", "s71", "s72", "s73", "s74", "s75", CLOB_V);
  if (threadIdx.x == 0) __hip_atomic_store(cnt, call, __ATOMIC_RELAXED, __HIP_MEMORY_SCOPE_AGENT);
}

extern "C" void kernel_launch(int rank, int nranks, ncclComm_t comm, hipStream_t stream,
                              void* const* d_in, const int* in_sizes, int n_in,
                              void* d_out, int out_size,
                              const int* in_dtypes, int out_dtype,
                              void* const* d_out_all,
                              void* const* d_ws_all, size_t ws_bytes) {
  if (nranks != NR || rank < 0 || rank >= NR || n_in != 4 || ws_bytes < WS_NEED ||
      in_sizes[0] != (int)SHARD || in_sizes[1] != KD * NL || out_size != NR * MS * NL) {
    fprintf(stderr, "ag_gemm_silu: unexpected problem shape\n");
    return;
  }
  HIP_CHECK(hipFuncSetAttribute((const void*)ag_gemm_silu, hipFuncAttributeMaxDynamicSharedMemorySize, LDS_BYTES));
  Params p;
  p.me = (unsigned char*)d_ws_all[rank];
  for (int r = 0; r < NR; r++) p.peer[r] = (unsigned char*)d_ws_all[(rank + 1 + r) % NR];
  p.x = (const unsigned char*)d_in[0];
  p.w = (const unsigned char*)d_in[1];
  p.sx = (const float*)d_in[2];
  p.sw = (const float*)d_in[3];
  p.out = (float*)d_out;
  p.rank = rank;
  p.pad[0] = p.pad[1] = p.pad[2] = 0;
  ag_gemm_silu<<<GRID, THREADS, LDS_BYTES, stream>>>(p);
  HIP_CHECK(hipGetLastError());
}
```

```cpp
#include <hip/hip_runtime.h>
#include <rccl/rccl.h>
#include <cstdio>

#define HIP_CHECK(expr) do { hipError_t _e = (expr); if (_e != hipSuccess) { \
    fprintf(stderr, "HIP error %s at %s:%d\n", hipGetErrorString(_e), __FILE__, __LINE__); \
    return; } } while (0)

typedef int      v8i   __attribute__((ext_vector_type(8)));
typedef int      v2i   __attribute__((ext_vector_type(2)));
typedef unsigned u32x4 __attribute__((ext_vector_type(4)));
typedef float    v4f   __attribute__((ext_vector_type(4)));
#define LAS __attribute__((address_space(3)))
#define CAS __attribute__((address_space(4)))
#define DI __device__ __forceinline__

constexpr int      NR        = 8;
constexpr int      KD        = 4096, NL = 2048, MS = 512;
constexpr unsigned SHARD     = MS * KD;
constexpr unsigned BLKB      = 16 * KD;
constexpr int      NBLK      = MS / 16;
constexpr size_t   STAGE_B   = (size_t)NR * SHARD;
constexpr size_t   FLAG_OFF  = 2 * STAGE_B;
constexpr unsigned FLAG_STR  = NBLK * 1024;
constexpr size_t   CNT_OFF   = FLAG_OFF + (NR + 1) * FLAG_STR;
constexpr size_t   TMO_OFF   = CNT_OFF + 256 * 128;
constexpr size_t   DIAG_OFF  = TMO_OFF + 4096;
constexpr size_t   WS_NEED   = DIAG_OFF + 256 * 8 * 64 * 8;
constexpr int      NSEND     = 32, GRID = 256, THREADS = 512;
constexpr unsigned PROG_OFF  = 131072;
constexpr unsigned LDS_BYTES = PROG_OFF + 64;
#define DEADLINE_TICKS_LIT 100000000
constexpr int      SRD3      = 0x00020000;

#define PACE 0
#define PACE0 0
#define SD 2
#define WIN 2
#define OUT_AUX 17
#define MAPMODE 0

struct Params {
  unsigned char* me;
  const unsigned char* x;
  const unsigned char* w;
  const float* sx;
  const float* sw;
  float* out;
  int rank;
  int pad[3];
  unsigned char* peer[NR];
};
typedef unsigned u32x16 __attribute__((ext_vector_type(16)));
DI unsigned long long mk64(unsigned lo, unsigned hi) { return ((unsigned long long)hi << 32) | lo; }

DI unsigned rfl(unsigned v) { return __builtin_amdgcn_readfirstlane(v); }
DI __amdgpu_buffer_rsrc_t mk_rsrc(const void* p, unsigned bytes) {
  unsigned long long a = (unsigned long long)p;
  unsigned lo = rfl((unsigned)a), hi = rfl((unsigned)(a >> 32));
  return __builtin_amdgcn_make_buffer_rsrc((void*)(((unsigned long long)hi << 32) | lo), (short)0, (int)bytes, SRD3);
}

DI u32x4 mk_srd(const void* p, unsigned bytes) {
  const unsigned long long a = (unsigned long long)p;
  return u32x4{rfl((unsigned)a), rfl((unsigned)(a >> 32) & 0xffffu), bytes, (unsigned)SRD3};
}

#define STAMP(slot) do {} while (0)

#define VMCNT(n)  asm volatile("s_waitcnt vmcnt(" #n ")" ::: "memory")
#define LGKM(n)   asm volatile("s_waitcnt lgkmcnt(" #n ")" ::: "memory")
#define LGKM0()   LGKM(0)
#define BARRIER() do { __builtin_amdgcn_s_barrier(); asm volatile("" ::: "memory"); } while (0)
#define SCHED()   __builtin_amdgcn_sched_barrier(0)

#define CLOB10(p) "v" #p "0", "v" #p "1", "v" #p "2", "v" #p "3", "v" #p "4", "v" #p "5", "v" #p "6", "v" #p "7", "v" #p "8", "v" #p "9"
#define CLOB_V "v20", "v21", "v22", "v23", "v24", "v25", "v26", "v27", "v28", "v29", CLOB10(3), CLOB10(4), CLOB10(5), CLOB10(6), "v70", "v71", "v72", "v73", "v81", "v82", "v83", "v84", "v85", "v86", "v87", "v88", "v89", CLOB10(9), CLOB10(10), CLOB10(11), CLOB10(12),   \
    CLOB10(13), CLOB10(14), CLOB10(15), CLOB10(16), CLOB10(17), CLOB10(18), CLOB10(19), CLOB10(20), CLOB10(21), CLOB10(22), CLOB10(23),        \
    CLOB10(24), "v250", "v251", "v252", "v253", "v254", "v255"
#define STR_(x) #x
#define STR(x) STR_(x)
#define OUT_BITS " sc0 sc1"
#define ASTAMP ".macro ASTAMP off, idx\n.endm\n.macro CSTAMP off\n.endm\n"
#define FAKE_AV ""
#define NPF 1
#define NEARTH 2
#define SPERM 1
#define TOPWAIT 1
#define SSLEEP 0
#define CODEPF 1
#define TLBN 0
#define TLBW ""
#define NPROBE 4
#define PTH 5
#define STAMPN 1
#define COMPUTE_ASM ".set STAMPN, " STR(STAMPN) "\n.set NEARTH, " STR(NEARTH) "\n.set TOPWAIT, " STR(TOPWAIT) "\n.set SSLEEP, " STR(SSLEEP) "\n.set CODEPF, " STR(CODEPF) "\n.set PTH, " STR(PTH) "\n.set NPF, " STR(NPF) "\n" ASTAMP R"(
.macro WDMA s
  .irp j, 0,1,2,3,4,5,6,7
    s_add_u32 m0, %[lb], ((\s)&1)*8192+\j*1024
    s_add_u32 %[t0], %[sw], ((\s)*128+\j*16)*2048
    .if \j & 1
      buffer_load_dwordx4 v48, %[rW], %[t0] offen lds
    .else
      buffer_load_dwordx4 %[wv0], %[rW], %[t0] offen lds
    .endif
  .endr
.endm
.macro WPF s
  .if NPF
    s_mov_b32 %[t0], \s*128*2048
    buffer_load_dword v73, %[vpf], %[rW], %[t0] offen
  .endif
.endm
.macro WTR s
  .irp t, 0,1,2,3
    ds_read_b64_tr_b8 v[128+32*\s+8*\t:129+32*\s+8*\t], v[49+\t] offset:(\s&1)*8192
    ds_read_b64_tr_b8 v[130+32*\s+8*\t:131+32*\s+8*\t], v[49+\t] offset:(\s&1)*8192+512
    ds_read_b64_tr_b8 v[132+32*\s+8*\t:133+32*\s+8*\t], v[49+\t] offset:(\s&1)*8192+4096
    ds_read_b64_tr_b8 v[134+32*\s+8*\t:135+32*\s+8*\t], v[49+\t] offset:(\s&1)*8192+4608
  .endr
.endm
.macro LD p, i
  .if \i < 4
    buffer_load_dwordx4 v[64+32*\p+4*\i:67+32*\p+4*\i], %[vstg], s[72:75], %[nso] offen offset:1024*\i sc0 sc1
  .else
    buffer_load_dwordx4 v[64+32*\p+4*\i:67+32*\p+4*\i], %[vstg2], s[72:75], %[nso] offen offset:1024*(\i-4) sc0 sc1
  .endif
.endm
.macro LDONE p
  s_add_u32 %[ops], %[ops], 8
  s_mov_b32 s[68+\p], %[ops]
  s_add_u32 %[iss], %[iss], 1
  s_add_u32 %[nso], %[nso], %[nst]
  s_cmp_eq_u32 %[iss], %[nl]
  s_cselect_b32 %[nso], %[nso1], %[nso]
  s_cselect_b32 %[nst], 65536, %[nst]
.endm
.macro BURST p
  .irp i, 0,1,2,3,4,5,6,7
    LD \p, \i
  .endr
  LDONE \p
.endm
.macro POLL own=0
  .if \own
    buffer_load_dword v31, %[vown], %[rF], 0 offen sc0 sc1
  .else
    s_sub_u32 %[pa], %[av], %[nl]
    s_lshl_b32 %[t1], %[pa], 10
    buffer_load_dword v31, %[vflag], %[rF], %[t1] offen sc0 sc1
  .endif
.endm
.macro TOAVAIL
  s_add_u32 %[t0], %[pa], %[base]
  v_add_u32 v30, %[t0], %[vexp]
  v_cmp_ge_u32 vcc, v31, v30
  s_ff0_i32_b32 %[t0], vcc_lo
  s_lshr_b32 %[t0], %[t0], 2
  s_add_u32 %[t0], %[t0], %[pa]
  s_min_u32 %[t0], %[t0], 32
  s_add_u32 %[t0], %[t0], %[nl]
  s_max_i32 %[av], %[av], %[t0]
.endm
.macro TOOWN
  v_cmp_le_u32 vcc, %[base], v31
  s_cmp_eq_u32 vcc_lo, %[omask]
  s_cselect_b32 %[av], %[nl], 0
.endm
.macro SPIN own=0
  s_memrealtime s[64:65]
  s_mov_b32 %[lim], 0
.Lspin\@:
  s_cmp_gt_i32 %[av], %[iss]
  s_cbranch_scc1 .Lgot\@
  POLL \own
  s_waitcnt vmcnt(0)
  .if \own
    TOOWN
  .else
    TOAVAIL
  .endif
  s_add_u32 %[lim], %[lim], 1
  s_and_b32 %[t0], %[lim], 31
  s_cbranch_scc1 .Lspin\@
  v_mov_b32 v30, 0
  global_load_dword v29, v30, %[tmo] sc1
  s_memrealtime s[66:67]
  s_waitcnt vmcnt(0) lgkmcnt(0)
  v_readfirstlane_b32 %[t0], v29
  s_cmp_eq_u32 %[t0], %[call]
  s_cbranch_scc1 .Ldead\@
  s_sub_u32 %[t0], s66, s64
  s_cmp_lt_u32 %[t0], )" STR(DEADLINE_TICKS_LIT) R"(
  s_cbranch_scc1 .Lspin\@
.Ldead\@:
  v_mov_b32 v29, %[call]
  global_store_dword v30, v29, %[tmo] sc1
  v_mov_b32 %[scale], 0x7fc00000
  s_add_u32 %[av], %[nl], 32
.Lgot\@:
.endm
.macro NSTAMP
  .if STAMPN == 1
    ASTAMP 2, %[n]
  .elseif STAMPN == 2
    s_and_b32 %[t0], %[n], 7
    s_cbranch_scc1 .Lns\@
    ASTAMP 2, %[n]
.Lns\@:
  .endif
.endm
.macro MM p, s, t
  .if \s == 0
    v_mfma_f32_16x16x128_f8f6f4 v[48-16*\p+4*\t:51-16*\p+4*\t], v[128+8*\t:135+8*\t], v[64+32*\p:71+32*\p], 0
  .else
    v_mfma_f32_16x16x128_f8f6f4 v[48-16*\p+4*\t:51-16*\p+4*\t], v[128+32*\s+8*\t:135+32*\s+8*\t], v[64+32*\p+8*\s:71+32*\p+8*\s], v[48-16*\p+4*\t:51-16*\p+4*\t]
  .endif
.endm
.macro MG p, s
  .irp t, 0,1,2,3
    MM \p, \s, \t
  .endr
.endm
.macro MPROG p, base
  s_waitcnt vmcnt(\base+6)
  MG \p, 0
  s_waitcnt vmcnt(\base+4)
  MG \p, 1
  s_waitcnt vmcnt(\base+2)
  MG \p, 2
  s_waitcnt vmcnt(\base)
  POLL
  MG \p, 3
.endm
.macro TWR p, t
  .if \t == 0
    ds_write_b128 %[wr0], v[48-16*\p:51-16*\p] offset:4096*\p
  .elseif \t == 1
    ds_write_b128 %[wr1], v[52-16*\p:55-16*\p] offset:4096*\p
  .elseif \t == 2
    ds_write_b128 %[wr2], v[56-16*\p:59-16*\p] offset:4096*\p
  .else
    ds_write_b128 %[wr3], v[60-16*\p:63-16*\p] offset:4096*\p
  .endif
.endm
.macro WFLIP
  v_xor_b32 %[wr0], 8192, %[wr0]
  v_xor_b32 %[wr1], 8192, %[wr1]
  v_xor_b32 %[wr2], 8192, %[wr2]
  v_xor_b32 %[wr3], 8192, %[wr3]
.endm
.macro TADDR e
  s_and_b32 %[t1], \e, 3
  s_lshl_b32 %[t1], %[t1], 12
  v_add_u32 v28, %[t1], %[rd]
  s_lshl_b32 %[oso], \e, 17
  s_add_u32 %[oso], %[oso], %[oso1]
.endm
.macro TOWNROW e
  s_mul_i32 %[oso], \e, 7*16*8192
  s_add_u32 %[oso], %[oso], %[oso0]
  s_cmp_lt_i32 \e, 0
  s_cselect_b32 %[oso], 0x40000000, %[oso]
.endm
.macro RD r, a, v
  ds_read_b128 v[\r:\r+3], \a offset:16384*\v
.endm
.macro OP4 op, d, a, b
  .irp i, 0,1,2,3
    \op v[\d+\i], v[\a+\i], v[\b+\i]
  .endr
.endm
.macro K4 op, d, k, a
  .irp i, 0,1,2,3
    \op v[\d+\i], \k, v[\a+\i]
  .endr
.endm
.macro U4 op, d, a
  .irp i, 0,1,2,3
    \op v[\d+\i], v[\a+\i]
  .endr
.endm
.macro MED4
  .irp i, 0,1,2,3
    v_med3_f32 v[24+\i], v[20+\i], -s70, s70
  .endr
.endm
.macro TSTORE
  buffer_store_dwordx4 v[24:27], %[vout], %[rO], %[oso] offen)" OUT_BITS R"(
.endm
.macro TFIN
  K4 v_mul_f32, 20, %[scale], 20
  MED4
  K4 v_mul_f32, 24, 0xbfb8aa3b, 24
  U4 v_exp_f32, 24, 24
  K4 v_add_f32, 24, 1.0, 24
  U4 v_rcp_f32, 24, 24
  OP4 v_mul_f32, 24, 20, 24
  TSTORE
.endm
.macro T tl, x:vararg
  .if \tl
    \x
  .endif
.endm
.macro LD7 p
  LD \p, 7
  s_add_u32 %[nso], %[nso], %[nst]
  s_add_u32 %[t0], %[n], 2
  s_cmp_eq_u32 %[t0], %[nl]
  s_cselect_b32 %[nso], %[nso1], %[nso]
  s_cselect_b32 %[nst], 65536, %[nst]
.endm
.macro HOT p, q, g, l
.Lhot\p\g\l:
  NSTAMP
  T \l==1, s_waitcnt vmcnt(13)
  T \l==4, s_waitcnt vmcnt(6)
  T \l==3, s_waitcnt vmcnt(6)
  MM \p, 0, 0
  T \l==3, POLL
  T \l==3, LD \q, 0
  T \q==\g, K4 v_mul_f32, 20, %[scale], 20
  T \p==\g, s_nop 0
  TWR \q, 0
  MM \p, 0, 1
  T \l==3, LD \q, 1
  T \l==1, LD7 \q
  T \q==\g, MED4
  MM \p, 0, 2
  T \l==3, LD \q, 2
  TWR \q, 1
  T \q==\g, K4 v_mul_f32, 24, 0xbfb8aa3b, 24
  MM \p, 0, 3
  T \l==3, LD \q, 3
  T (\l==1)||(\l==2), LD \p, 0
  T \q==\g, U4 v_exp_f32, 24, 24
  T \l==1, s_waitcnt vmcnt(13)
  T \l==4, s_waitcnt vmcnt(4)
  T \l==3, s_waitcnt vmcnt(9)
  MM \p, 1, 0
  T \l==3, LD \q, 4
  TWR \q, 2
  T \q==\g, K4 v_add_f32, 24, 1.0, 24
  MM \p, 1, 1
  T \l==3, LD \q, 5
  T (\l==1)||(\l==2), LD \p, 1
  T \q==\g, U4 v_rcp_f32, 24, 24
  v_mov_b32 v30, %[n]
  MM \p, 1, 2
  T \l==3, LD \q, 6
  TWR \q, 3
  ds_write_b32 %[vpw], v30
  ds_read_b32 v29, %[vpr]
  T \q==\g, OP4 v_mul_f32, 24, 20, 24
  MM \p, 1, 3
  T \l==3, LD \q, 7
  T \l==3, LDONE \q
  T (\l==1)||(\l==2), LD \p, 2
  s_sub_u32 %[tp], %[n], 2
  T \p==\g, TADDR %[tp]
  T \p==\g, s_cmp_lt_i32 %[tp], %[nl]
  T \p==\g, s_cbranch_scc1 .Lown\p\g\l
.Lowned\p\g\l:
  T \l==1, s_waitcnt vmcnt(13)
  T \l==4, s_waitcnt vmcnt(2)
  T \l==3, s_waitcnt vmcnt(11)
  MM \p, 2, 0
  T \q==\g, TSTORE
  s_waitcnt lgkmcnt(0)
  v_cmp_lt_i32 vcc, %[tp], v29
  s_cmp_eq_u32 vcc_lo, -1
  s_cbranch_scc0 .Lwait\p\g\l
.Lwaited\p\g\l:
  T \p==\g, RD 20, v28, 0
  T \p==\g, RD 32+16*\p, v28, 1
  T \p==\g, RD 36+16*\p, v28, 2
  T \p==\g, RD 40+16*\p, v28, 3
  MM \p, 2, 1
  T (\l==1)||(\l==2), LD \p, 3
  T \p==\g, v_add_u32 v29, 0x10000, v28
  T \p==\g, RD 44+16*\p, v29, 0
  T \p==\g, RD 24, v29, 1
  MM \p, 2, 2
  .if \q
    WFLIP
  .endif
  .if (\l==1)||(\l==2)
    s_add_u32 %[t0], %[n], PTH
    s_cmp_gt_i32 %[t0], %[av]
    s_cbranch_scc1 .Lpoll\p\g\l
.Lpolled\p\g\l:
  .elseif \l != 3
    POLL
  .endif
  MM \p, 2, 3
  T (\l==1)||(\l==2), LD \p, 4
  T \p==\g, s_waitcnt lgkmcnt(2)
  T \p==\g, OP4 v_add_f32, 20, 20, 32+16*\p
  T \p==\g, RD 32+16*\p, v29, 2
  T \l==1, s_waitcnt vmcnt(13)
  T \l==4, s_waitcnt vmcnt(1+((\q==\g)&1))
  T \l==3, s_waitcnt vmcnt(9+((\q==\g)&1))
  MM \p, 3, 0
  T \p==\g, OP4 v_add_f32, 36+16*\p, 36+16*\p, 40+16*\p
  T \p==\g, RD 40+16*\p, v29, 3
  MM \p, 3, 1
  T (\l==1)||(\l==2), LD \p, 5
  T \p==\g, OP4 v_add_f32, 20, 20, 36+16*\p
  MM \p, 3, 2
  T \p==\g, s_waitcnt lgkmcnt(2)
  T \p==\g, OP4 v_add_f32, 24, 44+16*\p, 24
  T \p==\g, OP4 v_add_f32, 20, 20, 24
  s_add_u32 %[n], %[n], 1
  s_add_u32 %[t0], %[n], 2
  MM \p, 3, 3
  T (\l==1)||(\l==2), LD \p, 6
  T \p==\g, s_waitcnt lgkmcnt(0)
  T \p==\g, OP4 v_add_f32, 24, 32+16*\p, 40+16*\p
  T \p==\g, OP4 v_add_f32, 20, 20, 24
  .if (\l==1)||(\l==2)
    s_cmp_ge_i32 %[t0], %[av]
    s_cbranch_scc1 .Lexit\p\g
    T \l==2, s_branch .Lhot\q\g\()1
  .else
    T \q==\g, s_add_u32 %[ops], %[ops], 1
    s_sub_u32 %[t0], %[n], 2-((\p==\g)&1)
    s_max_i32 %[tq], %[tq], %[t0]
    T \l==3, s_branch .Ltw\q\g
    T \l!=3, s_branch .Ltop\q\g
  .endif
.endm
.macro STUBS p, g, l
  .if (\l==1)||(\l==2)
.Lpoll\p\g\l:
    POLL
    s_branch .Lpolled\p\g\l
  .endif
.Lown\p\g\l:
  TOWNROW %[tp]
  s_branch .Lowned\p\g\l
.Lwait\p\g\l:
  ds_read_b32 v29, %[vpr]
  s_waitcnt lgkmcnt(0)
  v_cmp_lt_i32 vcc, %[tp], v29
  s_cmp_eq_u32 vcc_lo, -1
  s_cbranch_scc0 .Lwait\p\g\l
  s_branch .Lwaited\p\g\l
.endm
.macro COLD p, q, g
  STUBS \p, \g, 0
  STUBS \p, \g, 1
  STUBS \p, \g, 2
  STUBS \p, \g, 3
  STUBS \p, \g, 4
.Lexit\p\g:
  TOAVAIL
  s_add_u32 %[iss], %[n], 2
  s_cmp_lt_i32 %[iss], %[av]
  s_cbranch_scc1 .Lhot\q\g\()1
  LD7 \p
  s_mov_b32 s[68+\p], %[ops]
  s_sub_u32 s[69-\p], %[ops], 8
  s_sub_u32 %[t0], %[n], 2-((\p==\g)&1)
  s_max_i32 %[tq], %[tq], %[t0]
  s_waitcnt vmcnt(8)
  s_branch .Lhot\q\g\()0
.Ltop\p\g:
  .if TOPWAIT
    s_add_u32 %[t0], %[n], 2
    s_cmp_eq_u32 %[iss], %[t0]
    s_cbranch_scc1 .Ltw\p\g
    s_waitcnt vmcnt(0)
.Ltw\p\g:
  .endif
  TOAVAIL
  s_cmp_eq_u32 %[iss], %[n]
  s_cbranch_scc1 .Lcold\p\g
  s_add_u32 %[t0], %[n], 1
  s_cmp_lg_u32 %[iss], %[t0]
  s_cbranch_scc1 .Lt2\p\g
  s_cmp_ge_i32 %[iss], %[av]
  s_cbranch_scc1 .Lhot\p\g\()4
  s_add_u32 %[t0], %[n], NEARTH
  s_cmp_le_i32 %[av], %[t0]
  s_cbranch_scc1 .Lhot\p\g\()3
  BURST \q
.Lt2\p\g:
  s_waitcnt vmcnt(8)
  s_cmp_lt_i32 %[iss], %[av]
  s_cbranch_scc1 .Lhot\p\g\()2
  s_branch .Lhot\p\g\()0
.Lcold\p\g:
  s_cmp_ge_i32 %[iss], %[av]
  s_cbranch_scc1 .Lc1\p\g
  BURST \p
.Lc1\p\g:
  v_mov_b32 v30, %[n]
  TWR \q, 0
  TWR \q, 1
  TWR \q, 2
  TWR \q, 3
  ds_write_b32 %[vpw], v30
  .if \q==\g
    TFIN
    s_add_u32 %[ops], %[ops], 1
  .endif
  s_cmp_gt_u32 %[n], %[last]
  s_cbranch_scc1 .Lc3\p\g
  s_cmp_lg_u32 %[iss], %[n]
  s_cbranch_scc1 .Lc2\p\g
  SPIN
  BURST \p
.Lc2\p\g:
  s_cmp_ge_i32 %[iss], %[av]
  s_cbranch_scc1 .Lc3\p\g
  BURST \q
.Lc3\p\g:
  s_cmp_ge_i32 %[tq], %[n]
  s_cbranch_scc1 .Lc5\p\g
  TADDR %[tq]
  s_cmp_ge_i32 %[tq], %[nl]
  s_cbranch_scc1 .Lc4\p\g
  TOWNROW %[tq]
.Lc4\p\g:
  ds_read_b32 v29, %[vpr]
  s_waitcnt lgkmcnt(0)
  v_cmp_lt_i32 vcc, %[tq], v29
  s_cmp_eq_u32 vcc_lo, -1
  s_cbranch_scc0 .Lc4\p\g
  RD 20, v28, 0
  RD 24, v28, 1
  RD 32+16*\q, v28, 2
  RD 36+16*\q, v28, 3
  v_add_u32 v29, 0x10000, v28
  RD 40+16*\q, v29, 0
  RD 44+16*\q, v29, 1
  s_waitcnt lgkmcnt(4)
  OP4 v_add_f32, 20, 20, 24
  RD 24, v29, 2
  s_waitcnt lgkmcnt(3)
  OP4 v_add_f32, 32+16*\q, 32+16*\q, 36+16*\q
  RD 36+16*\q, v29, 3
  OP4 v_add_f32, 20, 20, 32+16*\q
  s_waitcnt lgkmcnt(2)
  OP4 v_add_f32, 40+16*\q, 40+16*\q, 44+16*\q
  OP4 v_add_f32, 20, 20, 40+16*\q
  s_waitcnt lgkmcnt(0)
  OP4 v_add_f32, 24, 24, 36+16*\q
  OP4 v_add_f32, 20, 20, 24
  TFIN
  s_add_u32 %[ops], %[ops], 1
  s_add_u32 %[tq], %[tq], 2
  s_branch .Lc3\p\g
.Lc5\p\g:
  s_cmp_gt_u32 %[n], %[last]
  s_cbranch_scc1 .Ldone
  .if \q
    WFLIP
  .endif
  s_mov_b32 %[oso], 0x40000000
  NSTAMP
  s_sub_u32 %[t0], %[ops], s[68+\p]
  s_add_u32 %[n], %[n], 1
  s_cmp_ge_u32 %[t0], 8
  s_cbranch_scc1 .Lc8\p\g
  s_cmp_ge_u32 %[t0], 2
  s_cbranch_scc1 .Lc7\p\g
  s_cmp_ge_u32 %[t0], 1
  s_cbranch_scc1 .Lc9\p\g
  MPROG \p, 0
  s_branch .Ltop\q\g
.Lc9\p\g:
  MPROG \p, 1
  s_branch .Ltop\q\g
.Lc7\p\g:
  MPROG \p, 2
  s_branch .Ltop\q\g
.Lc8\p\g:
  MPROG \p, 8
  s_branch .Ltop\q\g
.endm
.macro MAIN g
  HOT 0, 1, \g, 1
  HOT 1, 0, \g, 1
  s_branch .Lhot0\g\()1
  HOT 0, 1, \g, 2
  HOT 1, 0, \g, 2
  HOT 0, 1, \g, 0
  HOT 1, 0, \g, 0
  HOT 0, 1, \g, 3
  HOT 1, 0, \g, 3
  HOT 0, 1, \g, 4
  HOT 1, 0, \g, 4
  COLD 0, 1, \g
  COLD 1, 0, \g
.endm

  v_mov_b32 v30, 0
  ds_write_b32 %[vpw], v30
  s_waitcnt lgkmcnt(0)
  s_load_dword %[call], %[cnt], 0
  s_load_dword %[t1], %[psx], 0
  s_load_dword %[t2], %[psw], 0
  s_barrier
  v_xor_b32 v48, 32, %[wv0]
  s_cmp_eq_u32 %[cp], 0
  s_cbranch_scc1 .Lnocopy
  buffer_load_dwordx4 v[64:67], %[vcs], %[rX], 0 offen
  buffer_load_dwordx4 v[68:71], %[vcs], %[rX], 0 offen offset:64
.Lnocopy:
  WPF 0
  WPF 1
  s_cmp_eq_u32 %[cp], 0
  s_cbranch_scc0 .Lcopy1
  WDMA 0
  WPF 2
  WDMA 1
  WPF 3
.Lcopy1:
  v_mov_b32 v49, %[tr0]
  v_xor_b32 v50, 16, %[tr0]
  v_xor_b32 v51, 32, %[tr0]
  v_xor_b32 v52, 48, %[tr0]
  v_add_u32 %[wr0], 8192, %[wr0]
  s_mov_b32 %[oso], 0x40000000
  v_xor_b32 %[wr1], 64, %[wr0]
  v_xor_b32 %[wr2], 128, %[wr0]
  v_xor_b32 %[wr3], 192, %[wr0]
  v_add_u32 %[vstg2], 4096, %[vstg]
  s_mov_b32 s70, 0x42700000
  s_mov_b32 %[n], 0
  s_mov_b32 %[iss], 0
  s_mov_b32 %[ops], 0
  s_mov_b32 %[nso], %[nso0]
  s_mov_b32 %[nst], 7*65536
  s_mov_b32 %[tq], %[grp]
  s_mov_b32 %[pa], 0
  s_waitcnt lgkmcnt(0)
  s_add_u32 %[call], %[call], 1
  s_lshl_b32 %[base], %[call], 6
  v_mov_b32 %[scale], %[t1]
  v_mul_f32 %[scale], %[t2], %[scale]
  s_and_b32 %[t0], %[call], 1
  s_lshl_b32 %[t0], %[t0], 24
  s_mov_b64 s[72:73], %[aS]
  s_add_u32 s72, s72, %[t0]
  s_addc_u32 s73, s73, 0
  s_mov_b32 s74, 0x1000000
  s_mov_b32 s75, 0x20000
  s_cmp_eq_u32 %[cp], 0
  s_cbranch_scc1 .Lcopied
  v_mov_b32 v72, %[base]
  WPF 2
  WPF 3
  s_waitcnt vmcnt(4*NPF)
  buffer_store_dwordx4 v[64:67], %[vcd], s[72:75], 0 offen sc0 sc1
  buffer_store_dwordx4 v[68:71], %[vcd], s[72:75], 0 offen offset:1024 sc0 sc1
  s_waitcnt vmcnt(0)
  buffer_store_dword v72, %[vcf], %[rF], 0 offen sc0 sc1
  WDMA 0
  WDMA 1
.Lcopied:
  ASTAMP 42, 0
  s_waitcnt vmcnt(8)
  WTR 0
  ASTAMP 41, 0
  s_waitcnt lgkmcnt(0)
  WDMA 2
  s_waitcnt vmcnt(8)
  WTR 1
  s_waitcnt lgkmcnt(0)
  WDMA 3
  POLL 1
  s_waitcnt vmcnt(9)
  WTR 2
  s_waitcnt vmcnt(0)
  WTR 3
  ASTAMP 43, 0
  TOOWN
)" FAKE_AV R"(
  SPIN 1
  ASTAMP 44, 0
  v_mov_b32 v31, 0
  BURST 0
  BURST 1
  s_waitcnt lgkmcnt(0)
  ASTAMP 1, 0
  CSTAMP 48
  s_cmp_eq_u32 %[tq], 0
  s_cbranch_scc1 .Ltop00
  s_branch .Ltop01
  MAIN 0
  MAIN 1
.Ldone:
  CSTAMP 49
)"

__global__ __launch_bounds__(THREADS, 2) void ag_gemm_silu(Params) {
  extern __shared__ __attribute__((aligned(16))) unsigned char smem[];
  const int b = blockIdx.x, lane = threadIdx.x & 63;
  const int wv = (int)rfl(threadIdx.x >> 6);
  const int si = SPERM ? (b & 7) * (NSEND / 8) + (b >> 3) : b, u0 = (si * 7 + wv) * 2;
  const unsigned li = ((unsigned)u0 >> 6) & (NR - 1), un = (unsigned)u0 & 63;
  u32x16 ka;
  unsigned long long pw_;
  asm volatile("s_load_dwordx16 %0, %2, 0\n\ts_load_dwordx2 %1, %2, %3\n\ts_waitcnt lgkmcnt(0)"
               : "=&s"(ka), "=&s"(pw_) : "s"(__builtin_amdgcn_kernarg_segment_ptr()), "s"(64 + li * 8));
  Params p;
  p.me = (unsigned char*)mk64(ka[0], ka[1]); p.x = (const unsigned char*)mk64(ka[2], ka[3]); p.w = (const unsigned char*)mk64(ka[4], ka[5]);
  p.sx = (const float*)mk64(ka[6], ka[7]); p.sw = (const float*)mk64(ka[8], ka[9]); p.out = (float*)mk64(ka[10], ka[11]);
  const unsigned rank = ka[12] & (NR - 1);
  unsigned char* me = p.me;
  unsigned* cnt = (unsigned*)(me + CNT_OFF + (size_t)b * 128);
  unsigned* tmo = (unsigned*)(me + TMO_OFF);

  if (__builtin_expect(b < NSEND, 1)) {
    if (wv == 7) return;
    const unsigned svo = (lane & 15) * KD + (lane >> 4) * 16 + un * 64, dvo = lane * 16 + un * 1024;
    const int first = max(si * 14, (int)li * 64), last = min((si + 1) * 14, (int)(li + 1) * 64), ks = un / 2 + lane;
    const unsigned fo = (u0 == first && lane < (last - first) / 2) ? (ks >> 2) * 128 + (ks & 3) * 4 : 0x80000000u;
    const u32x4 rX = mk_srd(p.x, SHARD), rF = mk_srd((void*)(pw_ + FLAG_OFF + rank * FLAG_STR), FLAG_STR);
    unsigned call, so, fs, t0;
    asm volatile(R"(
.macro SLD r, d
  s_add_u32 %[t0], %[so], \d*65536
  buffer_load_dwordx4 v[32+8*\r:35+8*\r], %[svo], %[rX], %[t0] offen
  buffer_load_dwordx4 v[36+8*\r:39+8*\r], %[svo], %[rX], %[t0] offen offset:64
.endm
.macro SST r
  buffer_store_dwordx4 v[32+8*\r:35+8*\r], %[dvo], s[64:67], %[so] offen sc0 sc1
  buffer_store_dwordx4 v[36+8*\r:39+8*\r], %[dvo], s[64:67], %[so] offen offset:1024 sc0 sc1
.endm
.macro SPUB w
  s_waitcnt vmcnt(\w)
  s_barrier
  buffer_store_dword v64, %[fo], %[rF], %[fs] offen sc0 sc1
  s_add_u32 %[fs], %[fs], 1024
  s_add_u32 %[so], %[so], 65536
  v_add_u32 v64, 1, v64
.endm
  s_load_dword %[call], %[cnt], 0
  s_mov_b32 %[so], 0
  s_mov_b32 %[fs], 0
  SLD 0, 0
  SLD 1, 1
  s_mov_b64 s[64:65], %[aD]
  s_mov_b32 s66, 0x200000
  s_mov_b32 s67, 0x20000
)" TLBW R"(
  s_waitcnt lgkmcnt(0)
  s_add_u32 %[call], %[call], 1
  s_and_b32 %[t0], %[call], 1
  s_lshl_b32 %[t0], %[t0], 24
  s_add_u32 s64, s64, %[t0]
  s_addc_u32 s65, s65, 0
  s_lshl_b32 %[t0], %[call], 6
  v_mov_b32 v64, %[t0]
  v_add_u32 v64, 1, v64
  s_waitcnt vmcnt(2+)" STR(TLBN) R"()
  SST 0
  SLD 2, 2
  SLD 3, 3
  s_add_u32 %[so], %[so], 65536
  s_waitcnt vmcnt(6+)" STR(TLBN) R"()
  SST 1
  SLD 0, 3
  SPUB 8
  s_waitcnt vmcnt(7)
.Lsend:
  SST 2
  SLD 1, 3
  SPUB 7
  SST 3
  SLD 2, 3
  SPUB 7
  s_cmp_eq_u32 %[fs], 31*1024
  s_cbranch_scc1 .Lsent
  SST 0
  SLD 3, 3
  SPUB 7
  SST 1
  SLD 0, 3
  SPUB 7
  s_branch .Lsend
.Lsent:
  SPUB 0
)"      : [call] "=&s"(call), [so] "=&s"(so), [fs] "=&s"(fs), [t0] "=&s"(t0)
        : [rX] "s"(rX), [rF] "s"(rF), [cnt] "s"(cnt), [aD] "s"((pw_ + (size_t)rank * SHARD) & 0xffffffffffffull),
          [svo] "v"(svo), [dvo] "v"(dvo), [fo] "v"(fo)
        : "memory", "scc", "s64", "s65", "s66", "s67", "v32", "v33", "v34", "v35", "v36", "v37", "v38", "v39", CLOB10(4), CLOB10(5),
          "v60", "v61", "v62", "v63", "v64", "v65", "v66");
    if (threadIdx.x == 0) __hip_atomic_store(cnt, call, __ATOMIC_RELAXED, __HIP_MEMORY_SCOPE_AGENT);
    STAMP(2);
    return;
  }

  const int xcd = b & 7, slot = (b >> 3) - NSEND / 8;
  int cg, rg;
  cg = xcd * 4 + (slot & 3); rg = slot >> 2;
  const unsigned lds0 = (unsigned)(unsigned long long)(LAS unsigned char*)smem;
  const unsigned src = (rank + 1 + rg) & (NR - 1);
  const int q = lane >> 4, m = lane & 15;
  unsigned wvo0 = (lane >> 2) * NL + cg * 64 + (((lane & 3) ^ (q & 1)) * 16);
  unsigned tr0 = lds0 + wv * 16384 + (16 * q + (m >> 1)) * 64 + ((((m >> 3) & 1) | ((q & 1) << 1)) * 16) + 8 * (lane & 1);
  const unsigned vstg = lane * 16 + wv * 8192;
  const int nl = (rg < 4) ? 5 : 4;
  const unsigned vflag = lane < 4 * NPROBE ? src * FLAG_STR + (lane >> 2) * 1024 + wv * 128 + (lane & 3) * 4 : 0x80000000u;
  const unsigned vexp = (lane >> 2) + 1;
  unsigned vown = lane < 4 * nl ? rank * FLAG_STR + (rg + 7 * (lane >> 2)) * 1024 + wv * 128 + (lane & 3) * 4 : 0x80000000u;
  const unsigned item = wv * (GRID - NSEND) + (unsigned)(b - NSEND);
  unsigned vcs = (item >> 5) * BLKB + m * KD + (item & 31) * 128 + q * 16;
  unsigned vcd = rank * SHARD + item * 2048 + lane * 16;
  unsigned vcf = lane == 0 ? rank * FLAG_STR + (item >> 2) * 128 + (item & 3) * 4 : 0x80000000u;
  constexpr int LPR = MAPMODE == 0 ? 2 : MAPMODE == 3 ? 4 : 8, PL = (1024 * LPR + 223) / 224;
  const unsigned pfl = (slot * 8 + wv) * PL + lane, pfr = pfl / LPR;
  unsigned vpf = (lane < PL && pfr < 1024) ? ((pfr >> 7) * 512 + (pfr & 127)) * NL + ((cg >> 1) / LPR * LPR + pfl % LPR) * 128 : 0x80000000u;
  unsigned wr0 = lds0 + wv * 16384 + m * 256 + ((q ^ m) * 16);
  const int m_rd = 4 * (wv & 3) + q;
  const unsigned rd = lds0 + m_rd * 256 + ((m ^ m_rd) * 16);
  const unsigned vout = m_rd * (NL * 4) + cg * 256 + m * 16;
  const unsigned long long aS = (unsigned long long)me & 0xffffffffffffull;
  const u32x4 rW = mk_srd(p.w, KD * NL), rX = mk_srd(p.x, SHARD);
  const u32x4 rF = mk_srd(me + FLAG_OFF, (NR + 1) * FLAG_STR), rO = mk_srd(p.out, (unsigned)NR * MS * NL * 4u);
  unsigned call, base, n, iss, av, lim, ops, nso, nst, oso, tp, tq, pa, t0, t1, t2, t3;
  unsigned vstg2, wr1, wr2, wr3, scale;
  asm volatile(COMPUTE_ASM
      : [call] "=&s"(call), [base] "=&s"(base), [n] "=&s"(n), [iss] "=&s"(iss), [av] "=&s"(av), [lim] "=&s"(lim), [ops] "=&s"(ops),
        [nso] "=&s"(nso), [nst] "=&s"(nst), [oso] "=&s"(oso), [pa] "=&s"(pa), [tp] "=&s"(tp), [tq] "=&s"(tq), [t0] "=&s"(t0), [t1] "=&s"(t1), [t2] "=&s"(t2), [t3] "=&s"(t3),
        [wr0] "+v"(wr0), [vown] "+{v80}"(vown), [vcf] "+{v78}"(vcf), [vcd] "+{v77}"(vcd), [vcs] "+{v76}"(vcs), [tr0] "+{v75}"(tr0), [wv0] "+{v74}"(wvo0), [vpf] "+{v79}"(vpf), [vstg2] "=&v"(vstg2), [wr1] "=&v"(wr1), [wr2] "=&v"(wr2), [wr3] "=&v"(wr3), [scale] "=&v"(scale)
      : [rW] "s"(rW), [rX] "s"(rX), [rF] "s"(rF), [rO] "s"(rO), [lb] "s"(rfl(lds0 + wv * 16384)), [sw] "s"(rfl(wv * 512 * NL)),
        [nl] "s"(nl), [grp] "s"(rfl(wv >> 2)), [omask] "s"((1u << (4 * nl)) - 1), [last] "s"(nl + NBLK - 1),
        [nso0] "s"(rank * SHARD + rg * BLKB), [nso1] "s"(src * SHARD), [oso0] "s"((rank * MS + rg * 16) * (NL * 4)), [oso1] "s"((src * MS - nl * 16) * (NL * 4)),
        [aS] "s"(aS), [cw] "s"(rfl(slot * 8 + (7 - wv))), [cp] "s"(rfl(1024 - (int)item > 0 ? 1024 - item : 0)), [cnt] "s"(cnt), [tmo] "s"(tmo), [psx] "s"(p.sx), [psw] "s"(p.sw),
        [vstg] "v"(vstg), [vflag] "v"(vflag), [vexp] "v"(vexp), [rd] "v"(rd), [vpw] "v"(lds0 + PROG_OFF + wv * 4), [vpr] "v"(lds0 + PROG_OFF + (lane & 7) * 4), [vout] "v"(vout)
      : "memory", "scc", "vcc", "s64", "s65", "s66", "s67", "s68", "s69", "s70", "s71", "s72", "s73", "s74", "s75", CLOB_V);
  if (threadIdx.x == 0) __hip_atomic_store(cnt, call, __ATOMIC_RELAXED, __HIP_MEMORY_SCOPE_AGENT);
}

extern "C" void kernel_launch(int rank, int nranks, ncclComm_t comm, hipStream_t stream,
                              void* const* d_in, const int* in_sizes, int n_in,
                              void* d_out, int out_size,
                              const int* in_dtypes, int out_dtype,
                              void* const* d_out_all,
                              void* const* d_ws_all, size_t ws_bytes) {
  if (nranks != NR || rank < 0 || rank >= NR || n_in != 4 || ws_bytes < WS_NEED ||
      in_sizes[0] != (int)SHARD || in_sizes[1] != KD * NL || out_size != NR * MS * NL) {
    fprintf(stderr, "ag_gemm_silu: unexpected problem shape\n");
    return;
  }
  HIP_CHECK(hipFuncSetAttribute((const void*)ag_gemm_silu, hipFuncAttributeMaxDynamicSharedMemorySize, LDS_BYTES));
  Params p;
  p.me = (unsigned char*)d_ws_all[rank];
  for (int r = 0; r < NR; r++) p.peer[r] = (unsigned char*)d_ws_all[(rank + 1 + r) % NR];
  p.x = (const unsigned char*)d_in[0];
  p.w = (const unsigned char*)d_in[1];
  p.sx = (const float*)d_in[2];
  p.sw = (const float*)d_in[3];
  p.out = (float*)d_out;
  p.rank = rank;
  p.pad[0] = p.pad[1] = p.pad[2] = 0;
  ag_gemm_silu<<<GRID, THREADS, LDS_BYTES, stream>>>(p);
  HIP_CHECK(hipGetLastError());
}
```

```cpp
#include <hip/hip_runtime.h>
#include <rccl/rccl.h>
#include <cstdio>

#define HIP_CHECK(expr) do { hipError_t _e = (expr); if (_e != hipSuccess) { \
    fprintf(stderr, "HIP error %s at %s:%d\n", hipGetErrorString(_e), __FILE__, __LINE__); \
    return; } } while (0)

typedef int      v8i   __attribute__((ext_vector_type(8)));
typedef int      v2i   __attribute__((ext_vector_type(2)));
typedef unsigned u32x4 __attribute__((ext_vector_type(4)));
typedef float    v4f   __attribute__((ext_vector_type(4)));
#define LAS __attribute__((address_space(3)))
#define CAS __attribute__((address_space(4)))
#define DI __device__ __forceinline__

constexpr int      NR        = 8;
constexpr int      KD        = 4096, NL = 2048, MS = 512;
constexpr unsigned SHARD     = MS * KD;
constexpr unsigned BLKB      = 16 * KD;
constexpr int      NBLK      = MS / 16;
constexpr size_t   STAGE_B   = (size_t)NR * SHARD;
constexpr size_t   FLAG_OFF  = 2 * STAGE_B;
constexpr unsigned FLAG_STR  = NBLK * 1024;
constexpr size_t   CNT_OFF   = FLAG_OFF + (NR + 1) * FLAG_STR;
constexpr size_t   TMO_OFF   = CNT_OFF + 256 * 128;
constexpr size_t   DIAG_OFF  = TMO_OFF + 4096;
constexpr size_t   WS_NEED   = DIAG_OFF + 256 * 8 * 64 * 8;
constexpr int      NSEND     = 32, GRID = 256, THREADS = 512;
constexpr unsigned PROG_OFF  = 131072;
constexpr unsigned LDS_BYTES = PROG_OFF + 64;
#define DEADLINE_TICKS_LIT 100000000
constexpr int      SRD3      = 0x00020000;

#define PACE 0
#define PACE0 0
#define SD 2
#define WIN 2
#define OUT_AUX 2
#define MAPMODE 3
#define NOPF

struct Params {
  unsigned char* me;
  const unsigned char* x;
  const unsigned char* w;
  const float* sx;
  const float* sw;
  float* out;
  int rank;
  int pad[3];
  unsigned char* peer[NR];
};
typedef unsigned u32x16 __attribute__((ext_vector_type(16)));
DI unsigned long long mk64(unsigned lo, unsigned hi) { return ((unsigned long long)hi << 32) | lo; }

DI unsigned rfl(unsigned v) { return __builtin_amdgcn_readfirstlane(v); }
DI __amdgpu_buffer_rsrc_t mk_rsrc(const void* p, unsigned bytes) {
  unsigned long long a = (unsigned long long)p;
  unsigned lo = rfl((unsigned)a), hi = rfl((unsigned)(a >> 32));
  return __builtin_amdgcn_make_buffer_rsrc((void*)(((unsigned long long)hi << 32) | lo), (short)0, (int)bytes, SRD3);
}

DI u32x4 mk_srd(const void* p, unsigned bytes) {
  const unsigned long long a = (unsigned long long)p;
  return u32x4{rfl((unsigned)a), rfl((unsigned)(a >> 32) & 0xffffu), bytes, (unsigned)SRD3};
}

#define STAMP(slot) do {} while (0)

#define VMCNT(n)  asm volatile("s_waitcnt vmcnt(" #n ")" ::: "memory")
#define LGKM(n)   asm volatile("s_waitcnt lgkmcnt(" #n ")" ::: "memory")
#define LGKM0()   LGKM(0)
#define BARRIER() do { __builtin_amdgcn_s_barrier(); asm volatile("" ::: "memory"); } while (0)
#define SCHED()   __builtin_amdgcn_sched_barrier(0)

#define CLOB10(p) "v" #p "0", "v" #p "1", "v" #p "2", "v" #p "3", "v" #p "4", "v" #p "5", "v" #p "6", "v" #p "7", "v" #p "8", "v" #p "9"
#define CLOB_V "v20", "v21", "v22", "v23", "v24", "v25", "v26", "v27", "v28", "v29", CLOB10(3), CLOB10(4), CLOB10(5), CLOB10(6), "v70", "v71", "v72", "v73", "v81", "v82", "v83", "v85", "v86", "v87", "v88", "v89", CLOB10(9), CLOB10(10), CLOB10(11), CLOB10(12),   \
    CLOB10(13), CLOB10(14), CLOB10(15), CLOB10(16), CLOB10(17), CLOB10(18), CLOB10(19), CLOB10(20), CLOB10(21), CLOB10(22), CLOB10(23),        \
    CLOB10(24), "v250", "v251", "v252", "v253", "v254", "v255"
#define STR_(x) #x
#define STR(x) STR_(x)
#define OUT_BITS " nt"
#define ASTAMP ".macro ASTAMP off, idx\n.endm\n.macro CSTAMP off\n.endm\n"
#define FAKE_AV ""
#define PFLR 32
#define NPF1 0
#define NPF2 0
#define PSLEEP 8
#define NEARTH 2
#define SPERM 2
#define TOPWAIT 1
#define SSLEEP 0
#define CODEPF 0
#define TLBN 0
#define TLBW ""
#define WROT 2
#define WRFIRST 1
#define EARLYQ 0
#define NWARM 0
#define LASTW 0
#define WN 2
#define PRESL 0
#define NPROBE 4
#define PTH 5
#define STAMPN 1
#define COMPUTE_ASM ".set STAMPN, " STR(STAMPN) "\n.set NEARTH, " STR(NEARTH) "\n.set TOPWAIT, " STR(TOPWAIT) "\n.set SSLEEP, " STR(SSLEEP) "\n.set CODEPF, " STR(CODEPF) "\n.set WRFIRST, " STR(WRFIRST) "\n.set EARLYQ, " STR(EARLYQ) "\n.set NWARM, " STR(NWARM) "\n.set PRESL, " STR(PRESL) "\n.set PTH, " STR(PTH) "\n.set NPF1, " STR(NPF1) "\n.set NPF2, " STR(NPF2) "\n.set PFSTEP, 224*64/" STR(PFLR) "\n.set PSLEEP, " STR(PSLEEP) "\n" ASTAMP R"(
.macro WDMA i
  s_add_u32 %[t3], %[rot], \i
  s_and_b32 %[t3], %[t3], 3
  s_lshl_b32 %[t3], %[t3], 18
  s_add_u32 %[t3], %[t3], %[sw]
  .irp j, 0,1,2,3,4,5,6,7
    s_add_u32 m0, %[lb], ((\i)&1)*8192+\j*1024
    s_add_u32 %[t0], %[t3], \j*16*2048
    .if \j & 1
      buffer_load_dwordx4 v48, %[rW], %[t0] offen lds
    .else
      buffer_load_dwordx4 %[wv0], %[rW], %[t0] offen lds
    .endif
  .endr
.endm
.macro WPF c
  .rept \c
    v_bfe_u32 v81, %[vpf], 7, 3
    v_lshrrev_b32 v82, 10, %[vpf]
    v_lshlrev_b32 v81, 9, v81
    v_lshl_add_u32 v81, v82, 7, v81
    v_and_b32 v82, 127, %[vpf]
    v_add_u32 v81, v81, v82
    v_lshl_add_u32 v81, v81, 11, %[vpc]
    v_cmp_gt_u32 vcc, 4096, %[vpf]
    v_add_u32 %[vpf], PFSTEP, %[vpf]
    v_cndmask_b32 v81, v83, v81, vcc
    buffer_load_dwordx4 v[96:99], v81, %[rW], 0 offen
  .endr
.endm
.macro WTR1 s, i
  .irp t, 0,1,2,3
    ds_read_b64_tr_b8 v[128+32*\s+8*\t:129+32*\s+8*\t], v[49+\t] offset:(\i&1)*8192
    ds_read_b64_tr_b8 v[130+32*\s+8*\t:131+32*\s+8*\t], v[49+\t] offset:(\i&1)*8192+512
    ds_read_b64_tr_b8 v[132+32*\s+8*\t:133+32*\s+8*\t], v[49+\t] offset:(\i&1)*8192+4096
    ds_read_b64_tr_b8 v[134+32*\s+8*\t:135+32*\s+8*\t], v[49+\t] offset:(\i&1)*8192+4608
  .endr
.endm
.macro WTR i
  s_add_u32 %[t3], %[rot], \i
  s_and_b32 %[t3], %[t3], 3
  .irp s, 0,1,2,3
    s_cmp_lg_u32 %[t3], \s
    s_cbranch_scc1 .Lwt\i\s
    WTR1 \s, \i
.Lwt\i\s:
  .endr
.endm
.macro LD p, i
  .if \i < 4
    buffer_load_dwordx4 v[64+32*\p+4*\i:67+32*\p+4*\i], %[vstg], s[72:75], %[nso] offen offset:1024*\i sc0 sc1
  .else
    buffer_load_dwordx4 v[64+32*\p+4*\i:67+32*\p+4*\i], %[vstg2], s[72:75], %[nso] offen offset:1024*(\i-4) sc0 sc1
  .endif
.endm
.macro LDONE p
  s_add_u32 %[ops], %[ops], 8
  s_mov_b32 s[68+\p], %[ops]
  s_add_u32 %[iss], %[iss], 1
  s_add_u32 %[nso], %[nso], %[nst]
  s_cmp_eq_u32 %[iss], %[nl]
  s_cselect_b32 %[nso], %[nso1], %[nso]
  s_cselect_b32 %[nst], 65536, %[nst]
.endm
.macro BURST p
  .irp i, 0,1,2,3,4,5,6,7
    LD \p, \i
  .endr
  LDONE \p
.endm
.macro POLL own=0
  .if \own
    buffer_load_dword v31, %[vown], %[rF], 0 offen sc0 sc1
  .else
    s_sub_u32 %[pa], %[av], %[nl]
    s_lshl_b32 %[t1], %[pa], 10
    buffer_load_dword v31, %[vflag], %[rF], %[t1] offen sc0 sc1
  .endif
.endm
.macro TOAVAIL
  s_add_u32 %[t0], %[pa], %[base]
  v_add_u32 v30, %[t0], %[vexp]
  v_cmp_ge_u32 vcc, v31, v30
  s_ff0_i32_b32 %[t0], vcc_lo
  s_lshr_b32 %[t0], %[t0], 2
  s_add_u32 %[t0], %[t0], %[pa]
  s_min_u32 %[t0], %[t0], 32
  s_add_u32 %[t0], %[t0], %[nl]
  s_max_i32 %[av], %[av], %[t0]
.endm
.macro TOOWN
  v_cmp_le_u32 vcc, %[base], v31
  s_cmp_eq_u32 vcc_lo, %[omask]
  s_cselect_b32 %[av], %[nl], 0
.endm
.macro SPIN own=0
  s_memrealtime s[64:65]
  s_mov_b32 %[lim], 0
.Lspin\@:
  s_cmp_gt_i32 %[av], %[iss]
  s_cbranch_scc1 .Lgot\@
  .if \own
    s_sleep PSLEEP
  .elseif SSLEEP
    s_sleep SSLEEP
  .endif
  POLL \own
  s_waitcnt vmcnt(0)
  .if \own
    TOOWN
  .else
    TOAVAIL
  .endif
  s_add_u32 %[lim], %[lim], 1
  s_and_b32 %[t0], %[lim], 31
  s_cbranch_scc1 .Lspin\@
  v_mov_b32 v30, 0
  global_load_dword v29, v30, %[tmo] sc1
  s_memrealtime s[66:67]
  s_waitcnt vmcnt(0) lgkmcnt(0)
  v_readfirstlane_b32 %[t0], v29
  s_cmp_eq_u32 %[t0], %[call]
  s_cbranch_scc1 .Ldead\@
  s_sub_u32 %[t0], s66, s64
  s_cmp_lt_u32 %[t0], )" STR(DEADLINE_TICKS_LIT) R"(
  s_cbranch_scc1 .Lspin\@
.Ldead\@:
  v_mov_b32 v29, %[call]
  global_store_dword v30, v29, %[tmo] sc1
  v_mov_b32 %[scale], 0x7fc00000
  s_add_u32 %[av], %[nl], 32
.Lgot\@:
.endm
.macro NSTAMP
  .if STAMPN == 1
    ASTAMP 2, %[n]
  .elseif STAMPN == 2
    s_and_b32 %[t0], %[n], 7
    s_cbranch_scc1 .Lns\@
    ASTAMP 2, %[n]
.Lns\@:
  .endif
.endm
.macro MM p, s, t
  .if \s == 0
    v_mfma_f32_16x16x128_f8f6f4 v[48-16*\p+4*\t:51-16*\p+4*\t], v[128+8*\t:135+8*\t], v[64+32*\p:71+32*\p], 0
  .else
    v_mfma_f32_16x16x128_f8f6f4 v[48-16*\p+4*\t:51-16*\p+4*\t], v[128+32*\s+8*\t:135+32*\s+8*\t], v[64+32*\p+8*\s:71+32*\p+8*\s], v[48-16*\p+4*\t:51-16*\p+4*\t]
  .endif
.endm
.macro WARM
  .rept NWARM
    v_mfma_f32_16x16x128_f8f6f4 v[32:35], v[96:103], v[104:111], v[32:35]
    v_mfma_f32_16x16x128_f8f6f4 v[36:39], v[112:119], v[120:127], v[36:39]
    v_mfma_f32_16x16x128_f8f6f4 v[40:43], v[104:111], v[112:119], v[40:43]
    v_mfma_f32_16x16x128_f8f6f4 v[44:47], v[120:127], v[96:103], v[44:47]
  .endr
.endm
.macro MG p, s
  .irp t, 0,1,2,3
    MM \p, \s, \t
  .endr
.endm
.macro MPROG p, base
  .if EARLYQ && (\base < 8)
    POLL
    s_waitcnt vmcnt(\base+7)
    MG \p, 0
    s_waitcnt vmcnt(\base+5)
    MG \p, 1
    s_waitcnt vmcnt(\base+3)
    MG \p, 2
    s_waitcnt vmcnt(0)
    TOAVAIL
    s_cmp_ge_i32 %[iss], %[av]
    s_cbranch_scc1 .Lmp\@
    BURST (1-\p)
    s_mov_b32 s71, %[n]
.Lmp\@:
  .else
    s_waitcnt vmcnt(\base+6)
    MG \p, 0
    s_waitcnt vmcnt(\base+4)
    MG \p, 1
    s_waitcnt vmcnt(\base+2)
    MG \p, 2
    s_waitcnt vmcnt(\base)
  .endif
  POLL
  MG \p, 3
.endm
.macro TWR p, t
  .if \t == 0
    ds_write_b128 %[wr0], v[48-16*\p:51-16*\p] offset:4096*\p
  .elseif \t == 1
    ds_write_b128 %[wr1], v[52-16*\p:55-16*\p] offset:4096*\p
  .elseif \t == 2
    ds_write_b128 %[wr2], v[56-16*\p:59-16*\p] offset:4096*\p
  .else
    ds_write_b128 %[wr3], v[60-16*\p:63-16*\p] offset:4096*\p
  .endif
.endm
.macro WFLIP
  v_xor_b32 %[wr0], 8192, %[wr0]
  v_xor_b32 %[wr1], 8192, %[wr1]
  v_xor_b32 %[wr2], 8192, %[wr2]
  v_xor_b32 %[wr3], 8192, %[wr3]
.endm
.macro TADDR e
  s_and_b32 %[t1], \e, 3
  s_lshl_b32 %[t1], %[t1], 12
  v_add_u32 v28, %[t1], %[rd]
  s_lshl_b32 %[oso], \e, 17
  s_add_u32 %[oso], %[oso], %[oso1]
.endm
.macro TOWNROW e
  s_mul_i32 %[oso], \e, 7*16*8192
  s_add_u32 %[oso], %[oso], %[oso0]
  s_cmp_lt_i32 \e, 0
  s_cselect_b32 %[oso], 0x40000000, %[oso]
.endm
.macro RD r, a, v
  ds_read_b128 v[\r:\r+3], \a offset:16384*\v
.endm
.macro OP4 op, d, a, b
  .irp i, 0,1,2,3
    \op v[\d+\i], v[\a+\i], v[\b+\i]
  .endr
.endm
.macro K4 op, d, k, a
  .irp i, 0,1,2,3
    \op v[\d+\i], \k, v[\a+\i]
  .endr
.endm
.macro U4 op, d, a
  .irp i, 0,1,2,3
    \op v[\d+\i], v[\a+\i]
  .endr
.endm
.macro MED4
  .irp i, 0,1,2,3
    v_med3_f32 v[24+\i], v[20+\i], -s70, s70
  .endr
.endm
.macro TSTORE
  buffer_store_dwordx4 v[24:27], %[vout], %[rO], %[oso] offen)" OUT_BITS R"(
.endm
.macro TFIN
  K4 v_mul_f32, 20, %[scale], 20
  MED4
  K4 v_mul_f32, 24, 0xbfb8aa3b, 24
  U4 v_exp_f32, 24, 24
  K4 v_add_f32, 24, 1.0, 24
  U4 v_rcp_f32, 24, 24
  OP4 v_mul_f32, 24, 20, 24
  TSTORE
.endm
.macro T tl, x:vararg
  .if \tl
    \x
  .endif
.endm
.macro LD7 p
  LD \p, 7
  s_add_u32 %[nso], %[nso], %[nst]
  s_add_u32 %[t0], %[n], 2
  s_cmp_eq_u32 %[t0], %[nl]
  s_cselect_b32 %[nso], %[nso1], %[nso]
  s_cselect_b32 %[nst], 65536, %[nst]
.endm
.macro HOT p, q, g, l
.Lhot\p\g\l:
  NSTAMP
  T \l==1, s_waitcnt vmcnt(13)
  T \l==4, s_waitcnt vmcnt(6)
  T \l==3, s_waitcnt vmcnt(6)
  MM \p, 0, 0
  T \l==3, POLL
  T \l==3, LD \q, 0
  T \q==\g, K4 v_mul_f32, 20, %[scale], 20
  T \p==\g, s_nop 0
  TWR \q, 0
  MM \p, 0, 1
  T \l==3, LD \q, 1
  T \l==1, LD7 \q
  T \q==\g, MED4
  MM \p, 0, 2
  T \l==3, LD \q, 2
  TWR \q, 1
  T \q==\g, K4 v_mul_f32, 24, 0xbfb8aa3b, 24
  MM \p, 0, 3
  T \l==3, LD \q, 3
  T (\l==1)||(\l==2), LD \p, 0
  T \q==\g, U4 v_exp_f32, 24, 24
  T \l==1, s_waitcnt vmcnt(13)
  T \l==4, s_waitcnt vmcnt(4)
  T \l==3, s_waitcnt vmcnt(9)
  MM \p, 1, 0
  T \l==3, LD \q, 4
  TWR \q, 2
  T \q==\g, K4 v_add_f32, 24, 1.0, 24
  MM \p, 1, 1
  T \l==3, LD \q, 5
  T (\l==1)||(\l==2), LD \p, 1
  T \q==\g, U4 v_rcp_f32, 24, 24
  v_mov_b32 v30, %[n]
  MM \p, 1, 2
  T \l==3, LD \q, 6
  TWR \q, 3
  ds_write_b32 %[vpw], v30
  ds_read_b32 v29, %[vpr]
  T \q==\g, OP4 v_mul_f32, 24, 20, 24
  MM \p, 1, 3
  T \l==3, LD \q, 7
  T \l==3, LDONE \q
  T (\l==1)||(\l==2), LD \p, 2
  s_sub_u32 %[tp], %[n], 2
  T \p==\g, TADDR %[tp]
  T \p==\g, s_cmp_lt_i32 %[tp], %[nl]
  T \p==\g, s_cbranch_scc1 .Lown\p\g\l
.Lowned\p\g\l:
  T \l==1, s_waitcnt vmcnt(13)
  T \l==4, s_waitcnt vmcnt(2)
  T \l==3, s_waitcnt vmcnt(11)
  MM \p, 2, 0
  T \q==\g, TSTORE
  s_waitcnt lgkmcnt(0)
  v_cmp_lt_i32 vcc, %[tp], v29
  s_cmp_eq_u32 vcc_lo, -1
  s_cbranch_scc0 .Lwait\p\g\l
.Lwaited\p\g\l:
  T \p==\g, RD 20, v28, 0
  T \p==\g, RD 32+16*\p, v28, 1
  T \p==\g, RD 36+16*\p, v28, 2
  T \p==\g, RD 40+16*\p, v28, 3
  MM \p, 2, 1
  T (\l==1)||(\l==2), LD \p, 3
  T \p==\g, v_add_u32 v29, 0x10000, v28
  T \p==\g, RD 44+16*\p, v29, 0
  T \p==\g, RD 24, v29, 1
  MM \p, 2, 2
  .if \q
    WFLIP
  .endif
  .if (\l==1)||(\l==2)
    s_add_u32 %[t0], %[n], PTH
    s_cmp_gt_i32 %[t0], %[av]
    s_cbranch_scc1 .Lpoll\p\g\l
.Lpolled\p\g\l:
  .elseif \l != 3
    POLL
  .endif
  MM \p, 2, 3
  T (\l==1)||(\l==2), LD \p, 4
  T \p==\g, s_waitcnt lgkmcnt(2)
  T \p==\g, OP4 v_add_f32, 20, 20, 32+16*\p
  T \p==\g, RD 32+16*\p, v29, 2
  T \l==1, s_waitcnt vmcnt(13)
  T \l==4, s_waitcnt vmcnt(1+((\q==\g)&1))
  T \l==3, s_waitcnt vmcnt(9+((\q==\g)&1))
  MM \p, 3, 0
  T \p==\g, OP4 v_add_f32, 36+16*\p, 36+16*\p, 40+16*\p
  T \p==\g, RD 40+16*\p, v29, 3
  MM \p, 3, 1
  T (\l==1)||(\l==2), LD \p, 5
  T \p==\g, OP4 v_add_f32, 20, 20, 36+16*\p
  MM \p, 3, 2
  T \p==\g, s_waitcnt lgkmcnt(2)
  T \p==\g, OP4 v_add_f32, 24, 44+16*\p, 24
  T \p==\g, OP4 v_add_f32, 20, 20, 24
  s_add_u32 %[n], %[n], 1
  s_add_u32 %[t0], %[n], 2
  MM \p, 3, 3
  T (\l==1)||(\l==2), LD \p, 6
  T \p==\g, s_waitcnt lgkmcnt(0)
  T \p==\g, OP4 v_add_f32, 24, 32+16*\p, 40+16*\p
  T \p==\g, OP4 v_add_f32, 20, 20, 24
  .if (\l==1)||(\l==2)
    s_cmp_ge_i32 %[t0], %[av]
    s_cbranch_scc1 .Lexit\p\g
    T \l==2, s_branch .Lhot\q\g\()1
  .else
    T \q==\g, s_add_u32 %[ops], %[ops], 1
    s_sub_u32 %[t0], %[n], 2-((\p==\g)&1)
    s_max_i32 %[tq], %[tq], %[t0]
    T \l==3, s_branch .Ltw\q\g
    T \l!=3, s_branch .Ltop\q\g
  .endif
.endm
.macro STUBS p, g, l
  .if (\l==1)||(\l==2)
.Lpoll\p\g\l:
    POLL
    s_branch .Lpolled\p\g\l
  .endif
.Lown\p\g\l:
  TOWNROW %[tp]
  s_branch .Lowned\p\g\l
.Lwait\p\g\l:
  ds_read_b32 v29, %[vpr]
  s_waitcnt lgkmcnt(0)
  v_cmp_lt_i32 vcc, %[tp], v29
  s_cmp_eq_u32 vcc_lo, -1
  s_cbranch_scc0 .Lwait\p\g\l
  s_branch .Lwaited\p\g\l
.endm
.macro COLD p, q, g
  STUBS \p, \g, 0
  STUBS \p, \g, 1
  STUBS \p, \g, 2
  STUBS \p, \g, 3
  STUBS \p, \g, 4
.Lexit\p\g:
  TOAVAIL
  s_add_u32 %[iss], %[n], 2
  s_cmp_lt_i32 %[iss], %[av]
  s_cbranch_scc1 .Lhot\q\g\()1
  LD7 \p
  s_mov_b32 s[68+\p], %[ops]
  s_sub_u32 s[69-\p], %[ops], 8
  s_sub_u32 %[t0], %[n], 2-((\p==\g)&1)
  s_max_i32 %[tq], %[tq], %[t0]
  s_waitcnt vmcnt(8)
  s_branch .Lhot\q\g\()0
.Ltop\p\g:
  .if WRFIRST
    s_nop 7
    s_cmp_eq_u32 %[iss], %[n]
    s_cbranch_scc1 .Lc1\p\g
    .if EARLYQ > 1
      s_cmp_eq_u32 s71, %[n]
      s_cbranch_scc1 .Lc0\p\g
    .endif
  .endif
  .if TOPWAIT
    s_add_u32 %[t0], %[n], 2
    s_cmp_eq_u32 %[iss], %[t0]
    s_cbranch_scc1 .Ltw\p\g
    s_waitcnt vmcnt(0)
.Ltw\p\g:
  .endif
  TOAVAIL
  s_cmp_eq_u32 %[iss], %[n]
  s_cbranch_scc1 .Lcold\p\g
  s_add_u32 %[t0], %[n], 1
  s_cmp_lg_u32 %[iss], %[t0]
  s_cbranch_scc1 .Lt2\p\g
  s_cmp_ge_i32 %[iss], %[av]
  s_cbranch_scc1 .Lhot\p\g\()4
  s_add_u32 %[t0], %[n], NEARTH
  s_cmp_le_i32 %[av], %[t0]
  s_cbranch_scc1 .Lhot\p\g\()3
  BURST \q
.Lt2\p\g:
  s_waitcnt vmcnt(8)
  s_cmp_lt_i32 %[iss], %[av]
  s_cbranch_scc1 .Lhot\p\g\()2
  s_branch .Lhot\p\g\()0
.Lcold\p\g:
  s_cmp_ge_i32 %[iss], %[av]
  s_cbranch_scc1 .Lc1\p\g
  BURST \p
  s_branch .Lc1\p\g
.Lc0\p\g:
  TOAVAIL
.Lc1\p\g:
  v_mov_b32 v30, %[n]
  TWR \q, 0
  TWR \q, 1
  TWR \q, 2
  TWR \q, 3
  ds_write_b32 %[vpw], v30
  .if \q==\g
    TFIN
    s_add_u32 %[ops], %[ops], 1
  .endif
  s_cmp_gt_u32 %[n], %[last]
  s_cbranch_scc1 .Lc3\p\g
  s_cmp_lg_u32 %[iss], %[n]
  s_cbranch_scc1 .Lc2\p\g
  .if WRFIRST
    s_waitcnt vmcnt((\q==\g)&1)
    TOAVAIL
  .endif
  SPIN
  BURST \p
.Lc2\p\g:
  s_cmp_ge_i32 %[iss], %[av]
  s_cbranch_scc1 .Lc3\p\g
  BURST \q
.Lc3\p\g:
  s_cmp_ge_i32 %[tq], %[n]
  s_cbranch_scc1 .Lc5\p\g
  TADDR %[tq]
  s_cmp_ge_i32 %[tq], %[nl]
  s_cbranch_scc1 .Lc4\p\g
  TOWNROW %[tq]
.Lc4\p\g:
  ds_read_b32 v29, %[vpr]
  s_waitcnt lgkmcnt(0)
  v_cmp_lt_i32 vcc, %[tq], v29
  s_cmp_eq_u32 vcc_lo, -1
  s_cbranch_scc0 .Lc4\p\g
  RD 20, v28, 0
  RD 24, v28, 1
  RD 32+16*\q, v28, 2
  RD 36+16*\q, v28, 3
  v_add_u32 v29, 0x10000, v28
  RD 40+16*\q, v29, 0
  RD 44+16*\q, v29, 1
  s_waitcnt lgkmcnt(4)
  OP4 v_add_f32, 20, 20, 24
  RD 24, v29, 2
  s_waitcnt lgkmcnt(3)
  OP4 v_add_f32, 32+16*\q, 32+16*\q, 36+16*\q
  RD 36+16*\q, v29, 3
  OP4 v_add_f32, 20, 20, 32+16*\q
  s_waitcnt lgkmcnt(2)
  OP4 v_add_f32, 40+16*\q, 40+16*\q, 44+16*\q
  OP4 v_add_f32, 20, 20, 40+16*\q
  s_waitcnt lgkmcnt(0)
  OP4 v_add_f32, 24, 24, 36+16*\q
  OP4 v_add_f32, 20, 20, 24
  TFIN
  s_add_u32 %[ops], %[ops], 1
  s_add_u32 %[tq], %[tq], 2
  s_branch .Lc3\p\g
.Lc5\p\g:
  s_cmp_gt_u32 %[n], %[last]
  s_cbranch_scc1 .Ldone
  .if \q
    WFLIP
  .endif
  s_mov_b32 %[oso], 0x40000000
  NSTAMP
  s_sub_u32 %[t0], %[ops], s[68+\p]
  s_add_u32 %[n], %[n], 1
  s_cmp_ge_u32 %[t0], 8
  s_cbranch_scc1 .Lc8\p\g
  s_cmp_ge_u32 %[t0], 2
  s_cbranch_scc1 .Lc7\p\g
  s_cmp_ge_u32 %[t0], 1
  s_cbranch_scc1 .Lc9\p\g
  MPROG \p, 0
  s_branch .Ltop\q\g
.Lc9\p\g:
  MPROG \p, 1
  s_branch .Ltop\q\g
.Lc7\p\g:
  MPROG \p, 2
  s_branch .Ltop\q\g
.Lc8\p\g:
  MPROG \p, 8
  s_branch .Ltop\q\g
.endm
.macro MAIN g
  HOT 0, 1, \g, 1
  HOT 1, 0, \g, 1
  s_branch .Lhot0\g\()1
  HOT 0, 1, \g, 2
  HOT 1, 0, \g, 2
  HOT 0, 1, \g, 0
  HOT 1, 0, \g, 0
  HOT 0, 1, \g, 3
  HOT 1, 0, \g, 3
  HOT 0, 1, \g, 4
  HOT 1, 0, \g, 4
  COLD 0, 1, \g
  COLD 1, 0, \g
.endm

  v_mov_b32 v30, 0
  ds_write_b32 %[vpw], v30
  s_waitcnt lgkmcnt(0)
  s_load_dword %[call], %[cnt], 0
  s_load_dword %[t1], %[psx], 0
  s_load_dword %[t2], %[psw], 0
  .if PRESL
    s_sleep PRESL
  .endif
  s_barrier
  v_xor_b32 v48, 32, %[wv0]
  s_cmp_lg_u32 %[grp], 0
  s_cbranch_scc1 .Lnocopy
  buffer_load_dwordx4 v[64:67], %[vcs], %[rX], 0 offen
  buffer_load_dwordx4 v[68:71], %[vcs], %[rX], 0 offen offset:64
  buffer_load_dwordx4 v[88:91], %[vcs], %[rX], %[c2s] offen
  buffer_load_dwordx4 v[92:95], %[vcs], %[rX], %[c2s] offen offset:64
.Lnocopy:
  .if CODEPF
    s_cmp_ge_u32 %[cw], (.Ldone - .Lpc) / 1024
    s_cbranch_scc1 .Lnocode
    s_getpc_b64 s[64:65]
.Lpc:
    s_lshl_b32 %[t0], %[cw], 10
    v_and_b32 v81, 1023, %[vstg]
    v_add_u32 v81, %[t0], v81
    global_load_dwordx4 v[100:103], v81, s[64:65]
.Lnocode:
  .endif
  v_mov_b32 v83, 0x80000000
  WPF NPF1
  s_cmp_eq_u32 %[grp], 0
  s_cbranch_scc1 .Lcopy1
  WDMA 0
  WPF NPF2
  WDMA 1
.Lcopy1:
  v_mov_b32 v49, %[tr0]
  v_xor_b32 v50, 16, %[tr0]
  v_xor_b32 v51, 32, %[tr0]
  v_xor_b32 v52, 48, %[tr0]
  v_add_u32 %[wr0], 8192, %[wr0]
  s_mov_b32 %[oso], 0x40000000
  v_xor_b32 %[wr1], 64, %[wr0]
  v_xor_b32 %[wr2], 128, %[wr0]
  v_xor_b32 %[wr3], 192, %[wr0]
  v_add_u32 %[vstg2], 4096, %[vstg]
  s_mov_b32 s70, 0x42700000
  s_mov_b32 %[n], 0
  s_mov_b32 %[iss], 0
  s_mov_b32 %[ops], 0
  s_mov_b32 %[nso], %[nso0]
  s_mov_b32 %[nst], 7*65536
  s_mov_b32 %[tq], %[grp]
  s_mov_b32 %[pa], 0
  s_mov_b32 s71, -1
  s_waitcnt lgkmcnt(0)
  s_add_u32 %[call], %[call], 1
  s_lshl_b32 %[base], %[call], 6
  v_mov_b32 %[scale], %[t1]
  v_mul_f32 %[scale], %[t2], %[scale]
  s_and_b32 %[t0], %[call], 1
  s_lshl_b32 %[t0], %[t0], 24
  s_mov_b64 s[72:73], %[aS]
  s_add_u32 s72, s72, %[t0]
  s_addc_u32 s73, s73, 0
  s_mov_b32 s74, 0x1000000
  s_mov_b32 s75, 0x20000
  s_cmp_lg_u32 %[grp], 0
  s_cbranch_scc1 .Lcopied
  v_mov_b32 v72, %[base]
  s_add_u32 %[t0], %[c2s], 1024
  s_waitcnt vmcnt(NPF1)
  buffer_store_dwordx4 v[64:67], %[vcd], s[72:75], 0 offen sc0 sc1
  buffer_store_dwordx4 v[68:71], %[vcd], s[72:75], 0 offen offset:1024 sc0 sc1
  buffer_store_dwordx4 v[88:91], %[vcd], s[72:75], %[c2s] offen sc0 sc1
  buffer_store_dwordx4 v[92:95], %[vcd], s[72:75], %[t0] offen sc0 sc1
  WDMA 0
  WPF NPF2
  WDMA 1
  s_waitcnt vmcnt(16+NPF2)
  buffer_store_dword v72, %[vcf], %[rF], 0 offen sc0 sc1
.Lcopied:
  ASTAMP 42, 0
  WARM
  s_waitcnt vmcnt(8)
  WTR 0
  ASTAMP 41, 0
  s_waitcnt lgkmcnt(0)
  WDMA 2
  WARM
  s_waitcnt vmcnt(8)
  WTR 1
  s_waitcnt lgkmcnt(0)
  WDMA 3
  POLL 1
  WARM
  s_waitcnt vmcnt(9)
  WTR 2
  s_waitcnt vmcnt(0)
  WTR 3
  ASTAMP 43, 0
  TOOWN
)" FAKE_AV R"(
  SPIN 1
  ASTAMP 44, 0
  v_mov_b32 v31, 0
  BURST 0
  BURST 1
  s_waitcnt lgkmcnt(0)
  ASTAMP 1, 0
  CSTAMP 48
  s_cmp_eq_u32 %[tq], 0
  s_cbranch_scc1 .Ltop00
  s_branch .Ltop01
  MAIN 0
  MAIN 1
.Ldone:
  CSTAMP 49
)"

__global__ __launch_bounds__(THREADS, 2) void ag_gemm_silu(Params) {
  extern __shared__ __attribute__((aligned(16))) unsigned char smem[];
  const int b = blockIdx.x, lane = threadIdx.x & 63;
  const int wv = (int)rfl(threadIdx.x >> 6);
  const int si = SPERM ? (b & 7) * (NSEND / 8) + (b >> 3) : b, u0 = (si * 7 + wv) * 2;
  const unsigned li = b & 7, un = (((b >> 3) * 8 + wv) * 2) & 63;
  u32x16 ka;
  unsigned long long pw_;
  asm volatile("s_load_dwordx16 %0, %2, 0\n\ts_load_dwordx2 %1, %2, %3\n\ts_waitcnt lgkmcnt(0)"
               : "=&s"(ka), "=&s"(pw_) : "s"(__builtin_amdgcn_kernarg_segment_ptr()), "s"(64 + li * 8));
  Params p;
  p.me = (unsigned char*)mk64(ka[0], ka[1]); p.x = (const unsigned char*)mk64(ka[2], ka[3]); p.w = (const unsigned char*)mk64(ka[4], ka[5]);
  p.sx = (const float*)mk64(ka[6], ka[7]); p.sw = (const float*)mk64(ka[8], ka[9]); p.out = (float*)mk64(ka[10], ka[11]);
  const unsigned rank = ka[12] & (NR - 1);
  unsigned char* me = p.me;
  unsigned* cnt = (unsigned*)(me + CNT_OFF + (size_t)b * 128);
  unsigned* tmo = (unsigned*)(me + TMO_OFF);

  if (__builtin_expect(b < NSEND, 1)) {
    if (li == 7) {
      if (threadIdx.x == 0) __hip_atomic_store(cnt, *(const CAS unsigned*)cnt + 1, __ATOMIC_RELAXED, __HIP_MEMORY_SCOPE_AGENT);
      return;
    }
    const unsigned svo = (lane & 15) * KD + (lane >> 4) * 16 + un * 64, dvo = lane * 16 + un * 1024;
    const int first = max(si * 14, (int)li * 64), last = min((si + 1) * 14, (int)(li + 1) * 64), ks = un / 2 + lane;
    const unsigned fo = (SPERM == 2 ? (wv == 0 && lane < 8) : (u0 == first && lane < (last - first) / 2)) ? (ks >> 2) * 128 + (ks & 3) * 4 : 0x80000000u;
    const u32x4 rX = mk_srd(p.x, SHARD), rF = mk_srd((void*)(pw_ + FLAG_OFF + rank * FLAG_STR), FLAG_STR);
    unsigned call, so, fs, t0;
    asm volatile(".set WN, " STR(WN) "\n.set LASTW, " STR(LASTW) "\n.set PACEQ, " STR(PACE) "*16/10\n.set PACEB, " STR(PACE0) "*16/10\n" R"(
.macro SLD r, d
  s_add_u32 %[t0], %[so], \d*65536
  buffer_load_dwordx4 v[32+8*\r:35+8*\r], %[svo], %[rX], %[t0] offen
  buffer_load_dwordx4 v[36+8*\r:39+8*\r], %[svo], %[rX], %[t0] offen offset:64
.endm
.macro SST r
  .if PACEQ
.Lpc\@:
    s_memrealtime s[68:69]
    s_waitcnt lgkmcnt(0)
    s_lshl_b32 s68, s68, 4
    s_sub_u32 s68, s68, s70
    s_cmp_lt_i32 s68, 0
    s_cbranch_scc1 .Lpc\@
    s_add_u32 s70, s70, PACEQ
  .endif
  buffer_store_dwordx4 v[32+8*\r:35+8*\r], %[dvo], s[64:67], %[so] offen sc0 sc1
  buffer_store_dwordx4 v[36+8*\r:39+8*\r], %[dvo], s[64:67], %[so] offen offset:1024 sc0 sc1
.endm
.macro SPUB w
  s_waitcnt vmcnt(\w)
  s_barrier
  buffer_store_dword v64, %[fo], %[rF], %[fs] offen sc0 sc1
  s_add_u32 %[fs], %[fs], 1024
  s_add_u32 %[so], %[so], 65536
  v_add_u32 v64, 1, v64
.endm
  s_load_dword %[call], %[cnt], 0
  s_mov_b32 %[so], 0
  s_mov_b32 %[fs], 0
  SLD 0, 0
  SLD 1, 1
  s_mov_b64 s[64:65], %[aD]
  s_mov_b32 s66, 0x200000
  s_mov_b32 s67, 0x20000
)" TLBW R"(
  s_waitcnt lgkmcnt(0)
  s_add_u32 %[call], %[call], 1
  s_and_b32 %[t0], %[call], 1
  s_lshl_b32 %[t0], %[t0], 24
  s_add_u32 s64, s64, %[t0]
  s_addc_u32 s65, s65, 0
  s_lshl_b32 %[t0], %[call], 6
  v_mov_b32 v64, %[t0]
  v_add_u32 v64, 1, v64
  s_waitcnt vmcnt(2+)" STR(TLBN) R"()
  .if PACEQ
    s_memrealtime s[70:71]
    s_waitcnt lgkmcnt(0)
    s_lshl_b32 s70, s70, 4
    s_sub_u32 s70, s70, PACEB
  .endif
  SST 0
  SLD 2, 2
  SLD 3, 3
  s_add_u32 %[so], %[so], 65536
  s_waitcnt vmcnt(6+)" STR(TLBN) R"()
  .if WN == 2
  SST 1
  SLD 0, 3
  SPUB 8
  s_waitcnt vmcnt(7)
.Lsend:
  SST 2
  SLD 1, 3
  SPUB 7
  SST 3
  SLD 2, 3
  SPUB 7
  s_cmp_eq_u32 %[fs], 31*1024
  s_cbranch_scc1 .Lsent
  SST 0
  SLD 3, 3
  SPUB 7
  SST 1
  SLD 0, 3
  SPUB 7
  s_branch .Lsend
  .else
  SST 1
  SLD 0, 3
  s_add_u32 %[so], %[so], 65536
  s_waitcnt vmcnt(6)
  SST 2
  SLD 1, 3
  SPUB 12
  s_waitcnt vmcnt(9)
  SST 3
  SLD 2, 3
  SPUB 11
.Lsend:
  s_waitcnt vmcnt(10)
  SST 0
  SLD 3, 3
  SPUB 12
  s_waitcnt vmcnt(10)
  SST 1
  SLD 0, 3
  SPUB 12
  s_waitcnt vmcnt(10)
  SST 2
  SLD 1, 3
  SPUB 12
  s_waitcnt vmcnt(10)
  SST 3
  SLD 2, 3
  SPUB 12
  s_cmp_lg_u32 %[fs], 30*1024
  s_cbranch_scc1 .Lsend
  SPUB 8
  .endif
.Lsent:
  .if LASTW
    s_waitcnt vmcnt(0)
    buffer_store_dword v64, %[fo1], %[rF], %[fs] offen sc0 sc1
  .else
    SPUB 0
  .endif
)"      : [call] "=&s"(call), [so] "=&s"(so), [fs] "=&s"(fs), [t0] "=&s"(t0)
        : [rX] "s"(rX), [rF] "s"(rF), [cnt] "s"(cnt), [aD] "s"((pw_ + (size_t)rank * SHARD) & 0xffffffffffffull),
          [svo] "v"(svo), [dvo] "v"(dvo), [fo] "v"(fo), [fo1] "v"(lane == 0 ? (ks >> 2) * 128 + (ks & 3) * 4 : 0x80000000u)
        : "memory", "scc", "s64", "s65", "s66", "s67", "s68", "s69", "s70", "s71", "v32", "v33", "v34", "v35", "v36", "v37", "v38", "v39", CLOB10(4), CLOB10(5),
          "v60", "v61", "v62", "v63", "v64", "v65", "v66");
    if (threadIdx.x == 0) __hip_atomic_store(cnt, call, __ATOMIC_RELAXED, __HIP_MEMORY_SCOPE_AGENT);
    STAMP(2);
    return;
  }

  const int xcd = b & 7, slot = (b >> 3) - NSEND / 8;
  int cg, rg;
  { const int id = (xcd >> 2) * 28 + slot; cg = (xcd & 3) * 8 + (id & 7); rg = id >> 3; }
  const unsigned lds0 = (unsigned)(unsigned long long)(LAS unsigned char*)smem;
  const unsigned src = (rank + 1 + rg) & (NR - 1);
  const int q = lane >> 4, m = lane & 15;
  unsigned wvo0 = (lane >> 2) * NL + cg * 64 + (((lane & 3) ^ (q & 1)) * 16);
  unsigned tr0 = lds0 + wv * 16384 + (16 * q + (m >> 1)) * 64 + ((((m >> 3) & 1) | ((q & 1) << 1)) * 16) + 8 * (lane & 1);
  const unsigned vstg = lane * 16 + wv * 8192;
  const int nl = (rg < 4) ? 5 : 4;
  const unsigned vflag = lane < 4 * NPROBE ? src * FLAG_STR + (lane >> 2) * 1024 + wv * 128 + (lane & 3) * 4 : 0x80000000u;
  const unsigned vexp = (lane >> 2) + 1;
  unsigned vown = lane < 4 * nl ? rank * FLAG_STR + (rg + 7 * (lane >> 2)) * 1024 + wv * 128 + (lane & 3) * 4 : 0x80000000u;
  const unsigned item = (unsigned)(b - NSEND) * 4 + (wv & 3), c2s = item < 128 ? 28 * BLKB : 0x80000000u;
  unsigned vcs = (item >> 5) * BLKB + m * KD + (item & 31) * 128 + q * 16;
  unsigned vcd = rank * SHARD + item * 2048 + lane * 16;
  unsigned vcf = (lane == 0 || (lane == 1 && item < 128)) ? rank * FLAG_STR + ((item >> 2) + lane * 224) * 128 + (item & 3) * 4 : 0x80000000u;
  unsigned vpf = (slot * 8 + wv) * (64 / PFLR) + lane / PFLR;
  unsigned vpc = cg * 64 / (PFLR * 16) * (PFLR * 16) + lane % PFLR * 16;
  unsigned wr0 = lds0 + wv * 16384 + m * 256 + ((q ^ m) * 16);
  const int m_rd = 4 * (wv & 3) + q;
  const unsigned rd = lds0 + m_rd * 256 + ((m ^ m_rd) * 16);
  const unsigned vout = m_rd * (NL * 4) + cg * 256 + m * 16;
  const unsigned long long aS = (unsigned long long)me & 0xffffffffffffull;
  const u32x4 rW = mk_srd(p.w, KD * NL), rX = mk_srd(p.x, SHARD);
  const u32x4 rF = mk_srd(me + FLAG_OFF, (NR + 1) * FLAG_STR), rO = mk_srd(p.out, (unsigned)NR * MS * NL * 4u);
  unsigned call, base, n, iss, av, lim, ops, nso, nst, oso, tp, tq, pa, t0, t1, t2, t3;
  unsigned vstg2, wr1, wr2, wr3, scale;
  asm volatile(COMPUTE_ASM
      : [call] "=&s"(call), [base] "=&s"(base), [n] "=&s"(n), [iss] "=&s"(iss), [av] "=&s"(av), [lim] "=&s"(lim), [ops] "=&s"(ops),
        [nso] "=&s"(nso), [nst] "=&s"(nst), [oso] "=&s"(oso), [pa] "=&s"(pa), [tp] "=&s"(tp), [tq] "=&s"(tq), [t0] "=&s"(t0), [t1] "=&s"(t1), [t2] "=&s"(t2), [t3] "=&s"(t3),
        [wr0] "+v"(wr0), [vown] "+{v80}"(vown), [vcf] "+{v78}"(vcf), [vcd] "+{v77}"(vcd), [vcs] "+{v76}"(vcs), [tr0] "+{v75}"(tr0), [wv0] "+{v74}"(wvo0), [vpf] "+{v79}"(vpf), [vpc] "+{v84}"(vpc), [vstg2] "=&v"(vstg2), [wr1] "=&v"(wr1), [wr2] "=&v"(wr2), [wr3] "=&v"(wr3), [scale] "=&v"(scale)
      : [rW] "s"(rW), [rX] "s"(rX), [rF] "s"(rF), [rO] "s"(rO), [lb] "s"(rfl(lds0 + wv * 16384)), [sw] "s"(rfl(wv * 512 * NL)),
        [nl] "s"(nl), [grp] "s"(rfl(wv >> 2)), [omask] "s"((1u << (4 * nl)) - 1), [last] "s"(nl + NBLK - 1),
        [nso0] "s"(rank * SHARD + rg * BLKB), [nso1] "s"(src * SHARD), [oso0] "s"((rank * MS + rg * 16) * (NL * 4)), [oso1] "s"((src * MS - nl * 16) * (NL * 4)),
        [aS] "s"(aS), [rot] "s"(WROT ? (rg + (WROT > 1 ? (cg & 1) * 2 : 0)) & 3 : 0), [cw] "s"(rfl(slot * 8 + (7 - wv))), [c2s] "s"(rfl(c2s)), [cnt] "s"(cnt), [tmo] "s"(tmo), [psx] "s"(p.sx), [psw] "s"(p.sw),
        [vstg] "v"(vstg), [vflag] "v"(vflag), [vexp] "v"(vexp), [rd] "v"(rd), [vpw] "v"(lds0 + PROG_OFF + wv * 4), [vpr] "v"(lds0 + PROG_OFF + (lane & 7) * 4), [vout] "v"(vout)
      : "memory", "scc", "vcc", "s64", "s65", "s66", "s67", "s68", "s69", "s70", "s71", "s72", "s73", "s74", "s75", CLOB_V);
  if (threadIdx.x == 0) __hip_atomic_store(cnt, call, __ATOMIC_RELAXED, __HIP_MEMORY_SCOPE_AGENT);
}

extern "C" void kernel_launch(int rank, int nranks, ncclComm_t comm, hipStream_t stream,
                              void* const* d_in, const int* in_sizes, int n_in,
                              void* d_out, int out_size,
                              const int* in_dtypes, int out_dtype,
                              void* const* d_out_all,
                              void* const* d_ws_all, size_t ws_bytes) {
  if (nranks != NR || rank < 0 || rank >= NR || n_in != 4 || ws_bytes < WS_NEED ||
      in_sizes[0] != (int)SHARD || in_sizes[1] != KD * NL || out_size != NR * MS * NL) {
    fprintf(stderr, "ag_gemm_silu: unexpected problem shape\n");
    return;
  }
  HIP_CHECK(hipFuncSetAttribute((const void*)ag_gemm_silu, hipFuncAttributeMaxDynamicSharedMemorySize, LDS_BYTES));
  Params p;
  p.me = (unsigned char*)d_ws_all[rank];
  for (int r = 0; r < NR; r++) p.peer[r] = (unsigned char*)d_ws_all[(rank + 1 + r) % NR];
  p.x = (const unsigned char*)d_in[0];
  p.w = (const unsigned char*)d_in[1];
  p.sx = (const float*)d_in[2];
  p.sw = (const float*)d_in[3];
  p.out = (float*)d_out;
  p.rank = rank;
  p.pad[0] = p.pad[1] = p.pad[2] = 0;
  ag_gemm_silu<<<GRID, THREADS, LDS_BYTES, stream>>>(p);
  HIP_CHECK(hipGetLastError());
}
```
